```python
import numpy as np

kernel_name = "gpu_gemm_rs_m4096_k4096_n4096_f32_q8"


ABSMAX_RTOL = 0.02

COMPUTE_DTYPE = "float32"
EPILOGUE = "quant_int8"
IN_QUANT = None
OUT_QUANT = "int8"
IN_DTYPES = {"x": "float32", "w_mat": "float32"}
WORLD_SIZE = 8

def setup_inputs():

    x = np.random.randn(4096, 4096).astype(np.float32)
    w_mat = (np.random.randn(4096, 4096) / np.sqrt(4096)).astype(np.float32)
    return {"x": x, "w_mat": w_mat}

def _quant_dequant(y, amax):
    scale = amax / 127.0
    q = np.clip(np.round(y / scale), -127.0, 127.0)
    return (q * scale).astype(np.float32)
def _epilogue(y):
    return _quant_dequant(y, np.abs(y).max())

def reference(x, w_mat):
    y = (x @ w_mat).astype(np.float32)
    return _epilogue(y)
```

```cpp
#include <hip/hip_runtime.h>
#include <rccl/rccl.h>
#include <cstdio>

typedef unsigned u32;
typedef unsigned long long u64;
typedef u32 u32x4 __attribute__((ext_vector_type(4)));
typedef float f32x4 __attribute__((ext_vector_type(4)));
typedef float f32x2 __attribute__((ext_vector_type(2)));
typedef __bf16 bf16x2 __attribute__((ext_vector_type(2)));
typedef __bf16 bf16x8 __attribute__((ext_vector_type(8)));

namespace {

constexpr u32 OFF_SEQ = 0;
constexpr u32 OFF_TMO = 256;
constexpr u32 OFF_AM = 4096;
constexpr u32 OFF_XF = 16384;
constexpr u32 OFF_WF = 24576;
constexpr u32 OFF_DBG = 65536;
constexpr u32 MB = 1u << 20;
constexpr u32 OFF_XS = 1 * MB;
constexpr u32 OFF_WS = 9 * MB;
constexpr u32 OFF_T1 = 17 * MB;
constexpr u32 OFF_PS = 21 * MB;
constexpr u32 WS_USED = 37 * MB;
constexpr u32 SLOT_BYTES = 262144;
constexpr u32 OOB = 0x80000000u;
constexpr u64 DEADLINE = 100000000ull;
constexpr int WT = 17;
constexpr int PPW = 2;
constexpr int NSTAGE = 4;
constexpr int LDS_BYTES = NSTAGE * 32768;

struct Params {
  const float* x;
  const float* w;
  signed char* out;
  char* ws[8];
  int rank;
};

typedef __amdgpu_buffer_rsrc_t rsrc_t;
typedef __attribute__((address_space(3))) u32 lds_u32;

__device__ __forceinline__ rsrc_t mk_rsrc(const void* p, u32 bytes) {
  return __builtin_amdgcn_make_buffer_rsrc((void*)p, (short)0, (int)bytes, 0x00020000);
}
__device__ __forceinline__ u32 pk_bf16(float lo, float hi) {
  f32x2 v = {lo, hi};
  bf16x2 b = __builtin_convertvector(v, bf16x2);
  return __builtin_bit_cast(u32, b);
}
__device__ __forceinline__ float as_f(u32 v) { return __builtin_bit_cast(float, v); }
__device__ __forceinline__ u32 as_u(float v) { return __builtin_bit_cast(u32, v); }
__device__ __forceinline__ u32 rfl(u32 v) { return __builtin_amdgcn_readfirstlane(v); }
__device__ __forceinline__ u32 nz(u32 d) { return d == 0u ? 0x8000u : d; }

struct Spin {
  u64 t0; u32 n;
  __device__ __forceinline__ void start() { t0 = __builtin_amdgcn_s_memrealtime(); n = 0; }
  __device__ __forceinline__ bool expired(u32* tmo, u32 seq) {
    if ((++n & 31u) != 0u) return false;
    u32 t = rfl(__hip_atomic_load(tmo, __ATOMIC_RELAXED, __HIP_MEMORY_SCOPE_AGENT));
    if (t == seq) return true;
    if (__builtin_amdgcn_s_memrealtime() - t0 > DEADLINE) {
      __hip_atomic_store(tmo, seq, __ATOMIC_RELAXED, __HIP_MEMORY_SCOPE_AGENT);
      return true;
    }
    return false;
  }
};

#define STAMP(k) do {} while (0)

__device__ __forceinline__ float wave_max(float v) {
#pragma unroll
  for (int o = 32; o > 0; o >>= 1) v = fmaxf(v, __shfl_xor(v, o));
  return v;
}

__device__ __forceinline__ void dma_unit(rsrc_t r, u32 voff, u32 soff, u32 ldsaddr) {
  u32 keep;
  asm volatile(
      "s_nop 4\n\t"
      "s_mov_b32 %0, m0\n\t"
      "s_mov_b32 m0, %4\n\t"
      "s_nop 0\n\t"
      "buffer_load_dwordx4 %1, %2, %3 offen sc0 sc1 lds\n\t"
      "buffer_load_dwordx4 %1, %2, %3 offen offset:1024 sc0 sc1 lds\n\t"
      "buffer_load_dwordx4 %1, %2, %3 offen offset:2048 sc0 sc1 lds\n\t"
      "buffer_load_dwordx4 %1, %2, %3 offen offset:3072 sc0 sc1 lds\n\t"
      "s_mov_b32 m0, %0"
      : "=&s"(keep)
      : "v"(voff), "s"(r), "s"(soff), "s"(ldsaddr)
      : "memory");
}

__global__ void __launch_bounds__(512, 2) fused_gemm_rs_q8(Params p) {
  extern __shared__ __attribute__((aligned(16))) char smem[];
  const int b = blockIdx.x, L = b & 7, q = b >> 3;
  const int tid = threadIdx.x, lane = tid & 63;
  const int wave = (int)rfl((u32)tid >> 6);
  const int rank = p.rank & 7;
  char* myws = p.ws[rank];
  const rsrc_t rs_me = mk_rsrc(myws, WS_USED);
  u32* tmo = (u32*)(myws + OFF_TMO);
  const int peer = rank ^ L;
  const rsrc_t rs_peer = mk_rsrc(p.ws[peer], WS_USED);
  const u32 seqv = __hip_atomic_load((u32*)(myws + OFF_SEQ), __ATOMIC_RELAXED, __HIP_MEMORY_SCOPE_AGENT);
  const int myi = (rank >> 1) & 1, myj = (rank >> 2) & 1, myh = rank & 1;
  STAMP(0);

  if (q < 16) {
    volatile lds_u32* lds = (volatile lds_u32*)smem;
    if (tid == 0) lds[0] = 0u;
    char* am_dst = p.ws[lane & 7] + OFF_AM;
    __syncthreads();
    const int ppw = (L < 2) ? 8 : PPW;
    if (wave < ppw) {
      const rsrc_t rs_x = mk_rsrc(p.x, 4096u * 512u * 4u);
      const rsrc_t rs_w = mk_rsrc(p.w, 512u * 4096u * 4u);
      const rsrc_t rs_dst = (L == 1) ? rs_me : rs_peer;
      const u32 pi = (peer >> 1) & 1, pj = (peer >> 2) & 1, ph = peer & 1;
      const u32 slotbase = 4u * (u32)((L == 1) ? peer : rank);
      const int np = 16 * ppw;
      const u32 vx = (u32)(lane >> 3) * 2048u + (u32)(lane & 7) * 16u;
      const u32 vw = (u32)(lane >> 3) * 65536u + (u32)(lane & 7) * 16u;
      const u32 dx0 = (u32)(lane >> 3) * 128u + (u32)(((lane & 7) ^ (lane >> 4)) * 16);
      const u32 fl_lane = lane == 0 ? 0u : OOB;
      u32x4 ld[8];
      auto desc = [&](int u, u32& isw, u32& src, u32& dst, u32& flg) {
        u32 t, r;
        if (u < 384) { t = (u32)u / 96u; r = (u32)u % 96u; isw = r >= 64u; if (isw) r -= 64u; }
        else { u32 v = (u32)u - 384u; t = v >> 5; r = 32u + (v & 31u); isw = 1u; }
        const u32 slot = slotbase + t, k0 = ph * 256u + t * 64u;
        if (isw) {
          src = k0 * 16384u + (pj * 2048u + r * 32u) * 4u;
          dst = OFF_WS + slot * SLOT_BYTES + r * 4096u;
          flg = OFF_WF + (r >> 2) * 512u + slot * 16u + (r & 3u) * 4u;
        } else {
          src = ((r >> 3) * 512u + pi * 256u + (r & 7u) * 32u) * 2048u + k0 * 4u;
          dst = OFF_XS + slot * SLOT_BYTES + r * 4096u;
          flg = OFF_XF + (r >> 2) * 512u + slot * 16u + (r & 3u) * 4u;
        }
      };
      auto loads = [&](u32 isw, u32 src) {
        if (isw) {
#pragma unroll
          for (int j = 0; j < 8; ++j)
            ld[j] = __builtin_amdgcn_raw_buffer_load_b128(rs_w, vw, src + (u32)((j & 3) + 32 * (j >> 2)) * 16384u, 0);
        } else {
#pragma unroll
          for (int j = 0; j < 8; ++j)
            ld[j] = __builtin_amdgcn_raw_buffer_load_b128(rs_x, vx, src + (u32)(j >> 1) * 16384u + (u32)(j & 1) * 128u, 0);
        }
      };
      int u = wave * 16 + q;
      u32 isw, src, dst, flg, pflg = OOB;
      desc(u, isw, src, dst, flg);
      loads(isw, src);
      const u32 seq = rfl(seqv) + 1u;
      for (; u < 512; u += np) {
        asm volatile("s_waitcnt vmcnt(0)" ::: "memory");
        __builtin_amdgcn_raw_buffer_store_b32(seq, rs_dst, fl_lane | pflg, 0, WT);
        u32x4 o[4];
        if (isw) {
#pragma unroll
          for (int c = 0; c < 4; ++c)
#pragma unroll
            for (int d = 0; d < 4; ++d) o[c][d] = pk_bf16(as_f(ld[2 * d][c]), as_f(ld[2 * d + 1][c]));
        } else {
#pragma unroll
          for (int g = 0; g < 4; ++g) {
            o[g][0] = pk_bf16(as_f(ld[2 * g][0]), as_f(ld[2 * g][1]));
            o[g][1] = pk_bf16(as_f(ld[2 * g][2]), as_f(ld[2 * g][3]));
            o[g][2] = pk_bf16(as_f(ld[2 * g + 1][0]), as_f(ld[2 * g + 1][1]));
            o[g][3] = pk_bf16(as_f(ld[2 * g + 1][2]), as_f(ld[2 * g + 1][3]));
          }
        }
        if (isw) {
#pragma unroll
          for (int c = 0; c < 4; ++c)
            __builtin_amdgcn_raw_buffer_store_b128(o[c], rs_dst, (u32)((lane ^ c) * 16), dst + (u32)c * 1024u, WT);
        } else {
#pragma unroll
          for (int g = 0; g < 4; ++g)
            __builtin_amdgcn_raw_buffer_store_b128(o[g], rs_dst, dx0 ^ (u32)((g & 1) * 64), dst + (u32)g * 1024u, WT);
        }
        pflg = flg;
        if (u + np < 512) {
          desc(u + np, isw, src, dst, flg);
          loads(isw, src);
        }
      }
      asm volatile("s_waitcnt vmcnt(0)" ::: "memory");
      __builtin_amdgcn_raw_buffer_store_b32(seq, rs_dst, fl_lane | pflg, 0, WT);
      if (lane == 0) __hip_atomic_fetch_add((lds_u32*)smem, 1u, __ATOMIC_RELAXED, __HIP_MEMORY_SCOPE_WORKGROUP);
    }
    STAMP(1);
    const u32 seq = rfl(seqv) + 1u;
    const u32 par = seq & 1u;
    const u32 l16 = (u32)lane * 16u;
    const u32x4 zero4 = {0u, 0u, 0u, 0u};
    {
      const u32 so = OFF_T1 + (u32)(((L * 16 + q) * 8 + wave)) * 4096u;
      u32x4 d[4];
      Spin sp; sp.start();
      for (;;) {
        u32 v = __builtin_amdgcn_raw_buffer_load_b32(rs_me, 4092u, so, WT);
        if (rfl(v) != 0u) break;
        if (sp.expired(tmo, seq)) break;
        __builtin_amdgcn_s_sleep(16);
      }
      for (;;) {
        bool ok = true;
#pragma unroll
        for (int e = 0; e < 4; ++e) d[e] = __builtin_amdgcn_raw_buffer_load_b128(rs_me, l16, so + (u32)e * 1024u, WT);
#pragma unroll
        for (int e = 0; e < 4; ++e) ok = ok && d[e][0] && d[e][1] && d[e][2] && d[e][3];
        if (__all(ok)) break;
        if (sp.expired(tmo, seq)) break;
      }
#pragma unroll
      for (int e = 0; e < 4; ++e) __builtin_amdgcn_raw_buffer_store_b128(zero4, rs_me, l16, so + (u32)e * 1024u, WT);
      STAMP(2);
      sp.start();
      while (lds[0] < (u32)ppw) {
        if (sp.expired(tmo, seq)) break;
        __builtin_amdgcn_s_sleep(4);
      }
      const u32 tile = (u32)(((myi * 2 + myj) * 2 + (q & 1)) * 16 + (q >> 1));
      const u32 dd = OFF_PS + par * 8u * MB + (u32)myh * 4u * MB + (tile * 8u + (u32)wave) * 4096u;
#pragma unroll
      for (int e = 0; e < 4; ++e) __builtin_amdgcn_raw_buffer_store_b128(d[e], rs_peer, l16, dd + (u32)e * 1024u, WT);
      STAMP(3);
    }
    const u32 cw = (u32)(q * 8 + L);
    float y[2][16];
    {
      const u32 so = OFF_PS + par * 8u * MB + (cw * 8u + (u32)wave) * 4096u;
      Spin sp; sp.start();
      const u32 pv = lane == 0 ? 4092u : (lane == 1 ? 4u * MB + 4092u : OOB);
      for (;;) {
        u32 v = __builtin_amdgcn_raw_buffer_load_b32(rs_me, pv, so, WT);
        if (__all(lane >= 2 || v != 0u)) break;
        if (sp.expired(tmo, seq)) break;
        __builtin_amdgcn_s_sleep(4);
      }
      u32x4 d0[4], d1[4];
      for (;;) {
        bool ok = true;
#pragma unroll
        for (int e = 0; e < 4; ++e) {
          d0[e] = __builtin_amdgcn_raw_buffer_load_b128(rs_me, l16, so + (u32)e * 1024u, WT);
          d1[e] = __builtin_amdgcn_raw_buffer_load_b128(rs_me, l16, so + 4u * MB + (u32)e * 1024u, WT);
        }
#pragma unroll
        for (int e = 0; e < 4; ++e)
          ok = ok && d0[e][0] && d0[e][1] && d0[e][2] && d0[e][3] && d1[e][0] && d1[e][1] && d1[e][2] && d1[e][3];
        if (__all(ok)) break;
        if (sp.expired(tmo, seq)) break;
      }
      STAMP(4);
      float mx = 0.f;
#pragma unroll
      for (int e = 0; e < 4; ++e)
#pragma unroll
        for (int d = 0; d < 4; ++d) {
          const float lo = as_f(d0[e][d] << 16) + as_f(d1[e][d] << 16);
          const float hi = as_f(d0[e][d] & 0xffff0000u) + as_f(d1[e][d] & 0xffff0000u);
          y[e >> 1][(e & 1) * 8 + d * 2] = lo;
          y[e >> 1][(e & 1) * 8 + d * 2 + 1] = hi;
          mx = fmaxf(mx, fmaxf(fabsf(lo), fabsf(hi)));
        }
      mx = wave_max(mx);
      if (lane == 0) lds[8 + wave] = as_u(mx);
#pragma unroll
      for (int e = 0; e < 4; ++e) {
        __builtin_amdgcn_raw_buffer_store_b128(zero4, rs_me, l16, so + (u32)e * 1024u, WT);
        __builtin_amdgcn_raw_buffer_store_b128(zero4, rs_me, l16, so + 4u * MB + (u32)e * 1024u, WT);
      }
    }
    asm volatile("s_waitcnt lgkmcnt(0)\n\ts_barrier" ::: "memory");
    if (wave == 0 && lane < 8) {
      u32 m = lds[8];
#pragma unroll
      for (int k = 1; k < 8; ++k) m = max(m, lds[8 + k]);
      __hip_atomic_store((u64*)(am_dst + ((u32)rank * 128u + cw) * 8u), ((u64)seq << 32) | (u64)m, __ATOMIC_RELAXED,
                         __HIP_MEMORY_SCOPE_SYSTEM);
    }
    STAMP(5);
    u32 bad;
    {
      Spin sp; sp.start();
      u32x4 g;
      for (;;) {
        g = __builtin_amdgcn_raw_buffer_load_b128(rs_me, (u32)tid * 16u, OFF_AM, WT);
        bad = __builtin_amdgcn_raw_buffer_load_b32(rs_me, 0u, OFF_TMO, WT);
        if (__all(g[1] == seq && g[3] == seq)) break;
        if (sp.expired(tmo, seq)) { bad = seq; break; }
        __builtin_amdgcn_s_sleep(2);
      }
      float mx = wave_max(fmaxf(as_f(g[0]), as_f(g[2])));
      if (lane == 0) { lds[16 + wave] = as_u(mx); lds[24 + wave] = (bad == seq) ? 1u : 0u; }
    }
    asm volatile("s_waitcnt lgkmcnt(0)\n\ts_barrier" ::: "memory");
    STAMP(6);
    {
      u32 m = lds[16], anybad = lds[24];
#pragma unroll
      for (int k = 1; k < 8; ++k) { m = max(m, lds[16 + k]); anybad |= lds[24 + k]; }
      const float amax = as_f(m);
      const float scale = amax / 127.0f;
      const float inv = scale > 0.f ? 1.0f / scale : 0.f;
      const u32 bcol = cw & 15u, ra = (cw >> 4) & 1u, tj = (cw >> 5) & 1u, ti = (cw >> 6) & 1u;
      const u32 row0 = ti * 256u + ra * 128u + (u32)(wave >> 1) * 32u + (u32)(lane & 15);
      const u32 col0 = tj * 2048u + bcol * 128u + (u32)(wave & 1) * 64u + (u32)(lane >> 4) * 16u;
#pragma unroll
      for (int mm = 0; mm < 2; ++mm) {
        u32x4 o;
#pragma unroll
        for (int d = 0; d < 4; ++d) {
          u32 wd = 0;
#pragma unroll
          for (int k = 0; k < 4; ++k) {
            float t = __builtin_rintf(y[mm][d * 4 + k] * inv);
            t = fminf(fmaxf(t, -127.f), 127.f);
            wd |= ((u32)(int)t & 0xffu) << (8 * k);
          }
          o[d] = anybad ? 0x80808080u : wd;
        }
        *(u32x4*)(p.out + (size_t)(row0 + (u32)mm * 16u) * 4096u + col0) = o;
      }
    }
    if (b == 0 && tid == 0) __hip_atomic_store((u32*)(myws + OFF_SEQ), seq, __ATOMIC_RELAXED, __HIP_MEMORY_SCOPE_AGENT);
    STAMP(7);
  } else {
    const int g = q - 16;
    const u32 a = (u32)(2 * peer + (g & 1)), bb = (u32)(g >> 1);
    u32 goff, foff;
    {
      const int n = lane;
      const u32 t2 = (n < 8 || n >= 40) ? 1u : 0u;
      u32 slot;
      if (n < 16) { const int k = n & 7; slot = (k < 4) ? (u32)(4 * rank + k) : (u32)(4 * (rank ^ 1) + k - 4); }
      else { const int r = (n - 16) % 24; slot = (u32)(4 * (rank ^ (2 + r % 6)) + r / 6); }
      const u32 bcol = bb + 8u * t2;
      if (wave < 4) {
        goff = OFF_XS + slot * SLOT_BYTES + (a * 4u + (u32)wave) * 4096u;
        foff = OFF_XF + a * 512u + slot * 16u + (u32)wave * 4u;
      } else {
        goff = OFF_WS + slot * SLOT_BYTES + (bcol * 4u + (u32)(wave - 4)) * 4096u;
        foff = OFF_WF + bcol * 512u + slot * 16u + (u32)(wave - 4) * 4u;
      }
    }
    const u32 seq = rfl(seqv) + 1u;
    const u32 l16 = (u32)lane * 16u;
    const u32 lds_base = (u32)(size_t)(__attribute__((address_space(3))) char*)smem;
    const int wr = wave >> 1, wc = wave & 1, fi = lane & 15, kq = lane >> 4;
    u32 ax[2], aw[4];
#pragma unroll
    for (int ks = 0; ks < 2; ++ks) ax[ks] = (u32)((wr * 32 + fi) * 128 + (((kq + 4 * ks) ^ (fi >> 1)) * 16));
#pragma unroll
    for (int f = 0; f < 4; ++f)
      aw[f] = (u32)(16384 + wc * 8192 + (fi >> 3) * 4096 + (fi & 3) * 1024 + ((kq * 8 + 4 * ((fi >> 2) & 1) + (f ^ (fi & 3))) * 16));
    f32x4 acc1[2][4], acc2[2][4];
#pragma unroll
    for (int m = 0; m < 2; ++m)
#pragma unroll
      for (int f = 0; f < 4; ++f) { acc1[m][f] = (f32x4){0.f, 0.f, 0.f, 0.f}; acc2[m][f] = (f32x4){0.f, 0.f, 0.f, 0.f}; }
    u64 mask = 0;
    int iss = 0;
    u32 fpoll = 0;
    auto poll = [&]() -> u32 { return __builtin_amdgcn_raw_buffer_load_b32(rs_me, foff, 0, WT); };
    auto issue = [&](int k) {
      dma_unit(rs_me, l16, (u32)__builtin_amdgcn_readlane((int)goff, k),
               lds_base + (u32)(k & (NSTAGE - 1)) * 32768u + (u32)wave * 4096u);
    };
    auto compute = [&](int n, f32x4 (&acc)[2][4]) {
      const char* st = smem + (n & (NSTAGE - 1)) * 32768;
      bf16x8 xf[2][2], wf[4][2];
#pragma unroll
      for (int ks = 0; ks < 2; ++ks) {
#pragma unroll
        for (int m = 0; m < 2; ++m) xf[m][ks] = *(const bf16x8*)(st + ax[ks] + m * 2048);
#pragma unroll
        for (int f = 0; f < 4; ++f) wf[f][ks] = *(const bf16x8*)(st + aw[f] + ks * 512);
      }
#pragma unroll
      for (int ks = 0; ks < 2; ++ks)
#pragma unroll
        for (int f = 0; f < 4; ++f)
#pragma unroll
          for (int m = 0; m < 2; ++m)
            acc[m][f] = __builtin_amdgcn_mfma_f32_16x16x32_bf16(wf[f][ks], xf[m][ks], acc[m][f], 0, 0, 0);
    };
    auto step = [&](int n, f32x4 (&acc)[2][4]) {
      if (iss <= n) {
        if (!((mask >> n) & 1ull)) {
          Spin sp; sp.start();
          for (;;) {
            const u32 f = poll();
            mask = __ballot((int)(f - seq) >= 0);
            if ((mask >> n) & 1ull) break;
            if (sp.expired(tmo, seq)) { mask = ~0ull; break; }
            __builtin_amdgcn_s_sleep(2);
          }
        }
        issue(n);
        iss = n + 1;
      }
      const int k = iss - n - 1;
      if (k >= 3) asm volatile("s_waitcnt vmcnt(12)" ::: "memory");
      else if (k == 2) asm volatile("s_waitcnt vmcnt(8)" ::: "memory");
      else if (k == 1) asm volatile("s_waitcnt vmcnt(4)" ::: "memory");
      else asm volatile("s_waitcnt vmcnt(0)" ::: "memory");
      asm volatile("s_barrier" ::: "memory");
      const int lim = (n + NSTAGE < 64) ? n + NSTAGE : 64;
      while (iss < lim && ((mask >> iss) & 1ull)) { issue(iss); ++iss; }
      const bool stuck = iss < lim;
      if (stuck) asm volatile("buffer_load_dword %0, %1, %2, 0 offen sc0 sc1" : "+v"(fpoll) : "v"(foff), "s"(rs_me) : "memory");
      compute(n, acc);
      if (stuck) {
        __builtin_amdgcn_sched_barrier(0);
        asm volatile("s_waitcnt vmcnt(0)" : "+v"(fpoll) :: "memory");
        mask = __ballot((int)(fpoll - seq) >= 0);
        while (iss < lim && ((mask >> iss) & 1ull)) { issue(iss); ++iss; }
      }
      asm volatile("s_waitcnt lgkmcnt(0)" ::: "memory");
    };
    auto epilogue = [&](f32x4 (&acc)[2][4], rsrc_t r, u32 off) {
#pragma unroll
      for (int m = 0; m < 2; ++m)
#pragma unroll
        for (int hf = 0; hf < 2; ++hf) {
          u32x4 o;
          o[0] = nz(pk_bf16(acc[m][2 * hf][0], acc[m][2 * hf][1]));
          o[1] = nz(pk_bf16(acc[m][2 * hf][2], acc[m][2 * hf][3]));
          o[2] = nz(pk_bf16(acc[m][2 * hf + 1][0], acc[m][2 * hf + 1][1]));
          o[3] = nz(pk_bf16(acc[m][2 * hf + 1][2], acc[m][2 * hf + 1][3]));
          __builtin_amdgcn_raw_buffer_store_b128(o, r, l16, off + (u32)(m * 2 + hf) * 1024u, WT);
        }
    };
    for (int n = 0; n < 8; ++n) step(n, acc2);
    STAMP(1);
    for (int n = 8; n < 40; ++n) {
      step(n, acc1);
    }
    epilogue(acc1, rs_me, OFF_T1 + (u32)((L * 16 + g) * 8 + wave) * 4096u);
    STAMP(3);
    for (int n = 40; n < 64; ++n) step(n, acc2);
    {
      const u32 tile = (u32)(((myi * 2 + myj) * 2 + (g & 1)) * 16) + 8u + bb;
      epilogue(acc2, rs_peer, OFF_PS + (seq & 1u) * 8u * MB + (u32)myh * 4u * MB + (tile * 8u + (u32)wave) * 4096u);
    }
    STAMP(4);
  }
}

}

#define HIP_CHECK(expr) do { hipError_t _e = (expr); if (_e != hipSuccess) { \
    fprintf(stderr, "HIP error %s at %s:%d\n", hipGetErrorString(_e), __FILE__, __LINE__); \
    return; } } while (0)

extern "C" void kernel_launch(int rank, int nranks, ncclComm_t comm, hipStream_t stream, void* const* d_in,
                              const int* in_sizes, int n_in, void* d_out, int out_size, const int* in_dtypes,
                              int out_dtype, void* const* d_out_all, void* const* d_ws_all, size_t ws_bytes) {
  (void)comm; (void)in_dtypes; (void)out_dtype; (void)d_out_all;
  if (nranks != 8 || (unsigned)rank >= 8u || n_in != 2 || in_sizes[0] != 4096 * 512 || in_sizes[1] != 512 * 4096 ||
      out_size != 512 * 4096 || ws_bytes < (size_t)WS_USED) {
    fprintf(stderr, "kernel_launch: unexpected problem shape\n");
    return;
  }
  Params p;
  p.x = (const float*)d_in[0];
  p.w = (const float*)d_in[1];
  p.out = (signed char*)d_out;
  for (int i = 0; i < 8; ++i) p.ws[i] = (char*)d_ws_all[i];
  p.rank = rank;
  HIP_CHECK(hipFuncSetAttribute((const void*)fused_gemm_rs_q8, hipFuncAttributeMaxDynamicSharedMemorySize, LDS_BYTES));
  fused_gemm_rs_q8<<<dim3(256), dim3(512), LDS_BYTES, stream>>>(p);
  HIP_CHECK(hipGetLastError());
}
```

```cpp
#include <hip/hip_runtime.h>
#include <rccl/rccl.h>
#include <cstdio>

typedef unsigned u32;
typedef unsigned long long u64;
typedef u32 u32x4 __attribute__((ext_vector_type(4)));
typedef u32 u32x2 __attribute__((ext_vector_type(2)));
typedef float f32x4 __attribute__((ext_vector_type(4)));
typedef float f32x2 __attribute__((ext_vector_type(2)));
typedef __bf16 bf16x2 __attribute__((ext_vector_type(2)));
typedef __bf16 bf16x8 __attribute__((ext_vector_type(8)));

namespace {

constexpr u32 OFF_SEQ = 0;
constexpr u32 OFF_TMO = 256;
constexpr u32 OFF_TK = 1024;
constexpr u32 OFF_AM = 4096;
constexpr u32 OFF_XF = 16384;
constexpr u32 OFF_WF = 24576;
constexpr u32 OFF_TF = 32768;
constexpr u32 OFF_DBG = 65536;
constexpr u32 MB = 1u << 20;
constexpr u32 OFF_XS = 1 * MB;
constexpr u32 OFF_WS = 9 * MB;
constexpr u32 OFF_PS = 17 * MB;
constexpr u32 OFF_TL = 33 * MB;
constexpr u32 WS_USED = 41 * MB;
constexpr u32 SLOT_BYTES = 262144;
constexpr u32 OOB = 0x80000000u;
constexpr u64 DEADLINE = 100000000ull;
constexpr int WT = 17;
#define NP_ 16
#define PRE 24
#define POLL_SLEEP 2
#define RESET_SLEEP 64
#define OWN_DELAY 0
constexpr int NP = NP_;
constexpr int NSTAGE = 4;
constexpr int LDS_POLL = NSTAGE * 32768;
constexpr int LDS_BYTES = LDS_POLL + 4096;
constexpr int NTHREADS = 768;

struct Params {
  const float* x;
  const float* w;
  signed char* out;
  char* ws[8];
  int rank;
};

typedef __amdgpu_buffer_rsrc_t rsrc_t;
typedef __attribute__((address_space(3))) u32 lds_u32;

__device__ __forceinline__ rsrc_t mk_rsrc(const void* p, u32 bytes) {
  return __builtin_amdgcn_make_buffer_rsrc((void*)p, (short)0, (int)bytes, 0x00020000);
}
__device__ __forceinline__ u32 pk_bf16(float lo, float hi) {
  f32x2 v = {lo, hi};
  bf16x2 b = __builtin_convertvector(v, bf16x2);
  return __builtin_bit_cast(u32, b);
}
__device__ __forceinline__ float as_f(u32 v) { return __builtin_bit_cast(float, v); }
__device__ __forceinline__ u32 as_u(float v) { return __builtin_bit_cast(u32, v); }
__device__ __forceinline__ u32 rfl(u32 v) { return __builtin_amdgcn_readfirstlane(v); }
__device__ __forceinline__ u32 nz(u32 d) { return d == 0u ? 0x8000u : d; }

struct Spin {
  u64 t0; u32 n;
  __device__ __forceinline__ void start() { t0 = __builtin_amdgcn_s_memrealtime(); n = 0; }
  __device__ __forceinline__ bool expired(u32* tmo, u32 seq) {
    if ((++n & 31u) != 0u) return false;
    u32 t = rfl(__hip_atomic_load(tmo, __ATOMIC_RELAXED, __HIP_MEMORY_SCOPE_AGENT));
    if (t == seq) return true;
    if (__builtin_amdgcn_s_memrealtime() - t0 > DEADLINE) {
      __hip_atomic_store(tmo, seq, __ATOMIC_RELAXED, __HIP_MEMORY_SCOPE_AGENT);
      return true;
    }
    return false;
  }
};

#define STAMPC(k) do {} while (0)
#define STAMP(k) do {} while (0)

__device__ __forceinline__ u32 wave_max_u(u32 v) {
  v = max(v, (u32)__builtin_amdgcn_update_dpp(0, (int)v, 0x111, 0xf, 0xf, true));
  v = max(v, (u32)__builtin_amdgcn_update_dpp(0, (int)v, 0x112, 0xf, 0xf, true));
  v = max(v, (u32)__builtin_amdgcn_update_dpp(0, (int)v, 0x114, 0xf, 0xf, true));
  v = max(v, (u32)__builtin_amdgcn_update_dpp(0, (int)v, 0x118, 0xf, 0xf, true));
  v = max(v, (u32)__builtin_amdgcn_update_dpp(0, (int)v, 0x142, 0xa, 0xf, true));
  v = max(v, (u32)__builtin_amdgcn_update_dpp(0, (int)v, 0x143, 0xc, 0xf, true));
  return (u32)__builtin_amdgcn_readlane((int)v, 63);
}

#define EXP_W_OFF (u32)((lane ^ c) * 16)
#define EXP_X_OFF dx0 ^ (u32)((g & 1) * 64)
#define EXP_SEG0 0
#define DMA_SC "sc0 sc1"
__device__ __forceinline__ void dma8(rsrc_t r, u32 voff, u32 soff, u32 ldsaddr) {
  u32 soff2;
  asm volatile(
      "s_mov_b32 m0, %4\n\t"
      "s_add_u32 %0, %3, 0x1000\n\t"
      "buffer_load_dwordx4 %1, %2, %3 offen " DMA_SC " lds\n\t"
      "buffer_load_dwordx4 %1, %2, %3 offen offset:1024 " DMA_SC " lds\n\t"
      "buffer_load_dwordx4 %1, %2, %3 offen offset:2048 " DMA_SC " lds\n\t"
      "buffer_load_dwordx4 %1, %2, %3 offen offset:3072 " DMA_SC " lds\n\t"
      "s_add_u32 m0, %4, 0x1000\n\t"
      "s_nop 0\n\t"
      "buffer_load_dwordx4 %1, %2, %0 offen " DMA_SC " lds\n\t"
      "buffer_load_dwordx4 %1, %2, %0 offen offset:1024 " DMA_SC " lds\n\t"
      "buffer_load_dwordx4 %1, %2, %0 offen offset:2048 " DMA_SC " lds\n\t"
      "buffer_load_dwordx4 %1, %2, %0 offen offset:3072 " DMA_SC " lds"
      : "=&s"(soff2)
      : "v"(voff), "s"(r), "s"(soff), "s"(ldsaddr)
      : "memory", "scc");
}

__global__ void __launch_bounds__(NTHREADS, 3) fused_gemm_rs_q8(Params p) {
  extern __shared__ __attribute__((aligned(16))) char smem[];
  const int b = blockIdx.x, L = b & 7, q = b >> 3;
  const int tid = threadIdx.x, lane = tid & 63;
  const int wave = (int)rfl((u32)tid >> 6);
  const int rank = p.rank & 7;
  char* myws = p.ws[rank];
  const rsrc_t rs_me = mk_rsrc(myws, WS_USED);
  u32* tmo = (u32*)(myws + OFF_TMO);
  const int peer = rank ^ L;
  const rsrc_t rs_peer = mk_rsrc(p.ws[peer], WS_USED);
  const u32 seqv = __hip_atomic_load((u32*)(myws + OFF_SEQ), __ATOMIC_RELAXED, __HIP_MEMORY_SCOPE_AGENT);
  const int myi = (rank >> 1) & 1, myj = (rank >> 2) & 1, myh = rank & 1;
  const u32 l16 = (u32)lane * 16u;
  STAMP(0);

  if (q < 16) {
    const bool pusher = wave >= 8;
    if (pusher && (L == 0 || (wave - 8) * 16 + q >= NP)) return;
    if (OWN_DELAY > 0 && !pusher) __builtin_amdgcn_s_sleep(OWN_DELAY);
    {
      const u32 oid = (u32)((q * 2 + L) * 8 + wave);
      const int vl = pusher ? L : (int)(oid & 1u);
      const int vp = rank ^ vl;
      const rsrc_t rs_x = mk_rsrc(p.x, 4096u * 512u * 4u);
      const rsrc_t rs_w = mk_rsrc(p.w, 512u * 4096u * 4u);
      const rsrc_t rs_dst = pusher ? rs_peer : rs_me;
      const u32 pi = (vp >> 1) & 1, pj = (vp >> 2) & 1, ph = vp & 1;
      const u32 slotbase = 4u * (u32)(pusher ? rank : vp);
      const u32 vx = (u32)(lane >> 3) * 2048u + (u32)(lane & 7) * 16u;
      const u32 vw = (u32)(lane >> 3) * 65536u + (u32)(lane & 7) * 16u;
      const u32 dx0 = (u32)(lane >> 3) * 128u + (u32)(((lane & 7) ^ (lane >> 4)) * 16);
      const u32 fl_lane = lane == 0 ? 0u : OOB;
      u32x4 ld[8];
      auto desc = [&](int u, u32& isw, u32& src, u32& dst, u32& flg) {
        u32 t, r;
        if (u < 384) { t = (u32)u / 96u; r = (u32)u % 96u; isw = r >= 64u; if (isw) r -= 64u; }
        else { u32 v = (u32)u - 384u; t = v >> 5; r = 32u + (v & 31u); isw = 1u; }
        const u32 slot = slotbase + t, k0 = ph * 256u + t * 64u;
        if (isw) {
          src = k0 * 16384u + (pj * 2048u + r * 32u) * 4u;
          dst = OFF_WS + slot * SLOT_BYTES + r * 4096u;
          flg = OFF_WF + (r >> 2) * 512u + slot * 16u + (r & 3u) * 4u;
        } else {
          src = ((r >> 3) * 512u + pi * 256u + (r & 7u) * 32u) * 2048u + k0 * 4u;
          dst = OFF_XS + slot * SLOT_BYTES + r * 4096u;
          flg = OFF_XF + (r >> 2) * 512u + slot * 16u + (r & 3u) * 4u;
        }
      };
      auto loads = [&](u32 isw, u32 src) {
        if (isw) {
#pragma unroll
          for (int j = 0; j < 8; ++j)
            ld[j] = __builtin_amdgcn_raw_buffer_load_b128(rs_w, vw, src + (u32)((j & 3) + 32 * (j >> 2)) * 16384u, 0);
        } else {
#pragma unroll
          for (int j = 0; j < 8; ++j)
            ld[j] = __builtin_amdgcn_raw_buffer_load_b128(rs_x, vx, src + (u32)(j >> 1) * 16384u + (u32)(j & 1) * 128u, 0);
        }
      };
      const int ufirst = (L == 1) ? 512 : 0;
      auto draw = [&]() {
        return (u32)__builtin_amdgcn_raw_ptr_buffer_atomic_add_i32(1, rs_me, (int)fl_lane, (int)(OFF_TK + (u32)L * 128u), 0);
      };
      int u = pusher ? ufirst + (wave - 8) * 16 + q : (L < 2 ? (int)(oid >> 1) : 1000);
      int un = pusher ? u + NP : u + 128;
      u32 tkv = 0u;
      u32 pflg = OOB;
      u32 seq, tkbase;
      const u32 tl = OFF_TL + (u32)L * MB, tf = OFF_TF + (u32)L * 1024u;
      bool have = false;
      if (u < 512) {
        u32 isw, src, dst, flg;
        desc(u, isw, src, dst, flg);
        loads(isw, src);
        seq = rfl(seqv) + 1u;
        tkbase = (seq - 1u) * (u32)(768 - NP - ufirst) - (u32)(2 * NP + ufirst);
        bool drawn = false;
        for (;;) {
          asm volatile("s_waitcnt vmcnt(0)" ::: "memory");
          __builtin_amdgcn_raw_buffer_store_b32(seq, rs_dst, fl_lane | pflg, 0, WT);
          if (drawn) un = (int)(rfl(tkv) - tkbase);
          const u32 nfl = __builtin_amdgcn_raw_buffer_load_b32(rs_me, 0u, tf + ((u32)un & 255u) * 4u, WT);
          u32x4 o[4];
          if (isw) {
#pragma unroll
            for (int c = 0; c < 4; ++c)
#pragma unroll
              for (int d = 0; d < 4; ++d) o[c][d] = pk_bf16(as_f(ld[2 * d][c]), as_f(ld[2 * d + 1][c]));
          } else {
#pragma unroll
            for (int g = 0; g < 4; ++g) {
              o[g][0] = pk_bf16(as_f(ld[2 * g][0]), as_f(ld[2 * g][1]));
              o[g][1] = pk_bf16(as_f(ld[2 * g][2]), as_f(ld[2 * g][3]));
              o[g][2] = pk_bf16(as_f(ld[2 * g + 1][0]), as_f(ld[2 * g + 1][1]));
              o[g][3] = pk_bf16(as_f(ld[2 * g + 1][2]), as_f(ld[2 * g + 1][3]));
            }
          }
          if (isw) {
#pragma unroll
            for (int c = 0; c < 4; ++c)
              __builtin_amdgcn_raw_buffer_store_b128(o[c], rs_dst, EXP_W_OFF, dst + (u32)c * 1024u, WT);
          } else {
#pragma unroll
            for (int g = 0; g < 4; ++g)
              __builtin_amdgcn_raw_buffer_store_b128(o[g], rs_dst, EXP_X_OFF, dst + (u32)g * 1024u, WT);
          }
          pflg = flg;
          u = un;
          if (u >= 512) {
            have = pusher && rfl(nfl) == seq;
            asm volatile("" ::: "memory");
            if (have) {
#pragma unroll
              for (int e = 0; e < 4; ++e)
                ld[e] = __builtin_amdgcn_raw_buffer_load_b128(rs_me, l16, tl + ((u32)u - 512u) * 4096u + (u32)e * 1024u, WT);
            }
            break;
          }
          desc(u, isw, src, dst, flg);
          loads(isw, src);
          if (pusher) { tkv = draw(); drawn = true; } else un = u + 128;
        }
        if (pusher) tkv = draw();
      } else {
        seq = rfl(seqv) + 1u;
        tkbase = (seq - 1u) * (u32)(768 - NP - ufirst) - (u32)(2 * NP + ufirst);
        tkv = (u32)un + tkbase;
      }
      STAMP(1);
      if (pusher) {
        const u32 pbase = OFF_PS + (seq & 1u) * 8u * MB + (u32)myh * 4u * MB + (u32)((myi * 2 + myj) * 32) * 32768u;
        u32 fu = (u32)u - 512u;
        u32 tfl;
        for (;;) {
          asm volatile("s_waitcnt vmcnt(0)" ::: "memory");
          __builtin_amdgcn_raw_buffer_store_b32(seq, rs_dst, fl_lane | pflg, 0, WT);
          pflg = OOB;
          const u32 fn = rfl(tkv) - tkbase - 512u;
          if (!have) {
            Spin sp; sp.start();
            for (;;) {
              tfl = __builtin_amdgcn_raw_buffer_load_b32(rs_me, 0u, tf + fu * 4u, WT);
              if (rfl(tfl) == seq) break;
              if (sp.expired(tmo, seq)) break;
              __builtin_amdgcn_s_sleep(4);
            }
            asm volatile("" ::: "memory");
#pragma unroll
            for (int e = 0; e < 4; ++e) ld[e] = __builtin_amdgcn_raw_buffer_load_b128(rs_me, l16, tl + fu * 4096u + (u32)e * 1024u, WT);
          }
          const u32 nfl = __builtin_amdgcn_raw_buffer_load_b32(rs_me, 0u, tf + (fn & 255u) * 4u, WT);
          const u32 gg = (fu >> 3) & 15u;
          const u32 dd = pbase + (((gg & 1u) * 16u + (gg >> 1) + (fu >> 7) * 8u) * 8u + (fu & 7u)) * 4096u;
#pragma unroll
          for (int e = 0; e < 4; ++e) __builtin_amdgcn_raw_buffer_store_b128(ld[e], rs_peer, l16, dd + (u32)e * 1024u, WT);
          if (fn >= 256u) break;
          fu = fn;
          have = rfl(nfl) == seq;
          asm volatile("" ::: "memory");
          if (have) {
#pragma unroll
            for (int e = 0; e < 4; ++e) ld[e] = __builtin_amdgcn_raw_buffer_load_b128(rs_me, l16, tl + fu * 4096u + (u32)e * 1024u, WT);
          }
          tkv = draw();
        }
        return;
      }
      asm volatile("s_waitcnt vmcnt(0)" ::: "memory");
      __builtin_amdgcn_raw_buffer_store_b32(seq, rs_dst, fl_lane | pflg, 0, WT);
    }
    volatile lds_u32* lds = (volatile lds_u32*)smem;
    char* am_dst = p.ws[lane & 7] + OFF_AM;
    const u32 seq = rfl(seqv) + 1u;
    const u32 par = seq & 1u;
    const u32x4 zero4 = {0u, 0u, 0u, 0u};
    const u32 cw = (u32)b;
    if (L < 2) {
      const u32 zo = OFF_PS + (par ^ 1u) * 8u * MB + (u32)((q * 2 + L) * 8 + wave) * 32768u;
#pragma unroll 1
      for (u32 k = 0; k < 8u; ++k) {
        __builtin_amdgcn_s_sleep(RESET_SLEEP);
#pragma unroll
        for (int e = 0; e < 4; ++e) __builtin_amdgcn_raw_buffer_store_b128(zero4, rs_me, l16, zo + k * 4096u + (u32)e * 1024u, WT);
      }
      STAMP(12);
    }
    float y[2][16];
    {
      const u32 pb = OFF_PS + par * 8u * MB;
      const u32 so = pb + (cw * 8u + (u32)wave) * 4096u;
      Spin sp; sp.start();
      {
        const u32 bcol = cw & 15u, ra = (cw >> 4) & 1u;
        const u32 pos = ((bcol >> 3) * 16u + (bcol & 7u) * 2u + ra) * 8u + (u32)wave;
        const u32 pp = pos >= (u32)PRE ? pos - (u32)PRE : pos;
        const u32 pt = pp >> 3;
        const u32 po = pb + (((cw & 96u) + (pt & 1u) * 16u + (pt >> 4) * 8u + ((pt >> 1) & 7u)) * 8u + (pp & 7u)) * 4096u;
        const u32 pv = lane == 0 ? 4092u : (lane == 1 ? 4u * MB + 4092u : OOB);
        for (;;) {
          u32 v = __builtin_amdgcn_raw_buffer_load_b32(rs_me, pv, po, WT);
          if (__all(lane >= 2 || v != 0u)) break;
          if (sp.expired(tmo, seq)) break;
          __builtin_amdgcn_s_sleep(4);
        }
      }
      STAMP(8);
      u32x4 d0[4], d1[4];
      bool ok0 = false, ok1 = false;
      for (;;) {
        if (!ok0) {
#pragma unroll
          for (int e = 0; e < 4; ++e) d0[e] = __builtin_amdgcn_raw_buffer_load_b128(rs_me, l16, so + (u32)e * 1024u, WT);
        }
        if (!ok1) {
#pragma unroll
          for (int e = 0; e < 4; ++e) d1[e] = __builtin_amdgcn_raw_buffer_load_b128(rs_me, l16, so + 4u * MB + (u32)e * 1024u, WT);
        }
        bool a0 = true, a1 = true;
#pragma unroll
        for (int e = 0; e < 4; ++e) {
          a0 = a0 && d0[e][0] && d0[e][1] && d0[e][2] && d0[e][3];
          a1 = a1 && d1[e][0] && d1[e][1] && d1[e][2] && d1[e][3];
        }
        ok0 = __all(a0); ok1 = __all(a1);
        if (ok0 && ok1) break;
        if (sp.expired(tmo, seq)) break;
        __builtin_amdgcn_s_sleep(POLL_SLEEP);
      }
      STAMP(4);
      float mx = 0.f;
#pragma unroll
      for (int e = 0; e < 4; ++e)
#pragma unroll
        for (int d = 0; d < 4; ++d) {
          const float lo = as_f(d0[e][d] << 16) + as_f(d1[e][d] << 16);
          const float hi = as_f(d0[e][d] & 0xffff0000u) + as_f(d1[e][d] & 0xffff0000u);
          y[e >> 1][(e & 1) * 8 + d * 2] = lo;
          y[e >> 1][(e & 1) * 8 + d * 2 + 1] = hi;
          mx = fmaxf(mx, fmaxf(fabsf(lo), fabsf(hi)));
        }
      lds[8 + wave] = wave_max_u(as_u(mx));
      STAMP(9);
      asm volatile("s_waitcnt lgkmcnt(0)\n\ts_barrier" ::: "memory");
      STAMP(10);
      if (wave == 0 && lane < 8) {
        u32 m = lds[8];
#pragma unroll
        for (int k = 1; k < 8; ++k) m = max(m, lds[8 + k]);
        __hip_atomic_store((u64*)(am_dst + ((u32)rank * 128u + cw) * 8u), ((u64)seq << 32) | (u64)m, __ATOMIC_RELAXED,
                           __HIP_MEMORY_SCOPE_SYSTEM);
      }
    }
    STAMP(5);
    u32 bad;
    {
      Spin sp; sp.start();
      u32x4 g;
      for (;;) {
        g = __builtin_amdgcn_raw_buffer_load_b128(rs_me, (u32)tid * 16u, OFF_AM, WT);
        bad = __builtin_amdgcn_raw_buffer_load_b32(rs_me, 0u, OFF_TMO, WT);
        if (__all(g[1] == seq && g[3] == seq)) break;
        if (sp.expired(tmo, seq)) { bad = seq; break; }
        __builtin_amdgcn_s_sleep(2);
      }
      STAMP(11);
      lds[16 + wave] = (rfl(bad) == seq) ? 0xffffffffu : wave_max_u(max(g[0], g[2]));
    }
    asm volatile("s_waitcnt lgkmcnt(0)\n\ts_barrier" ::: "memory");
    STAMP(6);
    {
      u32 m = lds[16];
#pragma unroll
      for (int k = 1; k < 8; ++k) m = max(m, lds[16 + k]);
      const bool anybad = m == 0xffffffffu;
      const float inv = m ? 127.0f / as_f(m) : 0.f;
      const u32 bcol = cw & 15u, ra = (cw >> 4) & 1u, tj = (cw >> 5) & 1u, ti = (cw >> 6) & 1u;
      const u32 row0 = ti * 256u + ra * 128u + (u32)(wave >> 1) * 32u + (u32)(lane & 15);
      const u32 col0 = tj * 2048u + bcol * 128u + (u32)(wave & 1) * 64u + (u32)(lane >> 4) * 16u;
#pragma unroll
      for (int mm = 0; mm < 2; ++mm) {
        u32x4 o;
#pragma unroll
        for (int d = 0; d < 4; ++d) {
          u32 t[4];
#pragma unroll
          for (int k = 0; k < 4; ++k)
            t[k] = as_u(__builtin_amdgcn_fmed3f(__builtin_fmaf(y[mm][d * 4 + k], inv, 12582912.f), 12582912.f - 127.f, 12582912.f + 127.f));
          const u32 wd = __builtin_amdgcn_perm(__builtin_amdgcn_perm(t[3], t[2], 0x0c0c0400u), __builtin_amdgcn_perm(t[1], t[0], 0x0c0c0400u), 0x05040100u);
          o[d] = anybad ? 0x80808080u : wd;
        }
        *(u32x4*)(p.out + (size_t)(row0 + (u32)mm * 16u) * 4096u + col0) = o;
      }
    }
    if (b == 0 && tid == 0) __hip_atomic_store((u32*)(myws + OFF_SEQ), seq, __ATOMIC_RELAXED, __HIP_MEMORY_SCOPE_AGENT);
    STAMP(7);
  } else {
    const int g = q - 16;
    const u32 bb = (u32)(g >> 1);
    if (wave >= 8) {
      __builtin_amdgcn_s_setprio(3);
      const int j = wave - 8;
      const u32 a = (u32)(2 * peer + (g & 1));
      u32 goff, foff;
      {
        const int ROT8 = (int)bb + 4 * (g & 1), ROT6 = (int)bb + 3 * (g & 1);
        const int n = lane;
        const u32 t2 = ((n >= 14 && n < 21) || n >= 39) ? 1u : 0u;
        u32 slot;
        if (n < 8) {
          slot = (u32)(4 * (rank ^ ((n + g) & 1)) + (n >> 1));
        } else if ((n >= 14 && n < 21) || n == 39) {
          const int k = ((n == 39 ? 7 : n - 14) + ROT8) & 7;
          slot = (u32)(4 * (rank ^ (k >> 2)) + (k & 3));
        } else {
          const int r = (n < 14) ? n - 8 : (n < 39 ? n - 15 : n - 40);
          slot = (u32)(4 * (rank ^ (2 + (r + ROT6) % 6)) + r / 6);
        }
        const u32 bcol = bb + 8u * t2;
        if (j < 2) {
          goff = OFF_XS + slot * SLOT_BYTES + (a * 4u + (u32)(2 * j)) * 4096u;
          foff = OFF_XF + a * 512u + slot * 16u;
        } else {
          goff = OFF_WS + slot * SLOT_BYTES + (bcol * 4u + (u32)(2 * (j - 2))) * 4096u;
          foff = OFF_WF + bcol * 512u + slot * 16u;
        }
      }
      const u32 seq = rfl(seqv) + 1u;
      const u32 lds_base0 = rfl((u32)(size_t)(__attribute__((address_space(3))) char*)smem);
      const u32 lds_base = lds_base0 + (u32)j * 8192u;
      int cov = 0;
      int iss = 0, since = 64;
      bool pend = false;
      const u32 poll_lds = lds_base0 + (u32)LDS_POLL + (u32)j * 1024u;
      const volatile lds_u32* pl = (const volatile lds_u32*)(smem + LDS_POLL + j * 1024 + lane * 16 + (j & 1) * 8);
      auto prefix = [&](u32 f0, u32 f1) {
        const u64 miss = __ballot((int)(f0 - seq) < 0 || (int)(f1 - seq) < 0);
        return miss ? (int)__builtin_ctzll(miss) : 64;
      };
      auto issue = [&](int k) {
        u32 go = (u32)__builtin_amdgcn_readlane((int)goff, k);
        asm volatile("s_nop 4" : "+s"(go));
        dma8(rs_me, l16, go, lds_base + (u32)(k & (NSTAGE - 1)) * 32768u);
      };
#pragma unroll 1
      for (int n = 0; n < 64; ++n) {
        if (iss <= n) {
          if (cov <= n) {
            Spin sp; sp.start();
            for (;;) {
              const u32x2 fp = __builtin_amdgcn_raw_buffer_load_b64(rs_me, foff + (u32)(j & 1) * 8u, 0, WT);
              cov = prefix(fp[0], fp[1]);
              if (cov > n) break;
              if (sp.expired(tmo, seq)) { cov = 64; break; }
              __builtin_amdgcn_s_sleep(1);
            }
          }
          issue(n);
          iss = n + 1;
        }
        int k = iss - n - 1;
        if (pend && since < k) k = since;
        const int lim = (n + NSTAGE < 64) ? n + NSTAGE : 64;
        if (k >= 2) asm volatile("s_waitcnt vmcnt(16)\n\ts_barrier" ::: "memory");
        else if (k == 1) asm volatile("s_waitcnt vmcnt(8)\n\ts_barrier" ::: "memory");
        else asm volatile("s_waitcnt vmcnt(0)\n\ts_barrier" ::: "memory");
        if (pend) {
          const int c = prefix(pl[0], pl[1]);
          cov = c > cov ? c : cov;
          if (cov == 64) cov = 1000;
          pend = false;
        }
        if (cov < lim + 4) {
          asm volatile("s_waitcnt lgkmcnt(0)\n\t"
                       "s_mov_b32 m0, %2\n\t"
                       "s_nop 0\n\t"
                       "buffer_load_dwordx4 %0, %1, 0 offen sc0 sc1 lds"
                       :: "v"(foff), "s"(rs_me), "s"(poll_lds) : "memory");
          pend = true;
          since = 0;
        }
        const int top = lim < cov ? lim : cov;
        while (iss < top) { issue(iss); ++iss; ++since; }
        if (!pend && iss == n + NSTAGE) {
          const int last = (cov - 8 < 63 - NSTAGE + 1) ? cov - 8 : 63 - NSTAGE + 1;
          u32 la = (u32)(iss & (NSTAGE - 1)) * 32768u;
          u32 go = (u32)__builtin_amdgcn_readlane((int)goff, iss & 63);
#pragma unroll 1
          while (n < last) {
            asm volatile("s_waitcnt vmcnt(16)\n\ts_barrier" : "+s"(go) :: "memory");
            dma8(rs_me, l16, go, lds_base + la);
            ++n; ++iss;
            la = (la + 32768u) & (u32)(NSTAGE * 32768 - 1);
            go = (u32)__builtin_amdgcn_readlane((int)goff, iss & 63);
          }
        }
      }
      STAMP(1);
    } else {
      const int grp = wave >> 2, wr = (wave >> 1) & 1, wc = wave & 1, fi = lane & 15, kq = lane >> 4;
      u32 ax[2], aw[4];
#pragma unroll
      for (int ks = 0; ks < 2; ++ks) ax[ks] = (u32)((wr * 64 + fi) * 128 + (((kq + 4 * ks) ^ (fi >> 1)) * 16));
#pragma unroll
      for (int f = 0; f < 4; ++f)
        aw[f] = (u32)(16384 + wc * 8192 + (fi >> 3) * 4096 + (fi & 3) * 1024 + ((kq * 8 + 4 * ((fi >> 2) & 1) + (f ^ (fi & 3))) * 16));
      f32x4 acc[4][4];
#pragma unroll
      for (int m = 0; m < 4; ++m)
#pragma unroll
        for (int f = 0; f < 4; ++f) acc[m][f] = (f32x4){0.f, 0.f, 0.f, 0.f};
      bf16x8 x0[4], w0[4], x1[4], w1[4];
      auto rd = [&](int n, int ks, bf16x8 (&xf)[4], bf16x8 (&wf)[4]) {
        const char* st = smem + (n & (NSTAGE - 1)) * 32768;
#pragma unroll
        for (int m = 0; m < 4; ++m) {
          wf[m] = *(const bf16x8*)(st + aw[m] + ks * 512);
          xf[m] = *(const bf16x8*)(st + ax[ks] + m * 2048);
        }
      };
      auto mm = [&](bf16x8 (&xf)[4], bf16x8 (&wf)[4]) {
#pragma unroll
        for (int f = 0; f < 4; ++f)
#pragma unroll
          for (int m = 0; m < 4; ++m)
            acc[m][f] = __builtin_amdgcn_mfma_f32_16x16x32_bf16(wf[f], xf[m], acc[m][f], 0, 0, 0);
      };
#define ILV() do { _Pragma("unroll") for (int i_ = 0; i_ < 4; ++i_) { \
        __builtin_amdgcn_sched_group_barrier(0x008, 1, 0); __builtin_amdgcn_sched_group_barrier(0x002, 2, 0); \
        __builtin_amdgcn_sched_group_barrier(0x100, 2, 0); } \
        __builtin_amdgcn_sched_group_barrier(0x008, 12, 0); } while (0)
      auto run = [&](int n0, int n1) {
        rd(n0, 0, x0, w0);
#pragma unroll 1
        for (int n = n0; n < n1; ++n) {
          rd(n, 1, x1, w1);
          mm(x0, w0);
          ILV();
          __builtin_amdgcn_sched_barrier(0);
          asm volatile("s_waitcnt lgkmcnt(0)\n\ts_barrier" ::: "memory");
          rd(n + 1, 0, x0, w0);
          mm(x1, w1);
          ILV();
          __builtin_amdgcn_sched_barrier(0);
        }
      };
      auto skip = [&](int cnt) {
#pragma unroll 1
        for (int n = 0; n < cnt; ++n) asm volatile("s_barrier" ::: "memory");
      };
#pragma unroll 1
      for (int seg = grp + EXP_SEG0; seg < 4; seg += 2) {
        const int n0 = (int)((0x27150e00u >> (8 * seg)) & 0xffu), n1 = (int)((0x4027150eu >> (8 * seg)) & 0xffu);
        skip((int)((0x12070f01u >> (8 * seg)) & 0xffu));
        STAMP(seg >= 2 ? 2 : 1); STAMPC(12);
        run(n0, n1);
        STAMPC(13);
      }
      STAMP(3);
      const u32 seq = rfl(seqv) + 1u;
      const u32 obase = (L == 0) ? OFF_PS + (seq & 1u) * 8u * MB + (u32)myh * 4u * MB +
                                       (u32)(((myi * 2 + myj) * 2 + (g & 1)) * 16 + (int)bb + 8 * grp) * 32768u
                                 : OFF_TL + (u32)L * MB + (u32)(grp * 16 + g) * 32768u;
#pragma unroll
      for (int m = 0; m < 4; ++m)
#pragma unroll
        for (int hf = 0; hf < 2; ++hf) {
          u32x4 o;
          o[0] = nz(pk_bf16(acc[m][2 * hf][0], acc[m][2 * hf][1]));
          o[1] = nz(pk_bf16(acc[m][2 * hf][2], acc[m][2 * hf][3]));
          o[2] = nz(pk_bf16(acc[m][2 * hf + 1][0], acc[m][2 * hf + 1][1]));
          o[3] = nz(pk_bf16(acc[m][2 * hf + 1][2], acc[m][2 * hf + 1][3]));
          __builtin_amdgcn_raw_buffer_store_b128(
              o, rs_me, l16, obase + (u32)(((wr * 2 + (m >> 1)) * 2 + wc) * 4096 + ((m & 1) * 2 + hf) * 1024), WT);
        }
      if (L != 0) {
        asm volatile("s_waitcnt vmcnt(0)" ::: "memory");
        __builtin_amdgcn_raw_buffer_store_b32(seq, rs_me, lane < 2 ? (u32)lane * 8u : OOB,
                                              OFF_TF + (u32)L * 1024u + (u32)((grp * 16 + g) * 8 + wr * 4 + wc) * 4u, WT);
      }
      STAMP(4);
    }
  }
}

}

#define HIP_CHECK(expr) do { hipError_t _e = (expr); if (_e != hipSuccess) { \
    fprintf(stderr, "HIP error %s at %s:%d\n", hipGetErrorString(_e), __FILE__, __LINE__); \
    return; } } while (0)

extern "C" void kernel_launch(int rank, int nranks, ncclComm_t comm, hipStream_t stream, void* const* d_in,
                              const int* in_sizes, int n_in, void* d_out, int out_size, const int* in_dtypes,
                              int out_dtype, void* const* d_out_all, void* const* d_ws_all, size_t ws_bytes) {
  (void)comm; (void)in_dtypes; (void)out_dtype; (void)d_out_all;
  if (nranks != 8 || (unsigned)rank >= 8u || n_in != 2 || in_sizes[0] != 4096 * 512 || in_sizes[1] != 512 * 4096 ||
      out_size != 512 * 4096 || ws_bytes < (size_t)WS_USED) {
    fprintf(stderr, "kernel_launch: unexpected problem shape\n");
    return;
  }
  Params p;
  p.x = (const float*)d_in[0];
  p.w = (const float*)d_in[1];
  p.out = (signed char*)d_out;
  for (int i = 0; i < 8; ++i) p.ws[i] = (char*)d_ws_all[i];
  p.rank = rank;
  HIP_CHECK(hipFuncSetAttribute((const void*)fused_gemm_rs_q8, hipFuncAttributeMaxDynamicSharedMemorySize, LDS_BYTES));
  fused_gemm_rs_q8<<<dim3(256), dim3(NTHREADS), LDS_BYTES, stream>>>(p);
  HIP_CHECK(hipGetLastError());
}
```

```cpp
#include <hip/hip_runtime.h>
#include <rccl/rccl.h>
#include <cstdio>

typedef unsigned u32;
typedef unsigned long long u64;
typedef u32 u32x4 __attribute__((ext_vector_type(4)));
typedef u32 u32x2 __attribute__((ext_vector_type(2)));
typedef float f32x4 __attribute__((ext_vector_type(4)));
typedef float f32x2 __attribute__((ext_vector_type(2)));
typedef __bf16 bf16x2 __attribute__((ext_vector_type(2)));
typedef __bf16 bf16x8 __attribute__((ext_vector_type(8)));

namespace {

#define NP_ 16
#define S0_SLEEP 2
#define AM_SLEEP 1
#define AM_SLOW 1
#define AM_THR 56
#define GE_ 4
#define NP1_ 20
constexpr int NPX = NP_;
constexpr int NP1 = NP1_;
constexpr u32 GE = GE_;

constexpr u32 OFF_SEQ = 0;
constexpr u32 OFF_TMO = 256;
constexpr u32 OFF_TK = 1024;
constexpr u32 OFF_AM = 4096;
constexpr u32 OFF_XF = 16384;
constexpr u32 OFF_WF = 24576;
constexpr u32 OFF_TF = 32768;
constexpr u32 OFF_PF = 40960;
constexpr u32 OFF_DBG = 65536;
constexpr u32 MB = 1u << 20;
constexpr u32 OFF_XS = 1 * MB;
constexpr u32 OFF_WS = 9 * MB;
constexpr u32 OFF_PS = 17 * MB;
constexpr u32 OFF_TL = 33 * MB;
constexpr u32 OFF_PR = 41 * MB;
constexpr u32 WS_USED = 43 * MB;
constexpr u32 SLOT_BYTES = 262144;
constexpr u32 OOB = 0x80000000u;
constexpr u64 DEADLINE = 100000000ull;
constexpr int WT = 17;
constexpr int NSTAGE = 4;
constexpr int LDS_POLL = NSTAGE * 32768;
constexpr int LDS_BYTES = LDS_POLL + 4096;
constexpr int NTHREADS = 768;
static_assert(GE <= 8 && NPX <= 32 && NP1 <= 32, "layout");

struct Params {
  char* wsx[8];
  const float* x;
  const float* w;
  signed char* out;
  int rank;
};

typedef __amdgpu_buffer_rsrc_t rsrc_t;
typedef __attribute__((address_space(3))) u32 lds_u32;

__device__ __forceinline__ rsrc_t mk_rsrc(const void* p, u32 bytes) {
  return __builtin_amdgcn_make_buffer_rsrc((void*)p, (short)0, (int)bytes, 0x00020000);
}
__device__ __forceinline__ u32 pk_bf16(float lo, float hi) {
  f32x2 v = {lo, hi};
  bf16x2 b = __builtin_convertvector(v, bf16x2);
  return __builtin_bit_cast(u32, b);
}
__device__ __forceinline__ float as_f(u32 v) { return __builtin_bit_cast(float, v); }
__device__ __forceinline__ u32 as_u(float v) { return __builtin_bit_cast(u32, v); }
__device__ __forceinline__ float bf_lo(u32 v) { return as_f(v << 16); }
__device__ __forceinline__ float bf_hi(u32 v) { return as_f(v & 0xffff0000u); }
__device__ __forceinline__ u32 rfl(u32 v) { return __builtin_amdgcn_readfirstlane(v); }
__device__ __forceinline__ u32 sgpr(u32 s) { return rfl(s); }
__device__ __forceinline__ u32 next_seq(u32 v) { return rfl(v) + 1u; }
__device__ __forceinline__ u32 min16(const u32x4 (&d)[4]) {
  u32 m = min(min(d[0][0], d[0][1]), min(d[0][2], d[0][3]));
#pragma unroll
  for (int e = 1; e < 4; ++e) m = min(min(min(m, d[e][0]), d[e][1]), min(d[e][2], d[e][3]));
  return m;
}
__device__ __forceinline__ u32 nz(u32 d) { return d == 0u ? 0x8000u : d; }
__device__ __forceinline__ u32x4 ld16(rsrc_t r, u32 v, u32 s) { return __builtin_amdgcn_raw_buffer_load_b128(r, v, s, WT); }
__device__ __forceinline__ u32 ld4(rsrc_t r, u32 v, u32 s) { return __builtin_amdgcn_raw_buffer_load_b32(r, v, s, WT); }
__device__ __forceinline__ void st16(u32x4 d, rsrc_t r, u32 v, u32 s) { __builtin_amdgcn_raw_buffer_store_b128(d, r, v, s, WT); }

struct Spin {
  u64 t0; u32 n;
  __device__ __forceinline__ void start() { t0 = __builtin_amdgcn_s_memrealtime(); n = 0; }
  __device__ __forceinline__ bool expired(u32* tmo, u32 seq) {
    if ((++n & 31u) != 0u) return false;
    u32 t = rfl(__hip_atomic_load(tmo, __ATOMIC_RELAXED, __HIP_MEMORY_SCOPE_AGENT));
    if (t == seq) return true;
    if (__builtin_amdgcn_s_memrealtime() - t0 > DEADLINE) {
      __hip_atomic_store(tmo, seq, __ATOMIC_RELAXED, __HIP_MEMORY_SCOPE_AGENT);
      return true;
    }
    return false;
  }
};

#define STAMP_AT(off, val) do {} while (0)
#define STAMPC(k) do {} while (0)
#define STAMP(k) do {} while (0)

__device__ __forceinline__ u32 wave_max_u(u32 v) {
  v = max(v, (u32)__builtin_amdgcn_update_dpp(0, (int)v, 0x111, 0xf, 0xf, true));
  v = max(v, (u32)__builtin_amdgcn_update_dpp(0, (int)v, 0x112, 0xf, 0xf, true));
  v = max(v, (u32)__builtin_amdgcn_update_dpp(0, (int)v, 0x114, 0xf, 0xf, true));
  v = max(v, (u32)__builtin_amdgcn_update_dpp(0, (int)v, 0x118, 0xf, 0xf, true));
  v = max(v, (u32)__builtin_amdgcn_update_dpp(0, (int)v, 0x142, 0xa, 0xf, true));
  v = max(v, (u32)__builtin_amdgcn_update_dpp(0, (int)v, 0x143, 0xc, 0xf, true));
  return (u32)__builtin_amdgcn_readlane((int)v, 63);
}

__device__ __forceinline__ void dma8(rsrc_t r, u32 voff, u32 soff, u32 ldsaddr) {
  u32 soff2;
  asm volatile(
      "s_mov_b32 m0, %4\n\t"
      "s_add_u32 %0, %3, 0x1000\n\t"
      "buffer_load_dwordx4 %1, %2, %3 offen sc0 sc1 lds\n\t"
      "buffer_load_dwordx4 %1, %2, %3 offen offset:1024 sc0 sc1 lds\n\t"
      "buffer_load_dwordx4 %1, %2, %3 offen offset:2048 sc0 sc1 lds\n\t"
      "buffer_load_dwordx4 %1, %2, %3 offen offset:3072 sc0 sc1 lds\n\t"
      "s_add_u32 m0, %4, 0x1000\n\t"
      "s_nop 0\n\t"
      "buffer_load_dwordx4 %1, %2, %0 offen sc0 sc1 lds\n\t"
      "buffer_load_dwordx4 %1, %2, %0 offen offset:1024 sc0 sc1 lds\n\t"
      "buffer_load_dwordx4 %1, %2, %0 offen offset:2048 sc0 sc1 lds\n\t"
      "buffer_load_dwordx4 %1, %2, %0 offen offset:3072 sc0 sc1 lds"
      : "=&s"(soff2)
      : "v"(voff), "s"(r), "s"(soff), "s"(ldsaddr)
      : "memory", "scc");
}

__global__ void __launch_bounds__(NTHREADS, 3) fused_gemm_rs_q8(Params p) {
  extern __shared__ __attribute__((aligned(16))) char smem[];
  const int b = blockIdx.x, L = b & 7, q = b >> 3;
  const int tid = threadIdx.x, lane = tid & 63;
  const int wave = (int)rfl((u32)tid >> 6);
  const int rank = p.rank & 7;
  char* myws = p.wsx[0];
  asm volatile("" :: "s"(p.x), "s"(p.w), "s"(p.out));
  const rsrc_t rs_me = mk_rsrc(myws, WS_USED);
  u32* tmo = (u32*)(myws + OFF_TMO);
  const int peer = rank ^ L;
  const rsrc_t rs_peer = mk_rsrc(p.wsx[L], WS_USED);
  const u32 seqv = __hip_atomic_load((u32*)(myws + OFF_SEQ), __ATOMIC_RELAXED, __HIP_MEMORY_SCOPE_AGENT);
  const u32 myh = (u32)rank & 1u;
  const u32 myblk = (u32)((((rank >> 1) & 1) * 2 + ((rank >> 2) & 1)) * 32);
  const u32 l16 = (u32)lane * 16u;
  STAMP(0);

  if (q < 16) {
    const bool pusher = wave >= 8;
    const int NP = L == 1 ? NP1 : NPX;
    if (pusher && (L == 0 || (wave - 8) * 16 + q >= NP)) return;
    {
      const u32 oid = (u32)((q * 2 + L) * 8 + wave);
      const int vp = pusher ? peer : rank ^ (int)(oid & 1u);
      const u32 dr = (u32)(pusher ? peer : rank);
      const rsrc_t rs_x = mk_rsrc(p.x, 4096u * 512u * 4u);
      const rsrc_t rs_w = mk_rsrc(p.w, 512u * 4096u * 4u);
      const rsrc_t rs_dst = pusher ? rs_peer : rs_me;
      const u32 pi = (vp >> 1) & 1, pj = (vp >> 2) & 1, ph = vp & 1;
      const u32 slotbase = 4u * (u32)(pusher ? rank : vp);
      const u32 vx = (u32)(lane >> 3) * 2048u + (u32)(lane & 7) * 16u;
      const u32 vw = (u32)(lane >> 3) * 65536u + (u32)(lane & 7) * 16u;
      const u32 dx0 = (u32)(lane >> 3) * 128u + (u32)(((lane & 7) ^ (lane >> 4)) * 16);
      const u32 fl_lane = lane == 0 ? 0u : OOB;
      u32x4 ld[8];
      auto desc = [&](u32 v, u32& isw, u32& src, u32& dst, u32& flg) {
        constexpr u32 N0 = 12u + 2u * GE, NA = 36u - 2u * GE;
        u32 t, r;
        if (v < 4u * N0) {
          t = v / N0; const u32 e = v % N0;
          isw = e >= 12u;
          r = isw ? e - 12u : (((dr ^ (2u + (e >> 1))) * 2u + (dr & 1u)) * 2u + (e & 1u));
        } else if (v < 192u) {
          const u32 v2 = v - 4u * N0;
          t = v2 / NA; const u32 e = v2 % NA;
          isw = e >= 20u;
          r = isw ? 2u * GE + e - 20u
                  : (e < 12u ? (((dr ^ (2u + (e >> 1))) * 2u + ((dr & 1u) ^ 1u)) * 2u + (e & 1u)) : (dr & 6u) * 4u + e - 12u);
        } else {
          t = (v - 192u) >> 4; r = 16u + (v & 15u); isw = 1u;
        }
        const u32 slot = slotbase + t, k0 = ph * 256u + t * 64u;
        if (isw) {
          src = k0 * 16384u + (pj * 2048u + r * 64u) * 4u;
          dst = OFF_WS + slot * SLOT_BYTES + r * 8192u;
          flg = OFF_WF + (r >> 1) * 512u + slot * 16u + (r & 1u) * 8u;
        } else {
          src = ((r >> 2) * 512u + pi * 256u + (r & 3u) * 64u) * 2048u + k0 * 4u;
          dst = OFF_XS + slot * SLOT_BYTES + r * 8192u;
          flg = OFF_XF + (r >> 1) * 512u + slot * 16u + (r & 1u) * 8u;
        }
      };
      auto loads = [&](u32 isw, u32 src) {
        if (isw) {
#pragma unroll
          for (int j = 0; j < 8; ++j)
            ld[j] = __builtin_amdgcn_raw_buffer_load_b128(rs_w, vw, src + (u32)((j & 3) + 32 * (j >> 2)) * 16384u, 0);
        } else {
#pragma unroll
          for (int j = 0; j < 8; ++j)
            ld[j] = __builtin_amdgcn_raw_buffer_load_b128(rs_x, vx, src + (u32)(j >> 1) * 16384u + (u32)(j & 1) * 128u, 0);
        }
      };
      u32 seq;
      auto fdesc = [&](u32 f, u32& s1, u32& f1, u32& s2, u32& f2, u32& dst, u32& dfl) {
        const u32 reg = f & 7u;
        u32 sl = (u32)L, tile, g;
        s2 = OOB; dfl = OOB;
        bool direct = true;
        if (L == 1) {
          if (f < 48u * GE) {
            const u32 k = (f >> 3) % GE;
            sl = 2u + (f >> 3) / GE; tile = 0u; g = 2u * k + myh;
            dst = OFF_PR + (((sl ^ 1u) * 64u) + k * 8u + reg) * 4096u;
            dfl = OFF_PF + (((sl ^ 1u) * 64u) + k * 8u + reg) * 4u;
            direct = false;
          } else { const u32 k = (f - 48u * GE) >> 3; tile = k >> 4; g = k & 15u; }
        } else {
          const u32 k = f >> 3;
          if (k < 16u - 2u * GE) { tile = 0u; g = 2u * GE + k; }
          else if (k < 16u - GE) {
            const u32 kk = k - (16u - 2u * GE);
            tile = 0u; g = 2u * kk + (myh ^ 1u);
            s2 = OFF_PR + ((u32)L * 64u + kk * 8u + reg) * 4096u;
          } else { tile = 1u; g = k - (16u - GE); }
        }
        const u32 fu = tile * 128u + g * 8u + reg;
        s1 = OFF_TL + sl * MB + fu * 4096u;
        f1 = OFF_TF + sl * 1024u + fu * 4u;
        f2 = (s2 == OOB) ? f1 : OFF_PF + ((s2 - OFF_PR) >> 10);
        if (direct)
          dst = OFF_PS + (seq & 1u) * 8u * MB + myh * 4u * MB + ((myblk + (g & 1u) * 16u + (g >> 1) + tile * 8u) * 8u + reg) * 4096u;
      };
      u32 s1, f1, s2, f2, fdst, fdfl;
      const u32 two_lanes = lane < 2 ? 0u : OOB;
      auto fflags = [&]() { return ld4(rs_me, (lane == 1 ? f2 : f1) | two_lanes, 0u); };
      auto fready = [&](u32 v) { return __all(lane >= 2 || v == seq); };
      auto floads = [&]() {
#pragma unroll
        for (int e = 0; e < 4; ++e) ld[e] = ld16(rs_me, l16, s1 + (u32)e * 1024u);
        if (s2 != OOB) {
#pragma unroll
          for (int e = 0; e < 4; ++e) ld[4 + e] = ld16(rs_me, l16, s2 + (u32)e * 1024u);
        }
      };
      const u32 nin = (L == 1) ? 0u : 256u;
      const u32 ntot = (L == 1) ? 48u * GE + 256u : 512u - 8u * GE;
      auto draw = [&]() {
        return (u32)__builtin_amdgcn_raw_ptr_buffer_atomic_add_i32(1, rs_me, (int)fl_lane, (int)(OFF_TK + (u32)L * 128u), 0);
      };
      u32 v = pusher ? (u32)((wave - 8) * 16 + q) : (L < 2 ? oid >> 1 : 1000u);
      u32 vn = v + (pusher ? (u32)NP : 128u);
      u32 tkv = 0u, tkbase, pflg = OOB;
      bool have = false;
      auto emit = [&](u32 isw, u32 dst) {
        u32x4 o[4];
        if (isw) {
#pragma unroll
          for (int c = 0; c < 4; ++c)
#pragma unroll
            for (int d = 0; d < 4; ++d) o[c][d] = pk_bf16(as_f(ld[2 * d][c]), as_f(ld[2 * d + 1][c]));
#pragma unroll
          for (int c = 0; c < 4; ++c) st16(o[c], rs_dst, (u32)((lane ^ c) * 16), dst + (u32)c * 1024u);
        } else {
#pragma unroll
          for (int g = 0; g < 4; ++g) {
            o[g][0] = pk_bf16(as_f(ld[2 * g][0]), as_f(ld[2 * g][1]));
            o[g][1] = pk_bf16(as_f(ld[2 * g][2]), as_f(ld[2 * g][3]));
            o[g][2] = pk_bf16(as_f(ld[2 * g + 1][0]), as_f(ld[2 * g + 1][1]));
            o[g][3] = pk_bf16(as_f(ld[2 * g + 1][2]), as_f(ld[2 * g + 1][3]));
          }
#pragma unroll
          for (int g = 0; g < 4; ++g) st16(o[g], rs_dst, dx0 ^ (u32)((g & 1) * 64), dst + (u32)g * 1024u);
        }
        asm volatile("" ::: "memory");
      };
      if (pusher ? L != 1 : L < 2) {
        u32 isw, src, dst, flg;
        desc(v, isw, src, dst, flg);
        loads(isw, src);
        seq = next_seq(seqv);
        tkbase = (seq - 1u) * (ntot - (u32)NP) - (u32)(2 * NP);
        tkv = vn + tkbase;
        const u32x2 seq2 = {seq, seq};
#pragma unroll 1
        for (;;) {
          asm volatile("s_waitcnt vmcnt(0)" ::: "memory");
          emit(isw, dst);
          __builtin_amdgcn_raw_buffer_store_b64(seq2, rs_dst, fl_lane | pflg, 0, WT);
          loads(isw, src + (isw ? 128u : 65536u));
          vn = rfl(tkv) - tkbase;
          if (vn >= 256u) break;
          u32 nisw, nsrc, ndst, nflg;
          desc(vn, nisw, nsrc, ndst, nflg);
          asm volatile("s_waitcnt vmcnt(0)" ::: "memory");
          emit(isw, dst + 4096u);
          loads(nisw, nsrc);
          tkv = pusher ? draw() : 1000u + tkbase;
          pflg = flg;
          isw = nisw; src = nsrc; dst = ndst; flg = nflg;
        }
        v = vn;
        if (pusher) fdesc(v - 256u, s1, f1, s2, f2, fdst, fdfl);
        asm volatile("s_waitcnt vmcnt(0)" ::: "memory");
        const u32 nfl = pusher ? fflags() : 0u;
        emit(isw, dst + 4096u);
        pflg = flg;
        if (pusher) {
          have = fready(nfl);
          asm volatile("" ::: "memory");
          if (have) floads();
          tkv = draw();
        } else {
          asm volatile("s_waitcnt vmcnt(0)" ::: "memory");
          __builtin_amdgcn_raw_buffer_store_b64(seq2, rs_dst, fl_lane | pflg, 0, WT);
        }
      } else {
        seq = next_seq(seqv);
        tkbase = (seq - 1u) * (ntot - (u32)NP) - (u32)(2 * NP);
        tkv = vn + tkbase;
        if (pusher) fdesc(v, s1, f1, s2, f2, fdst, fdfl);
      }
      STAMP(1);
      if (pusher) {
        u32 pfl2 = pflg == OOB ? OOB : pflg + 4u;
        for (;;) {
          asm volatile("s_waitcnt vmcnt(0)" ::: "memory");
          __builtin_amdgcn_raw_buffer_store_b32(seq, rs_peer, (lane == 1 ? pfl2 : pflg) | two_lanes, 0, WT);
          pfl2 = OOB;
          vn = rfl(tkv) - tkbase;
          if (!have) {
            Spin sp; sp.start();
            for (;;) {
              if (fready(fflags())) break;
              if (sp.expired(tmo, seq)) break;
              __builtin_amdgcn_s_sleep(4);
            }
            asm volatile("" ::: "memory");
            floads();
          }
          const u32 cdst = fdst, cs2 = s2;
          pflg = fdfl;
          STAMP_AT(524288u + ((u32)L * 512u + v - nin) * 8u, __builtin_amdgcn_s_memrealtime());
          u32 nfl = 0u;
          if (vn < ntot) {
            fdesc(vn - nin, s1, f1, s2, f2, fdst, fdfl);
            nfl = fflags();
          }
          if (cs2 != OOB) {
#pragma unroll
            for (int e = 0; e < 4; ++e)
#pragma unroll
              for (int d = 0; d < 4; ++d)
                ld[e][d] = nz(pk_bf16(bf_lo(ld[e][d]) + bf_lo(ld[4 + e][d]), bf_hi(ld[e][d]) + bf_hi(ld[4 + e][d])));
          }
#pragma unroll
          for (int e = 0; e < 4; ++e) st16(ld[e], rs_peer, l16, cdst + (u32)e * 1024u);
          if (vn >= ntot) break;
          v = vn;
          have = fready(nfl);
          asm volatile("" ::: "memory");
          if (have) floads();
          tkv = draw();
        }
        STAMP(4);
        asm volatile("s_waitcnt vmcnt(0)" ::: "memory");
        __builtin_amdgcn_raw_buffer_store_b32(seq, rs_peer, fl_lane | pflg, 0, WT);
        STAMP(5);
        return;
      }
    }
    volatile lds_u32* lds = (volatile lds_u32*)smem;
    const u32 seq = next_seq(seqv);
    const u32 par = seq & 1u;
    const u32 cwi = (u32)((L - 1) * 16 + q);
    const u32 cw = L == 0 ? (u32)((q >> 1) * 16 + 14 + (q & 1)) : (cwi / 14u) * 16u + cwi % 14u;
    const u32 bcol = cw & 15u, ra = (cw >> 4) & 1u, tj = (cw >> 5) & 1u, ti = (cw >> 6) & 1u;
    const u32 ooff = (ti * 256u + ra * 128u + (u32)(wave >> 1) * 32u + (u32)(lane & 15)) * 4096u + tj * 2048u + bcol * 128u +
                     (u32)(wave & 1) * 64u + (u32)(lane >> 4) * 16u;
    const rsrc_t rs_out = mk_rsrc(p.out, 512u * 4096u);
    u64* am_dst = (u64*)(p.wsx[lane & 7] + OFF_AM + ((u32)rank * 128u + cw) * 8u);
    {
      u32 t0 = __builtin_amdgcn_raw_buffer_load_b32(rs_out, ooff, 0, 0);
      u32 t1 = 0u;
      if (tid < 8) t1 = (u32)__hip_atomic_load(am_dst, __ATOMIC_RELAXED, __HIP_MEMORY_SCOPE_SYSTEM);
      u32 t2 = ld4(rs_peer, tid == 64 ? sgpr(OFF_PS + par * 8u * MB + myh * 4u * MB + myblk * 32768u) : (tid == 65 ? OFF_PR : OOB), 0u);
      asm volatile("" :: "v"(t0), "v"(t1), "v"(t2));
    }
    if (L == 0) {
      const u32x4 zero4 = {0u, 0u, 0u, 0u};
      const u32 zo = sgpr(OFF_PS + (par ^ 1u) * 8u * MB + (u32)(q * 8 + wave) * 65536u);
#pragma unroll 1
      for (u32 k = 0; k < 16u; ++k) {
        __builtin_amdgcn_s_sleep(40);
#pragma unroll
        for (int e = 0; e < 4; ++e) st16(zero4, rs_me, l16, zo + k * 4096u + (u32)e * 1024u);
      }
      STAMP(12);
    }
    float y[2][16];
    {
      const u32 so = sgpr(OFF_PS + par * 8u * MB + (cw * 8u + (u32)wave) * 4096u);
      const bool pre = bcol < GE && (tj * 2u + ti) != ((u32)rank >> 1);
      bool ok0 = pre && ra == 0u, ok1 = pre && ra == 1u;
      u32x4 d0[4], d1[4];
#pragma unroll
      for (int e = 0; e < 4; ++e) { d0[e] = (u32x4){0u, 0u, 0u, 0u}; d1[e] = (u32x4){0u, 0u, 0u, 0u}; }
      Spin sp; sp.start();
      {
        const u32 pv = (lane == 1 ? (ok1 ? OOB : 4u * MB + 4092u) : (ok0 ? OOB : 4092u)) | (lane < 2 ? 0u : OOB);
        for (;;) {
          const u32 v = ld4(rs_me, pv, so);
          if (__all(pv >= OOB || v != 0u)) break;
          if (sp.expired(tmo, seq)) break;
          __builtin_amdgcn_s_sleep(S0_SLEEP);
        }
      }
      STAMP(8); STAMP_AT(((blockIdx.x * 12 + wave) * 16 + 10) * 8, (u64)sp.n);
      for (;;) {
        if (!ok0) {
#pragma unroll
          for (int e = 0; e < 4; ++e) d0[e] = ld16(rs_me, l16, so + (u32)e * 1024u);
        }
        if (!ok1) {
#pragma unroll
          for (int e = 0; e < 4; ++e) d1[e] = ld16(rs_me, l16, so + 4u * MB + (u32)e * 1024u);
        }
        ok0 = ok0 || __all(min16(d0) != 0u);
        ok1 = ok1 || __all(min16(d1) != 0u);
        if (ok0 && ok1) break;
        if (sp.expired(tmo, seq)) break;
        __builtin_amdgcn_s_sleep(1);
      }
      STAMP(4); STAMP_AT(((blockIdx.x * 12 + wave) * 16 + 9) * 8, (u64)sp.n);
      float mx = 0.f;
#pragma unroll
      for (int e = 0; e < 4; ++e)
#pragma unroll
        for (int d = 0; d < 4; ++d) {
          const float lo = bf_lo(d0[e][d]) + bf_lo(d1[e][d]);
          const float hi = bf_hi(d0[e][d]) + bf_hi(d1[e][d]);
          y[e >> 1][(e & 1) * 8 + d * 2] = lo;
          y[e >> 1][(e & 1) * 8 + d * 2 + 1] = hi;
          mx = fmaxf(mx, fmaxf(fabsf(lo), fabsf(hi)));
        }
      lds[8 + wave] = wave_max_u(as_u(mx));
      asm volatile("s_waitcnt lgkmcnt(0)\n\ts_barrier" ::: "memory");
      if (wave == 0 && lane < 8) {
        u32 m = lds[8];
#pragma unroll
        for (int k = 1; k < 8; ++k) m = max(m, lds[8 + k]);
        __hip_atomic_store(am_dst, ((u64)seq << 32) | (u64)m, __ATOMIC_RELAXED, __HIP_MEMORY_SCOPE_SYSTEM);
      }
    }
    STAMP(5);
    {
      Spin sp; sp.start();
      u32x4 g;
      u32 bad;
      for (;;) {
        g = ld16(rs_me, (u32)tid * 16u, OFF_AM);
        bad = ld4(rs_me, 0u, OFF_TMO);
        if (__all(g[1] == seq && g[3] == seq)) break;
        if (sp.expired(tmo, seq)) { bad = seq; break; }
        if (__popcll(__ballot(g[1] == seq && g[3] == seq)) < AM_THR) __builtin_amdgcn_s_sleep(AM_SLOW);
        else __builtin_amdgcn_s_sleep(AM_SLEEP);
      }
      STAMP(11);
      lds[16 + wave] = (rfl(bad) == seq) ? 0xffffffffu : wave_max_u(max(g[0], g[2]));
    }
    asm volatile("s_waitcnt lgkmcnt(0)\n\ts_barrier" ::: "memory");
    STAMP(6);
    {
      u32 m = lds[16];
#pragma unroll
      for (int k = 1; k < 8; ++k) m = max(m, lds[16 + k]);
      const bool anybad = m == 0xffffffffu;
      const float inv = m ? 127.0f / as_f(m) : 0.f;
#pragma unroll
      for (int mm = 0; mm < 2; ++mm) {
        u32x4 o;
#pragma unroll
        for (int d = 0; d < 4; ++d) {
          u32 t[4];
#pragma unroll
          for (int k = 0; k < 4; ++k)
            t[k] = as_u(__builtin_amdgcn_fmed3f(__builtin_fmaf(y[mm][d * 4 + k], inv, 12582912.f), 12582912.f - 127.f, 12582912.f + 127.f));
          const u32 wd = __builtin_amdgcn_perm(__builtin_amdgcn_perm(t[3], t[2], 0x0c0c0400u),
                                               __builtin_amdgcn_perm(t[1], t[0], 0x0c0c0400u), 0x05040100u);
          o[d] = anybad ? 0x80808080u : wd;
        }
        __builtin_amdgcn_raw_buffer_store_b128(o, rs_out, ooff, (u32)mm * 65536u, 0);
      }
    }
    if (b == 0 && tid == 0) __hip_atomic_store((u32*)(myws + OFF_SEQ), seq, __ATOMIC_RELAXED, __HIP_MEMORY_SCOPE_AGENT);
    STAMP(7);
  } else {
    const int g = q - 16;
    const u32 bb = (u32)(g >> 1);
    const bool early = L >= 2 && bb < GE && (u32)(g & 1) == myh;
    if (wave >= 8) {
      __builtin_amdgcn_s_setprio(3);
      const int j = wave - 8;
      const u32 a = (u32)(2 * peer + (g & 1));
      u32 goff, foff;
      {
        const int ROT8 = (int)bb + 4 * (g & 1), ROT6 = (int)bb + 3 * (g & 1);
        const int n = lane;
        int k1, k2, r;
        if (early) {
          k1 = (n >= 12 && n < 20) ? n - 12 : -1;
          k2 = (n >= 32 && n < 40) ? n - 32 : -1;
          r = n < 12 ? n : (n < 32 ? n - 8 : n - 40);
        } else {
          k1 = n < 8 ? n : -1;
          k2 = (n >= 8 && n < 15) ? n - 8 : (n == 39 ? 7 : -1);
          r = n < 39 ? n - 15 : n - 40;
        }
        const u32 t2 = (k2 >= 0 || n >= 40) ? 1u : 0u;
        u32 slot;
        if (k1 >= 0) slot = (u32)(4 * (rank ^ ((k1 + g) & 1)) + (k1 >> 1));
        else if (k2 >= 0) { const int k = (k2 + ROT8) & 7; slot = (u32)(4 * (rank ^ (k >> 2)) + (k & 3)); }
        else slot = (u32)(4 * (rank ^ (2 + (r + ROT6) % 6)) + r / 6);
        const u32 bcol = bb + 8u * t2;
        if (j < 2) {
          goff = OFF_XS + slot * SLOT_BYTES + (a * 4u + (u32)(2 * j)) * 4096u;
          foff = OFF_XF + a * 512u + slot * 16u;
        } else {
          goff = OFF_WS + slot * SLOT_BYTES + (bcol * 4u + (u32)(2 * (j - 2))) * 4096u;
          foff = OFF_WF + bcol * 512u + slot * 16u;
        }
      }
      const u32 seq = next_seq(seqv);
      const u32 lds_base0 = rfl((u32)(size_t)(__attribute__((address_space(3))) char*)smem);
      const u32 lds_base = lds_base0 + (u32)j * 8192u;
      int cov = 0;
      int iss = 0, since = 64;
      bool pend = false;
      const u32 poll_lds = lds_base0 + (u32)LDS_POLL + (u32)j * 1024u;
      const volatile lds_u32* pl = (const volatile lds_u32*)(smem + LDS_POLL + j * 1024 + lane * 16 + (j & 1) * 8);
      auto prefix = [&](u32 f) {
        const u64 miss = __ballot((int)(f - seq) < 0);
        return miss ? (int)__builtin_ctzll(miss) : 64;
      };
      auto issue = [&](int k) {
        u32 go = (u32)__builtin_amdgcn_readlane((int)goff, k);
        asm volatile("s_nop 4" : "+s"(go));
        dma8(rs_me, l16, go, lds_base + (u32)(k & (NSTAGE - 1)) * 32768u);
      };
#pragma unroll 1
      for (int n = 0; n < 64; ++n) {
        if (iss <= n) {
          if (cov <= n) {
            Spin sp; sp.start();
            for (;;) {
              cov = prefix(ld4(rs_me, foff + (u32)(j & 1) * 8u, 0u));
              if (cov > n) break;
              if (sp.expired(tmo, seq)) { cov = 64; break; }
              __builtin_amdgcn_s_sleep(1);
            }
          }
          issue(n);
          iss = n + 1;
        }
        int k = iss - n - 1;
        if (pend && since < k) k = since;
        const int lim = (n + NSTAGE < 64) ? n + NSTAGE : 64;
        if (k >= 2) asm volatile("s_waitcnt vmcnt(16)\n\ts_barrier" ::: "memory");
        else if (k == 1) asm volatile("s_waitcnt vmcnt(8)\n\ts_barrier" ::: "memory");
        else asm volatile("s_waitcnt vmcnt(0)\n\ts_barrier" ::: "memory");
        if (pend) {
          const int c = prefix(pl[0]);
          cov = c > cov ? c : cov;
          if (cov == 64) cov = 1000;
          pend = false;
        }
        if (cov < lim + 4) {
          asm volatile("s_waitcnt lgkmcnt(0)\n\t"
                       "s_mov_b32 m0, %2\n\t"
                       "s_nop 0\n\t"
                       "buffer_load_dwordx4 %0, %1, 0 offen sc0 sc1 lds"
                       :: "v"(foff), "s"(rs_me), "s"(poll_lds) : "memory");
          pend = true;
          since = 0;
        }
        const int top = lim < cov ? lim : cov;
        while (iss < top) { issue(iss); ++iss; ++since; }
        if (!pend && iss == n + NSTAGE) {
          const int last = (cov - 8 < 64 - NSTAGE) ? cov - 8 : 64 - NSTAGE;
          u32 la = (u32)(iss & (NSTAGE - 1)) * 32768u;
          u32 go = (u32)__builtin_amdgcn_readlane((int)goff, iss & 63);
#pragma unroll 1
          while (n < last) {
            asm volatile("s_waitcnt vmcnt(16)\n\ts_barrier" : "+s"(go) :: "memory");
            dma8(rs_me, l16, go, lds_base + la);
            ++n; ++iss;
            la = (la + 32768u) & (u32)(NSTAGE * 32768 - 1);
            go = (u32)__builtin_amdgcn_readlane((int)goff, iss & 63);
          }
        }
      }
    } else {
      const int grp = wave >> 2, wr = (wave >> 1) & 1, wc = wave & 1, fi = lane & 15, kq = lane >> 4;
      u32 ax[2], aw[4];
#pragma unroll
      for (int ks = 0; ks < 2; ++ks) ax[ks] = (u32)((wr * 64 + fi) * 128 + (((kq + 4 * ks) ^ (fi >> 1)) * 16));
#pragma unroll
      for (int f = 0; f < 4; ++f)
        aw[f] = (u32)(16384 + wc * 8192 + (fi >> 3) * 4096 + (fi & 3) * 1024 + ((kq * 8 + 4 * ((fi >> 2) & 1) + (f ^ (fi & 3))) * 16));
      f32x4 acc[4][4];
#pragma unroll
      for (int m = 0; m < 4; ++m)
#pragma unroll
        for (int f = 0; f < 4; ++f) acc[m][f] = (f32x4){0.f, 0.f, 0.f, 0.f};
      bf16x8 x0[4], w0[4], x1[4], w1[4];
      auto rd = [&](int n, int ks, bf16x8 (&xf)[4], bf16x8 (&wf)[4]) {
        const char* st = smem + (n & (NSTAGE - 1)) * 32768;
#pragma unroll
        for (int m = 0; m < 4; ++m) {
          wf[m] = *(const bf16x8*)(st + aw[m] + ks * 512);
          xf[m] = *(const bf16x8*)(st + ax[ks] + m * 2048);
        }
      };
      auto mm = [&](bf16x8 (&xf)[4], bf16x8 (&wf)[4]) {
#pragma unroll
        for (int f = 0; f < 4; ++f)
#pragma unroll
          for (int m = 0; m < 4; ++m)
            acc[m][f] = __builtin_amdgcn_mfma_f32_16x16x32_bf16(wf[f], xf[m], acc[m][f], 0, 0, 0);
      };
#define SGB(m, n) __builtin_amdgcn_sched_group_barrier(m, n, 0)
#define ILV() do { SGB(0x002, 5); SGB(0x100, 2); _Pragma("unroll") for (int i_ = 0; i_ < 6; ++i_) { SGB(0x008, 1); SGB(0x100, 1); } SGB(0x008, 10); } while (0)
      const u32 bounds = early ? 0x40402000u : 0x270f0800u;
      int nb = 0;
#pragma unroll 1
      for (int seg = grp; seg < 4; seg += 2) {
        const int n0 = (int)((bounds >> (8 * seg)) & 0xffu), n1 = seg == 3 ? 64 : (int)((bounds >> (8 * seg + 8)) & 0xffu);
        if (n0 >= n1) continue;
#pragma unroll 1
        for (; nb <= n0; ++nb) asm volatile("s_barrier" ::: "memory");
        STAMP(seg >= 2 ? 2 : 1); STAMPC(12);
        rd(n0, 0, x0, w0);
#pragma unroll 1
        for (int n = n0; n < n1; ++n) {
          rd(n, 1, x1, w1);
          mm(x0, w0);
          ILV();
          __builtin_amdgcn_sched_barrier(0);
          asm volatile("s_waitcnt lgkmcnt(0)\n\ts_barrier" ::: "memory");
          rd(n + 1, 0, x0, w0);
          mm(x1, w1);
          ILV();
          __builtin_amdgcn_sched_barrier(0);
        }
        nb = n1 + 1;
        STAMPC(13);
      }
      STAMP(3);
#define JOIN0 0
#define JOIN1 2
      if (grp == 0) for (int k = 0; k < JOIN0; ++k) asm volatile("s_barrier" ::: "memory");
      const u32 seq = next_seq(seqv);
      const u32 obase = (L == 0) ? OFF_PS + (seq & 1u) * 8u * MB + myh * 4u * MB + (myblk + (u32)((g & 1) * 16 + 8 * grp) + bb) * 32768u
                                 : OFF_TL + (u32)L * MB + (u32)(grp * 16 + g) * 32768u;
#pragma unroll
      for (int m = 0; m < 4; ++m)
#pragma unroll
        for (int hf = 0; hf < 2; ++hf) {
          u32x4 o;
          o[0] = nz(pk_bf16(acc[m][2 * hf][0], acc[m][2 * hf][1]));
          o[1] = nz(pk_bf16(acc[m][2 * hf][2], acc[m][2 * hf][3]));
          o[2] = nz(pk_bf16(acc[m][2 * hf + 1][0], acc[m][2 * hf + 1][1]));
          o[3] = nz(pk_bf16(acc[m][2 * hf + 1][2], acc[m][2 * hf + 1][3]));
          st16(o, rs_me, l16, obase + (u32)(((wr * 2 + (m >> 1)) * 2 + wc) * 4096 + ((m & 1) * 2 + hf) * 1024));
        }
      if (grp == 0) for (int k = 0; k < JOIN1; ++k) asm volatile("s_barrier" ::: "memory");
      if (L != 0) {
        asm volatile("s_waitcnt vmcnt(0)" ::: "memory");
        __builtin_amdgcn_raw_buffer_store_b32(seq, rs_me, lane < 2 ? (u32)lane * 8u : OOB,
                                              OFF_TF + (u32)L * 1024u + (u32)((grp * 16 + g) * 8 + wr * 4 + wc) * 4u, WT);
      }
      STAMP(4);
    }
  }
}

}

#define HIP_CHECK(expr) do { hipError_t _e = (expr); if (_e != hipSuccess) { \
    fprintf(stderr, "HIP error %s at %s:%d\n", hipGetErrorString(_e), __FILE__, __LINE__); \
    return; } } while (0)

extern "C" void kernel_launch(int rank, int nranks, ncclComm_t comm, hipStream_t stream, void* const* d_in,
                              const int* in_sizes, int n_in, void* d_out, int out_size, const int* in_dtypes,
                              int out_dtype, void* const* d_out_all, void* const* d_ws_all, size_t ws_bytes) {
  (void)comm; (void)in_dtypes; (void)out_dtype; (void)d_out_all;
  if (nranks != 8 || (unsigned)rank >= 8u || n_in != 2 || in_sizes[0] != 4096 * 512 || in_sizes[1] != 512 * 4096 ||
      out_size != 512 * 4096 || ws_bytes < (size_t)WS_USED) {
    fprintf(stderr, "kernel_launch: unexpected problem shape\n");
    return;
  }
  Params p;
  for (int i = 0; i < 8; ++i) p.wsx[i] = (char*)d_ws_all[rank ^ i];
  p.x = (const float*)d_in[0];
  p.w = (const float*)d_in[1];
  p.out = (signed char*)d_out;
  p.rank = rank;
  HIP_CHECK(hipFuncSetAttribute((const void*)fused_gemm_rs_q8, hipFuncAttributeMaxDynamicSharedMemorySize, LDS_BYTES));
  fused_gemm_rs_q8<<<dim3(256), dim3(NTHREADS), LDS_BYTES, stream>>>(p);
  HIP_CHECK(hipGetLastError());
}
```

```cpp
#include <hip/hip_runtime.h>
#include <rccl/rccl.h>
#include <cstdio>

typedef unsigned u32;
typedef unsigned long long u64;
typedef u32 u32x4 __attribute__((ext_vector_type(4)));
typedef u32 u32x2 __attribute__((ext_vector_type(2)));
typedef float f32x4 __attribute__((ext_vector_type(4)));
typedef float f32x2 __attribute__((ext_vector_type(2)));
typedef __bf16 bf16x2 __attribute__((ext_vector_type(2)));
typedef __bf16 bf16x8 __attribute__((ext_vector_type(8)));

namespace {

#define NP_ 16
#define S0_SLEEP 2
#define AM_SLEEP 1
#define AM_SLOW 1
#define AM_THR 56
#define PRE_COLS 2u
#define SPEC_THR 384
#define GE_ 4
#define NP1_ 20
constexpr int NPX = NP_;
constexpr int NP1 = NP1_;
constexpr u32 GE = GE_;

constexpr u32 OFF_SEQ = 0;
constexpr u32 OFF_TMO = 256;
constexpr u32 OFF_TK = 1024;
constexpr u32 OFF_AM = 4096;
constexpr u32 OFF_XF = 16384;
constexpr u32 OFF_WF = 24576;
constexpr u32 OFF_TF = 32768;
constexpr u32 OFF_PF = 40960;
constexpr u32 OFF_DBG = 65536;
constexpr u32 MB = 1u << 20;
constexpr u32 OFF_XS = 1 * MB;
constexpr u32 OFF_WS = 9 * MB;
constexpr u32 OFF_PS = 17 * MB;
constexpr u32 OFF_TL = 33 * MB;
constexpr u32 OFF_PR = 41 * MB;
constexpr u32 WS_USED = 43 * MB;
constexpr u32 SLOT_BYTES = 262144;
constexpr u32 OOB = 0x80000000u;
constexpr u64 DEADLINE = 100000000ull;
constexpr int WT = 17;
constexpr int NSTAGE = 4;
constexpr int LDS_POLL = NSTAGE * 32768;
constexpr int LDS_BYTES = LDS_POLL + 4096;
constexpr int NTHREADS = 768;
static_assert(GE <= 8 && NPX <= 32 && NP1 <= 32, "layout");

struct Params {
  char* wsx[8];
  const float* x;
  const float* w;
  signed char* out;
  int rank;
};

typedef __amdgpu_buffer_rsrc_t rsrc_t;
typedef __attribute__((address_space(3))) u32 lds_u32;

__device__ __forceinline__ rsrc_t mk_rsrc(const void* p, u32 bytes) {
  return __builtin_amdgcn_make_buffer_rsrc((void*)p, (short)0, (int)bytes, 0x00020000);
}
__device__ __forceinline__ u32 pk_bf16(float lo, float hi) {
  f32x2 v = {lo, hi};
  bf16x2 b = __builtin_convertvector(v, bf16x2);
  return __builtin_bit_cast(u32, b);
}
__device__ __forceinline__ float as_f(u32 v) { return __builtin_bit_cast(float, v); }
__device__ __forceinline__ u32 as_u(float v) { return __builtin_bit_cast(u32, v); }
__device__ __forceinline__ float bf_lo(u32 v) { return as_f(v << 16); }
__device__ __forceinline__ float bf_hi(u32 v) { return as_f(v & 0xffff0000u); }
__device__ __forceinline__ u32 rfl(u32 v) { return __builtin_amdgcn_readfirstlane(v); }
__device__ __forceinline__ u32 sgpr(u32 s) { return rfl(s); }
__device__ __forceinline__ u32 next_seq(u32 v) { return rfl(v) + 1u; }
__device__ __forceinline__ u32 min16(const u32x4 (&d)[4]) {
  u32 m = min(min(d[0][0], d[0][1]), min(d[0][2], d[0][3]));
#pragma unroll
  for (int e = 1; e < 4; ++e) m = min(min(min(m, d[e][0]), d[e][1]), min(d[e][2], d[e][3]));
  return m;
}
__device__ __forceinline__ u32 nz(u32 d) { return d == 0u ? 0x8000u : d; }
__device__ __forceinline__ u32x4 ld16(rsrc_t r, u32 v, u32 s) { return __builtin_amdgcn_raw_buffer_load_b128(r, v, s, WT); }
__device__ __forceinline__ u32 ld4(rsrc_t r, u32 v, u32 s) { return __builtin_amdgcn_raw_buffer_load_b32(r, v, s, WT); }
__device__ __forceinline__ void st16(u32x4 d, rsrc_t r, u32 v, u32 s) { __builtin_amdgcn_raw_buffer_store_b128(d, r, v, s, WT); }

struct Spin {
  u64 t0; u32 n;
  __device__ __forceinline__ void start() { t0 = __builtin_amdgcn_s_memrealtime(); n = 0; }
  __device__ __forceinline__ bool expired(u32* tmo, u32 seq) {
    if ((++n & 31u) != 0u) return false;
    u32 t = rfl(__hip_atomic_load(tmo, __ATOMIC_RELAXED, __HIP_MEMORY_SCOPE_AGENT));
    if (t == seq) return true;
    if (__builtin_amdgcn_s_memrealtime() - t0 > DEADLINE) {
      __hip_atomic_store(tmo, seq, __ATOMIC_RELAXED, __HIP_MEMORY_SCOPE_AGENT);
      return true;
    }
    return false;
  }
};

#define STAMP_AT(off, val) do {} while (0)
#define STAMPC(k) do {} while (0)
#define STAMP(k) do {} while (0)

__device__ __forceinline__ u32 wave_max_u(u32 v) {
  v = max(v, (u32)__builtin_amdgcn_update_dpp(0, (int)v, 0x111, 0xf, 0xf, true));
  v = max(v, (u32)__builtin_amdgcn_update_dpp(0, (int)v, 0x112, 0xf, 0xf, true));
  v = max(v, (u32)__builtin_amdgcn_update_dpp(0, (int)v, 0x114, 0xf, 0xf, true));
  v = max(v, (u32)__builtin_amdgcn_update_dpp(0, (int)v, 0x118, 0xf, 0xf, true));
  v = max(v, (u32)__builtin_amdgcn_update_dpp(0, (int)v, 0x142, 0xa, 0xf, true));
  v = max(v, (u32)__builtin_amdgcn_update_dpp(0, (int)v, 0x143, 0xc, 0xf, true));
  return (u32)__builtin_amdgcn_readlane((int)v, 63);
}

__device__ __forceinline__ void dma8(rsrc_t r, u32 voff, u32 soff, u32 ldsaddr) {
  u32 soff2;
  asm volatile(
      "s_mov_b32 m0, %4\n\t"
      "s_add_u32 %0, %3, 0x1000\n\t"
      "buffer_load_dwordx4 %1, %2, %3 offen sc0 sc1 lds\n\t"
      "buffer_load_dwordx4 %1, %2, %3 offen offset:1024 sc0 sc1 lds\n\t"
      "buffer_load_dwordx4 %1, %2, %3 offen offset:2048 sc0 sc1 lds\n\t"
      "buffer_load_dwordx4 %1, %2, %3 offen offset:3072 sc0 sc1 lds\n\t"
      "s_add_u32 m0, %4, 0x1000\n\t"
      "s_nop 0\n\t"
      "buffer_load_dwordx4 %1, %2, %0 offen sc0 sc1 lds\n\t"
      "buffer_load_dwordx4 %1, %2, %0 offen offset:1024 sc0 sc1 lds\n\t"
      "buffer_load_dwordx4 %1, %2, %0 offen offset:2048 sc0 sc1 lds\n\t"
      "buffer_load_dwordx4 %1, %2, %0 offen offset:3072 sc0 sc1 lds"
      : "=&s"(soff2)
      : "v"(voff), "s"(r), "s"(soff), "s"(ldsaddr)
      : "memory", "scc");
}

__global__ void __launch_bounds__(NTHREADS, 3) fused_gemm_rs_q8(Params p) {
  extern __shared__ __attribute__((aligned(16))) char smem[];
  const int b = blockIdx.x, L = b & 7, q = b >> 3;
  const int tid = threadIdx.x, lane = tid & 63;
  const int wave = (int)rfl((u32)tid >> 6);
  const int rank = p.rank & 7;
  char* myws = p.wsx[0];
  asm volatile("" :: "s"(p.x), "s"(p.w), "s"(p.out));
  const rsrc_t rs_me = mk_rsrc(myws, WS_USED);
  u32* tmo = (u32*)(myws + OFF_TMO);
  const int peer = rank ^ L;
  const rsrc_t rs_peer = mk_rsrc(p.wsx[L], WS_USED);
  const u32 seqv = __hip_atomic_load((u32*)(myws + OFF_SEQ), __ATOMIC_RELAXED, __HIP_MEMORY_SCOPE_AGENT);
  const u32 myh = (u32)rank & 1u;
  const u32 myblk = (u32)((((rank >> 1) & 1) * 2 + ((rank >> 2) & 1)) * 32);
  const u32 l16 = (u32)lane * 16u;
  STAMP(0);

  if (q < 16) {
    const bool pusher = wave >= 8;
    const int NP = L == 1 ? NP1 : NPX;
    if (pusher && (L == 0 || (wave - 8) * 16 + q >= NP)) return;
    {
      const u32 oid = (u32)((q * 2 + L) * 8 + wave);
      const int vp = pusher ? peer : rank ^ (int)(oid & 1u);
      const u32 dr = (u32)(pusher ? peer : rank);
      const rsrc_t rs_x = mk_rsrc(p.x, 4096u * 512u * 4u);
      const rsrc_t rs_w = mk_rsrc(p.w, 512u * 4096u * 4u);
      const rsrc_t rs_dst = pusher ? rs_peer : rs_me;
      const u32 pi = (vp >> 1) & 1, pj = (vp >> 2) & 1, ph = vp & 1;
      const u32 slotbase = 4u * (u32)(pusher ? rank : vp);
      const u32 vx = (u32)(lane >> 3) * 2048u + (u32)(lane & 7) * 16u;
      const u32 vw = (u32)(lane >> 3) * 65536u + (u32)(lane & 7) * 16u;
      const u32 dx0 = (u32)(lane >> 3) * 128u + (u32)(((lane & 7) ^ (lane >> 4)) * 16);
      const u32 fl_lane = lane == 0 ? 0u : OOB;
      u32x4 ld[8];
      auto desc = [&](u32 v, u32& isw, u32& src, u32& dst, u32& flg) {
        constexpr u32 N0 = 12u + 2u * GE, NA = 36u - 2u * GE;
        u32 t, r;
        if (v < 4u * N0) {
          t = v / N0; const u32 e = v % N0;
          isw = e >= 12u;
          r = isw ? e - 12u : (((dr ^ (2u + (e >> 1))) * 2u + (dr & 1u)) * 2u + (e & 1u));
        } else if (v < 192u) {
          const u32 v2 = v - 4u * N0;
          t = v2 / NA; const u32 e = v2 % NA;
          isw = e >= 20u;
          r = isw ? 2u * GE + e - 20u
                  : (e < 12u ? (((dr ^ (2u + (e >> 1))) * 2u + ((dr & 1u) ^ 1u)) * 2u + (e & 1u)) : (dr & 6u) * 4u + e - 12u);
        } else {
          t = (v - 192u) >> 4; r = 16u + (v & 15u); isw = 1u;
        }
        const u32 slot = slotbase + t, k0 = ph * 256u + t * 64u;
        if (isw) {
          src = k0 * 16384u + (pj * 2048u + r * 64u) * 4u;
          dst = OFF_WS + slot * SLOT_BYTES + r * 8192u;
          flg = OFF_WF + (r >> 1) * 512u + slot * 16u + (r & 1u) * 8u;
        } else {
          src = ((r >> 2) * 512u + pi * 256u + (r & 3u) * 64u) * 2048u + k0 * 4u;
          dst = OFF_XS + slot * SLOT_BYTES + r * 8192u;
          flg = OFF_XF + (r >> 1) * 512u + slot * 16u + (r & 1u) * 8u;
        }
      };
      auto loads = [&](u32 isw, u32 src) {
        if (isw) {
#pragma unroll
          for (int j = 0; j < 8; ++j)
            ld[j] = __builtin_amdgcn_raw_buffer_load_b128(rs_w, vw, src + (u32)((j & 3) + 32 * (j >> 2)) * 16384u, 0);
        } else {
#pragma unroll
          for (int j = 0; j < 8; ++j)
            ld[j] = __builtin_amdgcn_raw_buffer_load_b128(rs_x, vx, src + (u32)(j >> 1) * 16384u + (u32)(j & 1) * 128u, 0);
        }
      };
      u32 seq;
      auto fdesc = [&](u32 f, u32& s1, u32& f1, u32& s2, u32& f2, u32& dst, u32& dfl) {
        const u32 reg = f & 7u;
        u32 sl = (u32)L, tile, g;
        s2 = OOB; dfl = OOB;
        bool direct = true;
        if (L == 1) {
          if (f < 48u * GE) {
            const u32 k = (f >> 3) % GE;
            sl = 2u + (f >> 3) / GE; tile = 0u; g = 2u * k + myh;
            dst = OFF_PR + (((sl ^ 1u) * 64u) + k * 8u + reg) * 4096u;
            dfl = OFF_PF + (((sl ^ 1u) * 64u) + k * 8u + reg) * 4u;
            direct = false;
          } else { const u32 k = (f - 48u * GE) >> 3; tile = k >> 4; g = k & 15u; }
        } else {
          const u32 k = f >> 3;
          if (k < 16u - 2u * GE) { tile = 0u; g = 2u * GE + k; }
          else if (k < 16u - GE) {
            const u32 kk = k - (16u - 2u * GE);
            tile = 0u; g = 2u * kk + (myh ^ 1u);
            s2 = OFF_PR + ((u32)L * 64u + kk * 8u + reg) * 4096u;
          } else { tile = 1u; g = k - (16u - GE); }
        }
        const u32 fu = tile * 128u + g * 8u + reg;
        s1 = OFF_TL + sl * MB + fu * 4096u;
        f1 = OFF_TF + sl * 1024u + fu * 4u;
        f2 = (s2 == OOB) ? f1 : OFF_PF + ((s2 - OFF_PR) >> 10);
        if (direct)
          dst = OFF_PS + (seq & 1u) * 8u * MB + myh * 4u * MB + ((myblk + (g & 1u) * 16u + (g >> 1) + tile * 8u) * 8u + reg) * 4096u;
      };
      u32 s1, f1, s2, f2, fdst, fdfl;
      const u32 two_lanes = lane < 2 ? 0u : OOB;
      auto fflags = [&]() { return ld4(rs_me, (lane == 1 ? f2 : f1) | two_lanes, 0u); };
      auto fready = [&](u32 v) { return __all(lane >= 2 || v == seq); };
      auto floads = [&]() {
#pragma unroll
        for (int e = 0; e < 4; ++e) ld[e] = ld16(rs_me, l16, s1 + (u32)e * 1024u);
        if (s2 != OOB) {
#pragma unroll
          for (int e = 0; e < 4; ++e) ld[4 + e] = ld16(rs_me, l16, s2 + (u32)e * 1024u);
        }
      };
      const u32 nin = (L == 1) ? 0u : 256u;
      const u32 ntot = (L == 1) ? 48u * GE + 256u : 512u - 8u * GE;
      auto draw = [&]() {
        return (u32)__builtin_amdgcn_raw_ptr_buffer_atomic_add_i32(1, rs_me, (int)fl_lane, (int)(OFF_TK + (u32)L * 128u), 0);
      };
      u32 v = pusher ? (u32)((wave - 8) * 16 + q) : (L < 2 ? oid >> 1 : 1000u);
      u32 vn = v + (pusher ? (u32)NP : 128u);
      u32 tkv = 0u, tkbase, pflg = OOB;
      bool have = false;
      auto emit = [&](u32 isw, u32 dst) {
        u32x4 o[4];
        if (isw) {
#pragma unroll
          for (int c = 0; c < 4; ++c)
#pragma unroll
            for (int d = 0; d < 4; ++d) o[c][d] = pk_bf16(as_f(ld[2 * d][c]), as_f(ld[2 * d + 1][c]));
#pragma unroll
          for (int c = 0; c < 4; ++c) st16(o[c], rs_dst, (u32)((lane ^ c) * 16), dst + (u32)c * 1024u);
        } else {
#pragma unroll
          for (int g = 0; g < 4; ++g) {
            o[g][0] = pk_bf16(as_f(ld[2 * g][0]), as_f(ld[2 * g][1]));
            o[g][1] = pk_bf16(as_f(ld[2 * g][2]), as_f(ld[2 * g][3]));
            o[g][2] = pk_bf16(as_f(ld[2 * g + 1][0]), as_f(ld[2 * g + 1][1]));
            o[g][3] = pk_bf16(as_f(ld[2 * g + 1][2]), as_f(ld[2 * g + 1][3]));
          }
#pragma unroll
          for (int g = 0; g < 4; ++g) st16(o[g], rs_dst, dx0 ^ (u32)((g & 1) * 64), dst + (u32)g * 1024u);
        }
        asm volatile("" ::: "memory");
      };
      if (pusher ? L != 1 : L < 2) {
        u32 isw, src, dst, flg;
        desc(v, isw, src, dst, flg);
        loads(isw, src);
        seq = next_seq(seqv);
        tkbase = (seq - 1u) * (ntot - (u32)NP) - (u32)(2 * NP);
        tkv = vn + tkbase;
        const u32x2 seq2 = {seq, seq};
#pragma unroll 1
        for (;;) {
          asm volatile("s_waitcnt vmcnt(0)" ::: "memory");
          emit(isw, dst);
          __builtin_amdgcn_raw_buffer_store_b64(seq2, rs_dst, fl_lane | pflg, 0, WT);
          loads(isw, src + (isw ? 128u : 65536u));
          vn = rfl(tkv) - tkbase;
          if (vn >= 256u) break;
          u32 nisw, nsrc, ndst, nflg;
          desc(vn, nisw, nsrc, ndst, nflg);
          asm volatile("s_waitcnt vmcnt(0)" ::: "memory");
          emit(isw, dst + 4096u);
          loads(nisw, nsrc);
          tkv = pusher ? draw() : 1000u + tkbase;
          pflg = flg;
          isw = nisw; src = nsrc; dst = ndst; flg = nflg;
        }
        v = vn;
        if (pusher) fdesc(v - 256u, s1, f1, s2, f2, fdst, fdfl);
        asm volatile("s_waitcnt vmcnt(0)" ::: "memory");
        const u32 nfl = pusher ? fflags() : 0u;
        emit(isw, dst + 4096u);
        pflg = flg;
        if (pusher) {
          have = fready(nfl);
          asm volatile("" ::: "memory");
          if (have) floads();
          tkv = draw();
        } else {
          asm volatile("s_waitcnt vmcnt(0)" ::: "memory");
          __builtin_amdgcn_raw_buffer_store_b64(seq2, rs_dst, fl_lane | pflg, 0, WT);
        }
      } else {
        seq = next_seq(seqv);
        tkbase = (seq - 1u) * (ntot - (u32)NP) - (u32)(2 * NP);
        tkv = vn + tkbase;
        if (pusher) fdesc(v, s1, f1, s2, f2, fdst, fdfl);
      }
      STAMP(1);
      if (pusher) {
        u32 pfl2 = pflg == OOB ? OOB : pflg + 4u;
        for (;;) {
          asm volatile("s_waitcnt vmcnt(0)" ::: "memory");
          __builtin_amdgcn_raw_buffer_store_b32(seq, rs_peer, (lane == 1 ? pfl2 : pflg) | two_lanes, 0, WT);
          pfl2 = OOB;
          vn = rfl(tkv) - tkbase;
          if (!have) {
            Spin sp; sp.start();
            for (;;) {
              if (fready(fflags())) break;
              if (sp.expired(tmo, seq)) break;
              __builtin_amdgcn_s_sleep(4);
            }
            asm volatile("" ::: "memory");
            floads();
          }
          const u32 cdst = fdst, cs2 = s2;
          pflg = fdfl;
          STAMP_AT(524288u + ((u32)L * 512u + v - nin) * 8u, __builtin_amdgcn_s_memrealtime());
          u32 nfl = 0u;
          if (vn < ntot) {
            fdesc(vn - nin, s1, f1, s2, f2, fdst, fdfl);
            nfl = fflags();
          }
          if (cs2 != OOB) {
#pragma unroll
            for (int e = 0; e < 4; ++e)
#pragma unroll
              for (int d = 0; d < 4; ++d)
                ld[e][d] = nz(pk_bf16(bf_lo(ld[e][d]) + bf_lo(ld[4 + e][d]), bf_hi(ld[e][d]) + bf_hi(ld[4 + e][d])));
          }
#pragma unroll
          for (int e = 0; e < 4; ++e) st16(ld[e], rs_peer, l16, cdst + (u32)e * 1024u);
          if (vn >= ntot) break;
          v = vn;
          have = fready(nfl);
          asm volatile("" ::: "memory");
          if (have) floads();
          tkv = draw();
        }
        STAMP(4);
        asm volatile("s_waitcnt vmcnt(0)" ::: "memory");
        __builtin_amdgcn_raw_buffer_store_b32(seq, rs_peer, fl_lane | pflg, 0, WT);
        STAMP(5);
        return;
      }
    }
    volatile lds_u32* lds = (volatile lds_u32*)smem;
    const u32 seq = next_seq(seqv);
    const u32 par = seq & 1u;
    const u32 cwi = (u32)((L - 1) * 16 + q);
    const u32 cw = L == 0 ? (u32)((q >> 1) * 16 + 14 + (q & 1)) : (cwi / 14u) * 16u + cwi % 14u;
    const u32 bcol = cw & 15u, ra = (cw >> 4) & 1u, tj = (cw >> 5) & 1u, ti = (cw >> 6) & 1u;
    const u32 ooff = (ti * 256u + ra * 128u + (u32)(wave >> 1) * 32u + (u32)(lane & 15)) * 4096u + tj * 2048u + bcol * 128u +
                     (u32)(wave & 1) * 64u + (u32)(lane >> 4) * 16u;
    const rsrc_t rs_out = mk_rsrc(p.out, 512u * 4096u);
    u64* am_dst = (u64*)(p.wsx[lane & 7] + OFF_AM + ((u32)rank * 128u + cw) * 8u);
    {
      u32 t0 = __builtin_amdgcn_raw_buffer_load_b32(rs_out, ooff, 0, 0);
      u32 t1 = 0u;
      if (tid < 8) t1 = (u32)__hip_atomic_load(am_dst, __ATOMIC_RELAXED, __HIP_MEMORY_SCOPE_SYSTEM);
      u32 t2 = ld4(rs_peer, tid == 64 ? sgpr(OFF_PS + par * 8u * MB + myh * 4u * MB + myblk * 32768u) : (tid == 65 ? OFF_PR : OOB), 0u);
      asm volatile("" :: "v"(t0), "v"(t1), "v"(t2));
    }
    if (L == 0) {
      const u32x4 zero4 = {0u, 0u, 0u, 0u};
      const u32 zo = sgpr(OFF_PS + (par ^ 1u) * 8u * MB + (u32)(q * 8 + wave) * 65536u);
#pragma unroll 1
      for (u32 k = 0; k < 16u; ++k) {
        __builtin_amdgcn_s_sleep(40);
#pragma unroll
        for (int e = 0; e < 4; ++e) st16(zero4, rs_me, l16, zo + k * 4096u + (u32)e * 1024u);
      }
      STAMP(12);
    }
    float y[2][16];
    {
      const u32 so = sgpr(OFF_PS + par * 8u * MB + (cw * 8u + (u32)wave) * 4096u);
      const bool pre = bcol < GE && (tj * 2u + ti) != ((u32)rank >> 1);
      bool ok0 = pre && ra == 0u, ok1 = pre && ra == 1u;
      u32x4 d0[4], d1[4];
#pragma unroll
      for (int e = 0; e < 4; ++e) { d0[e] = (u32x4){0u, 0u, 0u, 0u}; d1[e] = (u32x4){0u, 0u, 0u, 0u}; }
      Spin sp; sp.start();
      {
        const u32 pv = (lane == 1 ? (ok1 ? OOB : 4u * MB + 4092u) : (ok0 ? OOB : 4092u)) | (lane < 2 ? 0u : OOB);
        const u32 po = L == 0 ? so - PRE_COLS * 32768u : so;
        for (;;) {
          const u32 v = ld4(rs_me, pv, po);
          if (__all(pv >= OOB || v != 0u)) break;
          if (sp.expired(tmo, seq)) break;
          __builtin_amdgcn_s_sleep(S0_SLEEP);
        }
      }
      STAMP(8); STAMP_AT(((blockIdx.x * 12 + wave) * 16 + 10) * 8, (u64)sp.n);
      for (;;) {
        if (!ok0) {
#pragma unroll
          for (int e = 0; e < 4; ++e) d0[e] = ld16(rs_me, l16, so + (u32)e * 1024u);
        }
        if (!ok1) {
#pragma unroll
          for (int e = 0; e < 4; ++e) d1[e] = ld16(rs_me, l16, so + 4u * MB + (u32)e * 1024u);
        }
        ok0 = ok0 || __all(min16(d0) != 0u);
        ok1 = ok1 || __all(min16(d1) != 0u);
        if (ok0 && ok1) break;
        if (sp.expired(tmo, seq)) break;
        __builtin_amdgcn_s_sleep(1);
      }
      STAMP(4); STAMP_AT(((blockIdx.x * 12 + wave) * 16 + 9) * 8, (u64)sp.n);
      float mx = 0.f;
#pragma unroll
      for (int e = 0; e < 4; ++e)
#pragma unroll
        for (int d = 0; d < 4; ++d) {
          const float lo = bf_lo(d0[e][d]) + bf_lo(d1[e][d]);
          const float hi = bf_hi(d0[e][d]) + bf_hi(d1[e][d]);
          y[e >> 1][(e & 1) * 8 + d * 2] = lo;
          y[e >> 1][(e & 1) * 8 + d * 2 + 1] = hi;
          mx = fmaxf(mx, fmaxf(fabsf(lo), fabsf(hi)));
        }
      lds[8 + wave] = wave_max_u(as_u(mx));
      asm volatile("s_waitcnt lgkmcnt(0)\n\ts_barrier" ::: "memory");
      if (wave == 0 && lane < 8) {
        u32 m = lds[8];
#pragma unroll
        for (int k = 1; k < 8; ++k) m = max(m, lds[8 + k]);
        __hip_atomic_store(am_dst, ((u64)seq << 32) | (u64)m, __ATOMIC_RELAXED, __HIP_MEMORY_SCOPE_SYSTEM);
      }
    }
    STAMP(5);
    auto quantize = [&](u32 m) {
      const bool anybad = m == 0xffffffffu;
      const float inv = m ? 127.0f / as_f(m) : 0.f;
#pragma unroll
      for (int mm = 0; mm < 2; ++mm) {
        u32x4 o;
#pragma unroll
        for (int d = 0; d < 4; ++d) {
          u32 t[4];
#pragma unroll
          for (int k = 0; k < 4; ++k)
            t[k] = as_u(__builtin_amdgcn_fmed3f(__builtin_fmaf(y[mm][d * 4 + k], inv, 12582912.f), 12582912.f - 127.f, 12582912.f + 127.f));
          const u32 wd = __builtin_amdgcn_perm(__builtin_amdgcn_perm(t[3], t[2], 0x0c0c0400u),
                                               __builtin_amdgcn_perm(t[1], t[0], 0x0c0c0400u), 0x05040100u);
          o[d] = anybad ? 0x80808080u : wd;
        }
        __builtin_amdgcn_raw_buffer_store_b128(o, rs_out, ooff, (u32)mm * 65536u, 0);
      }
    };
    {
      Spin sp; sp.start();
      u32 done_m = 0x7fffffffu;
      bool spec = true;
#pragma unroll 1
      for (u32 round = 0;; ++round) {
        u32x4 g;
        u32 bad, cnt;
        for (;;) {
          g = ld16(rs_me, (u32)tid * 16u, OFF_AM);
          bad = ld4(rs_me, 0u, OFF_TMO);
          cnt = (u32)__popcll(__ballot(g[1] == seq && g[3] == seq));
          if (cnt == 64u || (spec && cnt >= SPEC_THR / 8)) break;
          if (sp.expired(tmo, seq)) { bad = seq; cnt = 64u; break; }
          if (cnt < 48u) __builtin_amdgcn_s_sleep(8);
        }
        const u32 wm = wave_max_u(max(g[1] == seq ? g[0] : 0u, g[3] == seq ? g[2] : 0u));
        volatile lds_u32* sl = lds + 16 + (round & 1u) * 16u;
        sl[wave] = (rfl(bad) == seq) ? 0xffffffffu : wm;
        sl[8 + wave] = cnt;
        asm volatile("s_waitcnt lgkmcnt(0)\n\ts_barrier" ::: "memory");
        u32 m = sl[0], c = sl[8];
#pragma unroll
        for (int k = 1; k < 8; ++k) { m = max(m, sl[k]); c += sl[8 + k]; }
        m = rfl(m); c = rfl(c);
        if (c == 512u) STAMP(6);
        if (m != done_m) { quantize(m); done_m = m; }
        if (c == 512u) break;
        spec = false;
      }
    }
    if (b == 0 && tid == 0) __hip_atomic_store((u32*)(myws + OFF_SEQ), seq, __ATOMIC_RELAXED, __HIP_MEMORY_SCOPE_AGENT);
    STAMP(7);
  } else {
    const int g = q - 16;
    const u32 bb = (u32)(g >> 1);
    const bool early = L >= 2 && bb < GE && (u32)(g & 1) == myh;
    if (wave >= 8) {
      __builtin_amdgcn_s_setprio(3);
      const int j = wave - 8;
      const u32 a = (u32)(2 * peer + (g & 1));
      u32 goff, foff;
      {
        const int ROT8 = (int)bb + 4 * (g & 1), ROT6 = (int)bb + 3 * (g & 1);
        const int n = lane;
        int k1, k2, r;
        if (early) {
          k1 = (n >= 12 && n < 20) ? n - 12 : -1;
          k2 = (n >= 32 && n < 40) ? n - 32 : -1;
          r = n < 12 ? n : (n < 32 ? n - 8 : n - 40);
        } else {
          k1 = n < 8 ? n : -1;
          k2 = (n >= 8 && n < 15) ? n - 8 : (n == 39 ? 7 : -1);
          r = n < 39 ? n - 15 : n - 40;
        }
        const u32 t2 = (k2 >= 0 || n >= 40) ? 1u : 0u;
        u32 slot;
        if (k1 >= 0) slot = (u32)(4 * (rank ^ ((k1 + g) & 1)) + (k1 >> 1));
        else if (k2 >= 0) { const int k = (k2 + ROT8) & 7; slot = (u32)(4 * (rank ^ (k >> 2)) + (k & 3)); }
        else slot = (u32)(4 * (rank ^ (2 + (r + ROT6) % 6)) + r / 6);
        const u32 bcol = bb + 8u * t2;
        if (j < 2) {
          goff = OFF_XS + slot * SLOT_BYTES + (a * 4u + (u32)(2 * j)) * 4096u;
          foff = OFF_XF + a * 512u + slot * 16u;
        } else {
          goff = OFF_WS + slot * SLOT_BYTES + (bcol * 4u + (u32)(2 * (j - 2))) * 4096u;
          foff = OFF_WF + bcol * 512u + slot * 16u;
        }
      }
      const u32 seq = next_seq(seqv);
      const u32 lds_base0 = rfl((u32)(size_t)(__attribute__((address_space(3))) char*)smem);
      const u32 lds_base = lds_base0 + (u32)j * 8192u;
      int cov = 0;
      int iss = 0, since = 64;
      bool pend = false;
      const u32 poll_lds = lds_base0 + (u32)LDS_POLL + (u32)j * 1024u;
      const volatile lds_u32* pl = (const volatile lds_u32*)(smem + LDS_POLL + j * 1024 + lane * 16 + (j & 1) * 8);
      auto prefix = [&](u32 f) {
        const u64 miss = __ballot((int)(f - seq) < 0);
        return miss ? (int)__builtin_ctzll(miss) : 64;
      };
      auto issue = [&](int k) {
        u32 go = (u32)__builtin_amdgcn_readlane((int)goff, k);
        asm volatile("s_nop 4" : "+s"(go));
        dma8(rs_me, l16, go, lds_base + (u32)(k & (NSTAGE - 1)) * 32768u);
      };
#pragma unroll 1
      for (int n = 0; n < 64; ++n) {
        if (iss <= n) {
          if (cov <= n) {
            Spin sp; sp.start();
            for (;;) {
              cov = prefix(ld4(rs_me, foff + (u32)(j & 1) * 8u, 0u));
              if (cov > n) break;
              if (sp.expired(tmo, seq)) { cov = 64; break; }
              __builtin_amdgcn_s_sleep(1);
            }
          }
          issue(n);
          iss = n + 1;
        }
        int k = iss - n - 1;
        if (pend && since < k) k = since;
        const int lim = (n + NSTAGE < 64) ? n + NSTAGE : 64;
        if (k >= 2) asm volatile("s_waitcnt vmcnt(16)\n\ts_barrier" ::: "memory");
        else if (k == 1) asm volatile("s_waitcnt vmcnt(8)\n\ts_barrier" ::: "memory");
        else asm volatile("s_waitcnt vmcnt(0)\n\ts_barrier" ::: "memory");
        if (pend) {
          const int c = prefix(pl[0]);
          cov = c > cov ? c : cov;
          if (cov == 64) cov = 1000;
          pend = false;
        }
        if (cov < lim + 4) {
          asm volatile("s_waitcnt lgkmcnt(0)\n\t"
                       "s_mov_b32 m0, %2\n\t"
                       "s_nop 0\n\t"
                       "buffer_load_dwordx4 %0, %1, 0 offen sc0 sc1 lds"
                       :: "v"(foff), "s"(rs_me), "s"(poll_lds) : "memory");
          pend = true;
          since = 0;
        }
        const int top = lim < cov ? lim : cov;
        while (iss < top) { issue(iss); ++iss; ++since; }
        if (!pend && iss == n + NSTAGE) {
          const int last = (cov - 8 < 64 - NSTAGE) ? cov - 8 : 64 - NSTAGE;
          u32 la = (u32)(iss & (NSTAGE - 1)) * 32768u;
          u32 go = (u32)__builtin_amdgcn_readlane((int)goff, iss & 63);
#pragma unroll 1
          while (n < last) {
            asm volatile("s_waitcnt vmcnt(16)\n\ts_barrier" : "+s"(go) :: "memory");
            dma8(rs_me, l16, go, lds_base + la);
            ++n; ++iss;
            la = (la + 32768u) & (u32)(NSTAGE * 32768 - 1);
            go = (u32)__builtin_amdgcn_readlane((int)goff, iss & 63);
          }
        }
      }
    } else {
      const int grp = wave >> 2, wr = (wave >> 1) & 1, wc = wave & 1, fi = lane & 15, kq = lane >> 4;
      u32 ax[2], aw[4];
#pragma unroll
      for (int ks = 0; ks < 2; ++ks) ax[ks] = (u32)((wr * 64 + fi) * 128 + (((kq + 4 * ks) ^ (fi >> 1)) * 16));
#pragma unroll
      for (int f = 0; f < 4; ++f)
        aw[f] = (u32)(16384 + wc * 8192 + (fi >> 3) * 4096 + (fi & 3) * 1024 + ((kq * 8 + 4 * ((fi >> 2) & 1) + (f ^ (fi & 3))) * 16));
      f32x4 acc[4][4];
#pragma unroll
      for (int m = 0; m < 4; ++m)
#pragma unroll
        for (int f = 0; f < 4; ++f) acc[m][f] = (f32x4){0.f, 0.f, 0.f, 0.f};
      bf16x8 x0[4], w0[4], x1[4], w1[4];
      auto rd = [&](int n, int ks, bf16x8 (&xf)[4], bf16x8 (&wf)[4]) {
        const char* st = smem + (n & (NSTAGE - 1)) * 32768;
#pragma unroll
        for (int m = 0; m < 4; ++m) {
          wf[m] = *(const bf16x8*)(st + aw[m] + ks * 512);
          xf[m] = *(const bf16x8*)(st + ax[ks] + m * 2048);
        }
      };
      auto mm = [&](bf16x8 (&xf)[4], bf16x8 (&wf)[4]) {
#pragma unroll
        for (int f = 0; f < 4; ++f)
#pragma unroll
          for (int m = 0; m < 4; ++m)
            acc[m][f] = __builtin_amdgcn_mfma_f32_16x16x32_bf16(wf[f], xf[m], acc[m][f], 0, 0, 0);
      };
#define SGB(m, n) __builtin_amdgcn_sched_group_barrier(m, n, 0)
#define ILV() do { SGB(0x002, 5); SGB(0x100, 2); _Pragma("unroll") for (int i_ = 0; i_ < 6; ++i_) { SGB(0x008, 1); SGB(0x100, 1); } SGB(0x008, 10); } while (0)
      const u32 bounds = early ? 0x40402000u : 0x270f0800u;
      int nb = 0;
#pragma unroll 1
      for (int seg = grp; seg < 4; seg += 2) {
        const int n0 = (int)((bounds >> (8 * seg)) & 0xffu), n1 = seg == 3 ? 64 : (int)((bounds >> (8 * seg + 8)) & 0xffu);
        if (n0 >= n1) continue;
#pragma unroll 1
        for (; nb <= n0; ++nb) asm volatile("s_barrier" ::: "memory");
        STAMP(seg >= 2 ? 2 : 1); STAMPC(12);
        rd(n0, 0, x0, w0);
#pragma unroll 1
        for (int n = n0; n < n1; ++n) {
          rd(n, 1, x1, w1);
          mm(x0, w0);
          ILV();
          __builtin_amdgcn_sched_barrier(0);
          asm volatile("s_waitcnt lgkmcnt(0)\n\ts_barrier" ::: "memory");
          rd(n + 1, 0, x0, w0);
          mm(x1, w1);
          ILV();
          __builtin_amdgcn_sched_barrier(0);
        }
        nb = n1 + 1;
        STAMPC(13);
      }
      STAMP(3);
#define JOIN0 0
#define JOIN1 2
      if (grp == 0) for (int k = 0; k < JOIN0; ++k) asm volatile("s_barrier" ::: "memory");
      const u32 seq = next_seq(seqv);
      const u32 obase = (L == 0) ? OFF_PS + (seq & 1u) * 8u * MB + myh * 4u * MB + (myblk + (u32)((g & 1) * 16 + 8 * grp) + bb) * 32768u
                                 : OFF_TL + (u32)L * MB + (u32)(grp * 16 + g) * 32768u;
#pragma unroll
      for (int m = 0; m < 4; ++m)
#pragma unroll
        for (int hf = 0; hf < 2; ++hf) {
          u32x4 o;
          o[0] = nz(pk_bf16(acc[m][2 * hf][0], acc[m][2 * hf][1]));
          o[1] = nz(pk_bf16(acc[m][2 * hf][2], acc[m][2 * hf][3]));
          o[2] = nz(pk_bf16(acc[m][2 * hf + 1][0], acc[m][2 * hf + 1][1]));
          o[3] = nz(pk_bf16(acc[m][2 * hf + 1][2], acc[m][2 * hf + 1][3]));
          st16(o, rs_me, l16, obase + (u32)(((wr * 2 + (m >> 1)) * 2 + wc) * 4096 + ((m & 1) * 2 + hf) * 1024));
        }
      if (grp == 0) for (int k = 0; k < JOIN1; ++k) asm volatile("s_barrier" ::: "memory");
      if (L != 0) {
        asm volatile("s_waitcnt vmcnt(0)" ::: "memory");
        __builtin_amdgcn_raw_buffer_store_b32(seq, rs_me, lane < 2 ? (u32)lane * 8u : OOB,
                                              OFF_TF + (u32)L * 1024u + (u32)((grp * 16 + g) * 8 + wr * 4 + wc) * 4u, WT);
      }
      STAMP(4);
    }
  }
}

}

#define HIP_CHECK(expr) do { hipError_t _e = (expr); if (_e != hipSuccess) { \
    fprintf(stderr, "HIP error %s at %s:%d\n", hipGetErrorString(_e), __FILE__, __LINE__); \
    return; } } while (0)

extern "C" void kernel_launch(int rank, int nranks, ncclComm_t comm, hipStream_t stream, void* const* d_in,
                              const int* in_sizes, int n_in, void* d_out, int out_size, const int* in_dtypes,
                              int out_dtype, void* const* d_out_all, void* const* d_ws_all, size_t ws_bytes) {
  (void)comm; (void)in_dtypes; (void)out_dtype; (void)d_out_all;
  if (nranks != 8 || (unsigned)rank >= 8u || n_in != 2 || in_sizes[0] != 4096 * 512 || in_sizes[1] != 512 * 4096 ||
      out_size != 512 * 4096 || ws_bytes < (size_t)WS_USED) {
    fprintf(stderr, "kernel_launch: unexpected problem shape\n");
    return;
  }
  Params p;
  for (int i = 0; i < 8; ++i) p.wsx[i] = (char*)d_ws_all[rank ^ i];
  p.x = (const float*)d_in[0];
  p.w = (const float*)d_in[1];
  p.out = (signed char*)d_out;
  p.rank = rank;
  HIP_CHECK(hipFuncSetAttribute((const void*)fused_gemm_rs_q8, hipFuncAttributeMaxDynamicSharedMemorySize, LDS_BYTES));
  fused_gemm_rs_q8<<<dim3(256), dim3(NTHREADS), LDS_BYTES, stream>>>(p);
  HIP_CHECK(hipGetLastError());
}
```

```cpp
#include <hip/hip_runtime.h>
#include <rccl/rccl.h>
#include <cstdio>

typedef unsigned u32;
typedef unsigned long long u64;
typedef u32 u32x4 __attribute__((ext_vector_type(4)));
typedef u32 u32x2 __attribute__((ext_vector_type(2)));
typedef float f32x4 __attribute__((ext_vector_type(4)));
typedef float f32x2 __attribute__((ext_vector_type(2)));
typedef __bf16 bf16x2 __attribute__((ext_vector_type(2)));
typedef __bf16 bf16x8 __attribute__((ext_vector_type(8)));

namespace {

#define NP_ 16
#define S0_SLEEP 2
#define AM_SLEEP 1
#define AM_SLOW 1
#define AM_THR 56
#define PRE_COLS 2u
#define SPEC_THR 384
#define GE_ 4
#define NP1_ 22
constexpr int NPX = NP_;
constexpr int NP1 = NP1_;
constexpr u32 GE = GE_;

constexpr u32 OFF_SEQ = 0;
constexpr u32 OFF_TMO = 256;
constexpr u32 OFF_TK = 1024;
constexpr u32 OFF_AM = 4096;
constexpr u32 OFF_XF = 16384;
constexpr u32 OFF_WF = 32768;
constexpr u32 OFF_TF = 49152;
constexpr u32 OFF_PF = 57344;
constexpr u32 OFF_DF = 61440;
constexpr u32 OFF_DBG = 65536;
constexpr u32 MB = 1u << 20;
constexpr u32 OFF_XS = 1 * MB;
constexpr u32 OFF_WS = 11 * MB;
constexpr u32 OFF_PS = 21 * MB;
constexpr u32 OFF_TL = 37 * MB;
constexpr u32 OFF_PR = 45 * MB;
constexpr u32 OFF_DS = 47 * MB;
constexpr u32 WS_USED = 51 * MB;
constexpr u32 SLOT_BYTES = 262144;
constexpr u32 OOB = 0x80000000u;
constexpr u64 DEADLINE = 100000000ull;
constexpr int WT = 17;
constexpr int NSTAGE = 4;
constexpr int LDS_POLL = NSTAGE * 32768;
constexpr int LDS_BYTES = LDS_POLL + 4096;
constexpr int NTHREADS = 768;
static_assert(GE <= 8 && NPX <= 32 && NP1 <= 32, "layout");

struct Params {
  char* wsx[8];
  const float* x;
  const float* w;
  signed char* out;
  int rank;
};

template <bool B> struct BoolTag { static constexpr bool value = B; };
typedef __amdgpu_buffer_rsrc_t rsrc_t;
typedef __attribute__((address_space(3))) u32 lds_u32;

__device__ __forceinline__ rsrc_t mk_rsrc(const void* p, u32 bytes) {
  return __builtin_amdgcn_make_buffer_rsrc((void*)p, (short)0, (int)bytes, 0x00020000);
}
__device__ __forceinline__ u32 pk_bf16(float lo, float hi) {
  f32x2 v = {lo, hi};
  bf16x2 b = __builtin_convertvector(v, bf16x2);
  return __builtin_bit_cast(u32, b);
}
__device__ __forceinline__ float as_f(u32 v) { return __builtin_bit_cast(float, v); }
__device__ __forceinline__ u32 as_u(float v) { return __builtin_bit_cast(u32, v); }
__device__ __forceinline__ float bf_lo(u32 v) { return as_f(v << 16); }
__device__ __forceinline__ float bf_hi(u32 v) { return as_f(v & 0xffff0000u); }
__device__ __forceinline__ u32 rfl(u32 v) { return __builtin_amdgcn_readfirstlane(v); }
__device__ __forceinline__ u32 sgpr(u32 s) { return rfl(s); }
__device__ __forceinline__ u32 next_seq(u32 v) { return rfl(v) + 1u; }
__device__ __forceinline__ u32 min16(const u32x4 (&d)[4]) {
  u32 m = min(min(d[0][0], d[0][1]), min(d[0][2], d[0][3]));
#pragma unroll
  for (int e = 1; e < 4; ++e) m = min(min(min(m, d[e][0]), d[e][1]), min(d[e][2], d[e][3]));
  return m;
}
__device__ __forceinline__ u32 nz(u32 d) { return d == 0u ? 0x8000u : d; }
__device__ __forceinline__ u32x4 ld16(rsrc_t r, u32 v, u32 s) { return __builtin_amdgcn_raw_buffer_load_b128(r, v, s, WT); }
__device__ __forceinline__ u32 ld4(rsrc_t r, u32 v, u32 s) { return __builtin_amdgcn_raw_buffer_load_b32(r, v, s, WT); }
__device__ __forceinline__ void st16(u32x4 d, rsrc_t r, u32 v, u32 s) { __builtin_amdgcn_raw_buffer_store_b128(d, r, v, s, WT); }

struct Spin {
  u64 t0; u32 n;
  __device__ __forceinline__ void start() { t0 = __builtin_amdgcn_s_memrealtime(); n = 0; }
  __device__ __forceinline__ bool expired(u32* tmo, u32 seq) {
    if ((++n & 31u) != 0u) return false;
    u32 t = rfl(__hip_atomic_load(tmo, __ATOMIC_RELAXED, __HIP_MEMORY_SCOPE_AGENT));
    if (t == seq) return true;
    if (__builtin_amdgcn_s_memrealtime() - t0 > DEADLINE) {
      __hip_atomic_store(tmo, seq, __ATOMIC_RELAXED, __HIP_MEMORY_SCOPE_AGENT);
      return true;
    }
    return false;
  }
};

#define STAMP_AT(off, val) do {} while (0)
#define STAMPC(k) do {} while (0)
#define STAMP(k) do {} while (0)

__device__ __forceinline__ u32 wave_max_u(u32 v) {
  v = max(v, (u32)__builtin_amdgcn_update_dpp(0, (int)v, 0x111, 0xf, 0xf, true));
  v = max(v, (u32)__builtin_amdgcn_update_dpp(0, (int)v, 0x112, 0xf, 0xf, true));
  v = max(v, (u32)__builtin_amdgcn_update_dpp(0, (int)v, 0x114, 0xf, 0xf, true));
  v = max(v, (u32)__builtin_amdgcn_update_dpp(0, (int)v, 0x118, 0xf, 0xf, true));
  v = max(v, (u32)__builtin_amdgcn_update_dpp(0, (int)v, 0x142, 0xa, 0xf, true));
  v = max(v, (u32)__builtin_amdgcn_update_dpp(0, (int)v, 0x143, 0xc, 0xf, true));
  return (u32)__builtin_amdgcn_readlane((int)v, 63);
}

__device__ __forceinline__ void dma8(rsrc_t r, u32 voff, u32 soff, u32 ldsaddr) {
  u32 soff2;
  asm volatile(
      "s_mov_b32 m0, %4\n\t"
      "s_add_u32 %0, %3, 0x1000\n\t"
      "buffer_load_dwordx4 %1, %2, %3 offen sc0 sc1 lds\n\t"
      "buffer_load_dwordx4 %1, %2, %3 offen offset:1024 sc0 sc1 lds\n\t"
      "buffer_load_dwordx4 %1, %2, %3 offen offset:2048 sc0 sc1 lds\n\t"
      "buffer_load_dwordx4 %1, %2, %3 offen offset:3072 sc0 sc1 lds\n\t"
      "s_add_u32 m0, %4, 0x1000\n\t"
      "s_nop 0\n\t"
      "buffer_load_dwordx4 %1, %2, %0 offen sc0 sc1 lds\n\t"
      "buffer_load_dwordx4 %1, %2, %0 offen offset:1024 sc0 sc1 lds\n\t"
      "buffer_load_dwordx4 %1, %2, %0 offen offset:2048 sc0 sc1 lds\n\t"
      "buffer_load_dwordx4 %1, %2, %0 offen offset:3072 sc0 sc1 lds"
      : "=&s"(soff2)
      : "v"(voff), "s"(r), "s"(soff), "s"(ldsaddr)
      : "memory", "scc");
}

__global__ void __launch_bounds__(NTHREADS, 3) fused_gemm_rs_q8(Params p) {
  extern __shared__ __attribute__((aligned(16))) char smem[];
  const int b = blockIdx.x, L = b & 7, q = b >> 3;
  const int tid = threadIdx.x, lane = tid & 63;
  const int wave = (int)rfl((u32)tid >> 6);
  const int rank = p.rank & 7;
  char* myws = p.wsx[0];
  asm volatile("" :: "s"(p.x), "s"(p.w), "s"(p.out));
  const rsrc_t rs_me = mk_rsrc(myws, WS_USED);
  u32* tmo = (u32*)(myws + OFF_TMO);
  const int peer = rank ^ L;
  const rsrc_t rs_peer = mk_rsrc(p.wsx[L], WS_USED);
  const u32 seqv = __hip_atomic_load((u32*)(myws + OFF_SEQ), __ATOMIC_RELAXED, __HIP_MEMORY_SCOPE_AGENT);
  const u32 myh = (u32)rank & 1u;
  const u32 myblk = (u32)((((rank >> 1) & 1) * 2 + ((rank >> 2) & 1)) * 32);
  const u32 l16 = (u32)lane * 16u;
  STAMP(0);

  if (q < 16) {
    const bool pusher = wave >= 8;
    const int NP = L == 1 ? NP1 : NPX;
    if (pusher && (L == 0 || (wave - 8) * 16 + q >= NP)) return;
    {
      const u32 oid = (u32)((q * 2 + L) * 8 + wave);
      const int vp = pusher ? peer : rank ^ (int)(oid & 1u);
      const u32 dr = (u32)(pusher ? peer : rank);
      const rsrc_t rs_x = mk_rsrc(p.x, 4096u * 512u * 4u);
      const rsrc_t rs_w = mk_rsrc(p.w, 512u * 4096u * 4u);
      const rsrc_t rs_dst = pusher ? rs_peer : rs_me;
      const u32 pi = (vp >> 1) & 1, pj = (vp >> 2) & 1, ph = vp & 1;
      const u32 slotbase = 4u * (u32)(pusher ? rank : vp);
      const u32 vx = (u32)(lane >> 3) * 2048u + (u32)(lane & 7) * 16u;
      const u32 vw = (u32)(lane >> 3) * 65536u + (u32)(lane & 7) * 16u;
      const u32 dx0 = (u32)(lane >> 3) * 128u + (u32)(((lane & 7) ^ (lane >> 4)) * 16);
      const u32 fl_lane = lane == 0 ? 0u : OOB;
      u32x4 ld[8];
      auto desc_t = [&](auto push_tag, u32 v, u32& isw, u32& src, u32& dst, u32& flg) {
        constexpr bool P = decltype(push_tag)::value;
        constexpr u32 NX = P ? 10u : 12u, N0 = NX + 2u * GE, NA = NX + 24u - 2u * GE, E0 = 4u * (N0 + NA);
        auto owner = [&](u32 k) { u32 lo = 2u + k; if (P && lo >= (u32)L) ++lo; return dr ^ lo; };
        u32 t, r, slot = slotbase, ii = pi, jj = pj;
        if (v < 4u * N0) {
          t = v / N0; const u32 e = v % N0;
          isw = e >= NX;
          r = isw ? e - NX : ((owner(e >> 1) * 2u + (dr & 1u)) * 2u + (e & 1u));
        } else if (v < E0) {
          const u32 v2 = v - 4u * N0;
          t = v2 / NA; const u32 e = v2 % NA;
          isw = e >= NX + 8u;
          r = isw ? 2u * GE + e - (NX + 8u)
                  : (e < NX ? ((owner(e >> 1) * 2u + ((dr & 1u) ^ 1u)) * 2u + (e & 1u)) : (dr & 6u) * 4u + e - NX);
        } else if (!P && v < E0 + 144u) {
          const u32 v2 = v - E0;
          t = v2 / 36u; const u32 e = v2 % 36u;
          isw = e < 32u;
          r = isw ? e : dr * 4u + e - 32u;
          slot = 32u + (oid & 1u) * 4u; ii ^= 1u; jj ^= 1u;
        } else {
          const u32 v2 = v - (P ? E0 : E0 + 144u);
          t = v2 >> 4; r = 16u + (v2 & 15u); isw = 1u;
        }
        slot += t;
        const u32 k0 = ph * 256u + t * 64u;
        if (isw) {
          src = k0 * 16384u + (jj * 2048u + r * 64u) * 4u;
          dst = OFF_WS + slot * SLOT_BYTES + r * 8192u;
          flg = OFF_WF + (r >> 1) * 1024u + slot * 16u + (r & 1u) * 8u;
        } else {
          src = ((r >> 2) * 512u + ii * 256u + (r & 3u) * 64u) * 2048u + k0 * 4u;
          dst = OFF_XS + slot * SLOT_BYTES + r * 8192u;
          flg = OFF_XF + (r >> 1) * 1024u + slot * 16u + (r & 1u) * 8u;
        }
      };
      auto desc = [&](u32 v, u32& isw, u32& src, u32& dst, u32& flg) {
        if (pusher) desc_t(BoolTag<true>(), v, isw, src, dst, flg);
        else desc_t(BoolTag<false>(), v, isw, src, dst, flg);
      };
      auto loads = [&](u32 isw, u32 src) {
        if (isw) {
#pragma unroll
          for (int j = 0; j < 8; ++j)
            ld[j] = __builtin_amdgcn_raw_buffer_load_b128(rs_w, vw, src + (u32)((j & 3) + 32 * (j >> 2)) * 16384u, 0);
        } else {
#pragma unroll
          for (int j = 0; j < 8; ++j)
            ld[j] = __builtin_amdgcn_raw_buffer_load_b128(rs_x, vx, src + (u32)(j >> 1) * 16384u + (u32)(j & 1) * 128u, 0);
        }
      };
      u32 seq;
      auto fdesc = [&](u32 f, u32& s1, u32& f1, u32& s2, u32& f2, u32& dst, u32& dfl) {
        const u32 reg = f & 7u;
        u32 sl = (u32)L, tile, g;
        s2 = OOB; dfl = OOB;
        bool direct = true;
        if (L == 1) {
          if (f < 48u * GE) {
            const u32 k = (f >> 3) % GE;
            sl = 2u + (f >> 3) / GE; tile = 0u; g = 2u * k + myh;
            dst = OFF_PR + (((sl ^ 1u) * 64u) + k * 8u + reg) * 4096u;
            dfl = OFF_PF + (((sl ^ 1u) * 64u) + k * 8u + reg) * 4u;
            direct = false;
          } else { const u32 k = (f - 48u * GE) >> 3; tile = k >> 4; g = k & 15u; }
        } else {
          const u32 k = f >> 3;
          if (k < 16u - 2u * GE) { tile = 0u; g = 2u * GE + k; }
          else if (k < 16u - GE) {
            const u32 kk = k - (16u - 2u * GE);
            tile = 0u; g = 2u * kk + (myh ^ 1u);
            s2 = OFF_PR + ((u32)L * 64u + kk * 8u + reg) * 4096u;
          } else { tile = 1u; g = k - (16u - GE); }
        }
        const u32 fu = tile * 128u + g * 8u + reg;
        s1 = OFF_TL + sl * MB + fu * 4096u;
        f1 = OFF_TF + sl * 1024u + fu * 4u;
        f2 = (s2 == OOB) ? f1 : OFF_PF + ((s2 - OFF_PR) >> 10);
        if (direct)
          dst = OFF_PS + (seq & 1u) * 8u * MB + myh * 4u * MB + ((myblk + (g & 1u) * 16u + (g >> 1) + tile * 8u) * 8u + reg) * 4096u;
      };
      u32 s1, f1, s2, f2, fdst, fdfl;
      const u32 two_lanes = lane < 2 ? 0u : OOB;
      auto fflags = [&]() { return ld4(rs_me, (lane == 1 ? f2 : f1) | two_lanes, 0u); };
      auto fready = [&](u32 v) { return __all(lane >= 2 || v == seq); };
      auto floads = [&]() {
#pragma unroll
        for (int e = 0; e < 4; ++e) ld[e] = ld16(rs_me, l16, s1 + (u32)e * 1024u);
        if (s2 != OOB) {
#pragma unroll
          for (int e = 0; e < 4; ++e) ld[4 + e] = ld16(rs_me, l16, s2 + (u32)e * 1024u);
        }
      };
      const u32 nin = pusher ? (L == 1 ? 0u : 240u) : 400u;
      const u32 ntot = (L == 1) ? 48u * GE + 256u : 496u - 8u * GE;
      auto draw = [&]() {
        return (u32)__builtin_amdgcn_raw_ptr_buffer_atomic_add_i32(1, rs_me, (int)fl_lane, (int)(OFF_TK + (u32)L * 128u), 0);
      };
      u32 v = pusher ? (u32)((wave - 8) * 16 + q) : (L < 2 ? oid >> 1 : 1000u);
      u32 vn = v + (pusher ? (u32)NP : 128u);
      u32 tkv = 0u, tkbase, pflg = OOB;
      bool have = false;
      auto emit = [&](u32 isw, u32 dst) {
        u32x4 o[4];
        if (isw) {
#pragma unroll
          for (int c = 0; c < 4; ++c)
#pragma unroll
            for (int d = 0; d < 4; ++d) o[c][d] = pk_bf16(as_f(ld[2 * d][c]), as_f(ld[2 * d + 1][c]));
#pragma unroll
          for (int c = 0; c < 4; ++c) st16(o[c], rs_dst, (u32)((lane ^ c) * 16), dst + (u32)c * 1024u);
        } else {
#pragma unroll
          for (int g = 0; g < 4; ++g) {
            o[g][0] = pk_bf16(as_f(ld[2 * g][0]), as_f(ld[2 * g][1]));
            o[g][1] = pk_bf16(as_f(ld[2 * g][2]), as_f(ld[2 * g][3]));
            o[g][2] = pk_bf16(as_f(ld[2 * g + 1][0]), as_f(ld[2 * g + 1][1]));
            o[g][3] = pk_bf16(as_f(ld[2 * g + 1][2]), as_f(ld[2 * g + 1][3]));
          }
#pragma unroll
          for (int g = 0; g < 4; ++g) st16(o[g], rs_dst, dx0 ^ (u32)((g & 1) * 64), dst + (u32)g * 1024u);
        }
        asm volatile("" ::: "memory");
      };
      if (pusher ? L != 1 : L < 2) {
        u32 isw, src, dst, flg;
        desc(v, isw, src, dst, flg);
        loads(isw, src);
        seq = next_seq(seqv);
        tkbase = (seq - 1u) * (ntot - (u32)NP) - (u32)(2 * NP);
        tkv = vn + tkbase;
        const u32x2 seq2 = {seq, seq};
#pragma unroll 1
        for (;;) {
          asm volatile("s_waitcnt vmcnt(0)" ::: "memory");
          emit(isw, dst);
          __builtin_amdgcn_raw_buffer_store_b64(seq2, rs_dst, fl_lane | pflg, 0, WT);
          loads(isw, src + (isw ? 128u : 65536u));
          vn = rfl(tkv) - tkbase;
          if (vn >= nin) break;
          u32 nisw, nsrc, ndst, nflg;
          desc(vn, nisw, nsrc, ndst, nflg);
          asm volatile("s_waitcnt vmcnt(0)" ::: "memory");
          emit(isw, dst + 4096u);
          loads(nisw, nsrc);
          tkv = pusher ? draw() : vn + 128u + tkbase;
          pflg = flg;
          isw = nisw; src = nsrc; dst = ndst; flg = nflg;
        }
        v = vn;
        if (pusher) fdesc(v - 240u, s1, f1, s2, f2, fdst, fdfl);
        asm volatile("s_waitcnt vmcnt(0)" ::: "memory");
        const u32 nfl = pusher ? fflags() : 0u;
        emit(isw, dst + 4096u);
        pflg = flg;
        if (pusher) {
          have = fready(nfl);
          asm volatile("" ::: "memory");
          if (have) floads();
          tkv = draw();
        } else {
          asm volatile("s_waitcnt vmcnt(0)" ::: "memory");
          __builtin_amdgcn_raw_buffer_store_b64(seq2, rs_dst, fl_lane | pflg, 0, WT);
        }
      } else {
        seq = next_seq(seqv);
        tkbase = (seq - 1u) * (ntot - (u32)NP) - (u32)(2 * NP);
        tkv = vn + tkbase;
        if (pusher) fdesc(v, s1, f1, s2, f2, fdst, fdfl);
      }
      STAMP(1);
      if (pusher) {
        u32 pfl2 = pflg == OOB ? OOB : pflg + 4u;
        for (;;) {
          asm volatile("s_waitcnt vmcnt(0)" ::: "memory");
          __builtin_amdgcn_raw_buffer_store_b32(seq, rs_peer, (lane == 1 ? pfl2 : pflg) | two_lanes, 0, WT);
          pfl2 = OOB;
          vn = rfl(tkv) - tkbase;
          if (!have) {
            Spin sp; sp.start();
            for (;;) {
              if (fready(fflags())) break;
              if (sp.expired(tmo, seq)) break;
              __builtin_amdgcn_s_sleep(4);
            }
            asm volatile("" ::: "memory");
            floads();
          }
          const u32 cdst = sgpr(fdst), cs2 = s2;
          pflg = fdfl;
          STAMP_AT(524288u + ((u32)L * 512u + v - nin) * 8u, __builtin_amdgcn_s_memrealtime());
          u32 nfl = 0u;
          if (vn < ntot) {
            fdesc(vn - nin, s1, f1, s2, f2, fdst, fdfl);
            nfl = fflags();
          }
          if (cs2 != OOB) {
#pragma unroll
            for (int e = 0; e < 4; ++e)
#pragma unroll
              for (int d = 0; d < 4; ++d)
                ld[e][d] = nz(pk_bf16(bf_lo(ld[e][d]) + bf_lo(ld[4 + e][d]), bf_hi(ld[e][d]) + bf_hi(ld[4 + e][d])));
          }
#pragma unroll
          for (int e = 0; e < 4; ++e) st16(ld[e], rs_peer, l16, cdst + (u32)e * 1024u);
          if (vn >= ntot) break;
          v = vn;
          have = fready(nfl);
          asm volatile("" ::: "memory");
          if (have) floads();
          tkv = draw();
        }
        STAMP(4);
        asm volatile("s_waitcnt vmcnt(0)" ::: "memory");
        __builtin_amdgcn_raw_buffer_store_b32(seq, rs_peer, fl_lane | pflg, 0, WT);
        STAMP(5);
        return;
      }
    }
    volatile lds_u32* lds = (volatile lds_u32*)smem;
    const u32 seq = next_seq(seqv);
    const u32 par = seq & 1u;
    const u32 cwi = (u32)((L - 1) * 16 + q);
    const u32 cw = L == 0 ? (u32)((q >> 1) * 16 + 14 + (q & 1)) : (cwi / 14u) * 16u + cwi % 14u;
    const u32 bcol = cw & 15u, ra = (cw >> 4) & 1u, tj = (cw >> 5) & 1u, ti = (cw >> 6) & 1u;
    const u32 ooff = (ti * 256u + ra * 128u + (u32)(wave >> 1) * 32u + (u32)(lane & 15)) * 4096u + tj * 2048u + bcol * 128u +
                     (u32)(wave & 1) * 64u + (u32)(lane >> 4) * 16u;
    const rsrc_t rs_out = mk_rsrc(p.out, 512u * 4096u);
    u64* am_dst = (u64*)(p.wsx[lane & 7] + OFF_AM + ((u32)rank * 128u + cw) * 8u);
    {
      u32 t0 = __builtin_amdgcn_raw_buffer_load_b32(rs_out, ooff, 0, 0);
      u32 t1 = 0u;
      if (tid < 8) t1 = (u32)__hip_atomic_load(am_dst, __ATOMIC_RELAXED, __HIP_MEMORY_SCOPE_SYSTEM);
      u32 t2 = ld4(rs_peer, tid == 64 ? sgpr(OFF_PS + par * 8u * MB + myh * 4u * MB + myblk * 32768u) : (tid == 65 ? OFF_PR : OOB), 0u);
      asm volatile("" :: "v"(t0), "v"(t1), "v"(t2));
    }
    if (L == 0) {
      const u32x4 zero4 = {0u, 0u, 0u, 0u};
      const u32 zo = sgpr(OFF_PS + (par ^ 1u) * 8u * MB + (u32)(q * 8 + wave) * 65536u);
#pragma unroll 1
      for (u32 k = 0; k < 16u; ++k) {
        __builtin_amdgcn_s_sleep(40);
#pragma unroll
        for (int e = 0; e < 4; ++e) st16(zero4, rs_me, l16, zo + k * 4096u + (u32)e * 1024u);
      }
      STAMP(12);
    }
    float y[2][16];
    {
      const u32 so = sgpr(OFF_PS + par * 8u * MB + (cw * 8u + (u32)wave) * 4096u);
      const bool pre = bcol < GE && (tj * 2u + ti) != ((u32)rank >> 1);
      bool ok0 = pre && ra == 0u, ok1 = pre && ra == 1u;
      u32x4 d0[4], d1[4], d2[4];
#pragma unroll
      for (int e = 0; e < 4; ++e) { d0[e] = (u32x4){0u, 0u, 0u, 0u}; d1[e] = d0[e]; d2[e] = d0[e]; }
      Spin sp; sp.start();
      if ((cw & 96u) != myblk) {
        for (;;) {
          if (rfl(ld4(rs_me, 0u, OFF_DF + (cw * 8u + (u32)wave) * 4u)) == seq) break;
          if (sp.expired(tmo, seq)) break;
          __builtin_amdgcn_s_sleep(32);
        }
        asm volatile("" ::: "memory");
#pragma unroll
        for (int e = 0; e < 4; ++e) d2[e] = ld16(rs_me, l16, OFF_DS + (cw * 8u + (u32)wave) * 4096u + (u32)e * 1024u);
      }
      {
        const u32 pv = (lane == 1 ? (ok1 ? OOB : 4u * MB + 4092u) : (ok0 ? OOB : 4092u)) | (lane < 2 ? 0u : OOB);
        const u32 po = L == 0 ? so - PRE_COLS * 32768u : so;
        for (;;) {
          const u32 v = ld4(rs_me, pv, po);
          if (__all(pv >= OOB || v != 0u)) break;
          if (sp.expired(tmo, seq)) break;
          __builtin_amdgcn_s_sleep(S0_SLEEP);
        }
      }
      STAMP(8); STAMP_AT(((blockIdx.x * 12 + wave) * 16 + 10) * 8, (u64)sp.n);
      for (;;) {
        if (!ok0) {
#pragma unroll
          for (int e = 0; e < 4; ++e) d0[e] = ld16(rs_me, l16, so + (u32)e * 1024u);
        }
        if (!ok1) {
#pragma unroll
          for (int e = 0; e < 4; ++e) d1[e] = ld16(rs_me, l16, so + 4u * MB + (u32)e * 1024u);
        }
        ok0 = ok0 || __all(min16(d0) != 0u);
        ok1 = ok1 || __all(min16(d1) != 0u);
        if (ok0 && ok1) break;
        if (sp.expired(tmo, seq)) break;
        __builtin_amdgcn_s_sleep(1);
      }
      STAMP(4); STAMP_AT(((blockIdx.x * 12 + wave) * 16 + 9) * 8, (u64)sp.n);
      float mx = 0.f;
#pragma unroll
      for (int e = 0; e < 4; ++e)
#pragma unroll
        for (int d = 0; d < 4; ++d) {
          const float lo = (bf_lo(d0[e][d]) + bf_lo(d1[e][d])) + bf_lo(d2[e][d]);
          const float hi = (bf_hi(d0[e][d]) + bf_hi(d1[e][d])) + bf_hi(d2[e][d]);
          y[e >> 1][(e & 1) * 8 + d * 2] = lo;
          y[e >> 1][(e & 1) * 8 + d * 2 + 1] = hi;
          mx = fmaxf(mx, fmaxf(fabsf(lo), fabsf(hi)));
        }
      lds[8 + wave] = wave_max_u(as_u(mx));
      asm volatile("s_waitcnt lgkmcnt(0)\n\ts_barrier" ::: "memory");
      if (wave == 0 && lane < 8) {
        u32 m = lds[8];
#pragma unroll
        for (int k = 1; k < 8; ++k) m = max(m, lds[8 + k]);
        __hip_atomic_store(am_dst, ((u64)seq << 32) | (u64)m, __ATOMIC_RELAXED, __HIP_MEMORY_SCOPE_SYSTEM);
      }
    }
    STAMP(5);
    auto quantize = [&](u32 m) {
      const bool anybad = m == 0xffffffffu;
      const float inv = m ? 127.0f / as_f(m) : 0.f;
#pragma unroll
      for (int mm = 0; mm < 2; ++mm) {
        u32x4 o;
#pragma unroll
        for (int d = 0; d < 4; ++d) {
          u32 t[4];
#pragma unroll
          for (int k = 0; k < 4; ++k)
            t[k] = as_u(__builtin_amdgcn_fmed3f(__builtin_fmaf(y[mm][d * 4 + k], inv, 12582912.f), 12582912.f - 127.f, 12582912.f + 127.f));
          const u32 wd = __builtin_amdgcn_perm(__builtin_amdgcn_perm(t[3], t[2], 0x0c0c0400u),
                                               __builtin_amdgcn_perm(t[1], t[0], 0x0c0c0400u), 0x05040100u);
          o[d] = anybad ? 0x80808080u : wd;
        }
        __builtin_amdgcn_raw_buffer_store_b128(o, rs_out, ooff, (u32)mm * 65536u, 0);
      }
    };
    {
      Spin sp; sp.start();
      u32 done_m = 0x7fffffffu;
      bool spec = true;
#pragma unroll 1
      for (u32 round = 0;; ++round) {
        u32x4 g;
        u32 bad, cnt;
        for (;;) {
          g = ld16(rs_me, (u32)tid * 16u, OFF_AM);
          bad = ld4(rs_me, 0u, OFF_TMO);
          cnt = (u32)__popcll(__ballot(g[1] == seq && g[3] == seq));
          if (cnt == 64u || (spec && cnt >= SPEC_THR / 8)) break;
          if (sp.expired(tmo, seq)) { bad = seq; cnt = 64u; break; }
          if (cnt < 48u) __builtin_amdgcn_s_sleep(8);
        }
        const u32 wm = wave_max_u(max(g[1] == seq ? g[0] : 0u, g[3] == seq ? g[2] : 0u));
        volatile lds_u32* sl = lds + 16 + (round & 1u) * 16u;
        sl[wave] = (rfl(bad) == seq) ? 0xffffffffu : wm;
        sl[8 + wave] = cnt;
        asm volatile("s_waitcnt lgkmcnt(0)\n\ts_barrier" ::: "memory");
        u32 m = sl[0], c = sl[8];
#pragma unroll
        for (int k = 1; k < 8; ++k) { m = max(m, sl[k]); c += sl[8 + k]; }
        m = rfl(m); c = rfl(c);
        if (c == 512u) STAMP(6);
        if (m != done_m) { quantize(m); done_m = m; }
        if (c == 512u) break;
        spec = false;
      }
    }
    if (b == 0 && tid == 0) __hip_atomic_store((u32*)(myws + OFF_SEQ), seq, __ATOMIC_RELAXED, __HIP_MEMORY_SCOPE_AGENT);
    STAMP(7);
  } else {
    const int g = q - 16;
    const u32 bb = (u32)(g >> 1);
    const bool early = L >= 2 && bb < GE && (u32)(g & 1) == myh;
    const u32 dquad = (u32)((L - 2) >> 1);
    if (wave >= 8) {
      __builtin_amdgcn_s_setprio(3);
      const int j = wave - 8;
      const u32 a = (u32)(2 * peer + (g & 1));
      u32 goff, foff;
      {
        const int ROT8 = (int)bb + 4 * (g & 1), ROT6 = (int)bb + 3 * (g & 1);
        const int n = lane;
        int k1, k2, kd, r;
        if (L < 2) {
          k1 = n < 8 ? n : -1; kd = -1;
          k2 = (n >= 8 && n < 15) ? n - 8 : (n == 39 ? 7 : -1);
          r = n < 39 ? n - 15 : n - 40;
        } else if (early) {
          k1 = (n >= 10 && n < 18) ? n - 10 : -1;
          kd = (n >= 28 && n < 36) ? n - 28 : -1;
          k2 = (n >= 36 && n < 44) ? n - 36 : -1;
          r = n < 10 ? n : (n < 28 ? n - 8 : n - 44);
        } else {
          k1 = n < 8 ? n : -1;
          kd = (n >= 8 && n < 16) ? n - 8 : -1;
          k2 = (n >= 16 && n < 23) ? n - 16 : (n == 43 ? 7 : -1);
          r = n < 43 ? n - 23 : n - 44;
        }
        r = r < 0 ? 0 : r;
        u32 xslot, wslot, ab = a, bcol = bb + ((k2 >= 0 || n >= 44 || (L < 2 && n >= 40)) ? 8u : 0u);
        if (k1 >= 0) xslot = wslot = (u32)(4 * (rank ^ ((k1 + g) & 1)) + (k1 >> 1));
        else if (k2 >= 0) { const int k = (k2 + ROT8) & 7; xslot = wslot = (u32)(4 * (rank ^ (k >> 2)) + (k & 3)); }
        else if (kd >= 0) {
          const int k = (kd + ROT8) & 7;
          const u32 own = (u32)(4 * (rank ^ (k >> 2)) + (k & 3));
          xslot = dquad != 0u ? 32u + (u32)k : own;
          wslot = dquad != 1u ? 32u + (u32)k : own;
          ab = (u32)(2 * rank + (g & 1)); bcol = (u32)(L & 1) * 8u + bb;
        } else if (L < 2) xslot = wslot = (u32)(4 * (rank ^ (2 + (r + ROT6) % 6)) + r / 6);
        else {
          int ls = 2 + (r + ROT6) % 5;
          if (ls >= L) ++ls;
          xslot = wslot = (u32)(4 * (rank ^ ls) + r / 5);
        }
        if (j < 2) {
          goff = OFF_XS + xslot * SLOT_BYTES + (ab * 4u + (u32)(2 * j)) * 4096u;
          foff = OFF_XF + ab * 1024u + xslot * 16u;
        } else {
          goff = OFF_WS + wslot * SLOT_BYTES + (bcol * 4u + (u32)(2 * (j - 2))) * 4096u;
          foff = OFF_WF + bcol * 1024u + wslot * 16u;
        }
      }
      const u32 seq = next_seq(seqv);
      const u32 lds_base0 = rfl((u32)(size_t)(__attribute__((address_space(3))) char*)smem);
      const u32 lds_base = lds_base0 + (u32)j * 8192u;
      int cov = 0;
      int iss = 0, since = 64;
      bool pend = false;
      const u32 poll_lds = lds_base0 + (u32)LDS_POLL + (u32)j * 1024u;
      const volatile lds_u32* pl = (const volatile lds_u32*)(smem + LDS_POLL + j * 1024 + lane * 16 + (j & 1) * 8);
      auto prefix = [&](u32 f) {
        const u64 miss = __ballot((int)(f - seq) < 0);
        return miss ? (int)__builtin_ctzll(miss) : 64;
      };
      auto issue = [&](int k) {
        u32 go = (u32)__builtin_amdgcn_readlane((int)goff, k);
        asm volatile("s_nop 4" : "+s"(go));
        dma8(rs_me, l16, go, lds_base + (u32)(k & (NSTAGE - 1)) * 32768u);
      };
#pragma unroll 1
      for (int n = 0; n < 64; ++n) {
        if (iss <= n) {
          if (cov <= n) {
            Spin sp; sp.start();
            for (;;) {
              cov = prefix(ld4(rs_me, foff + (u32)(j & 1) * 8u, 0u));
              if (cov > n) break;
              if (sp.expired(tmo, seq)) { cov = 64; break; }
              __builtin_amdgcn_s_sleep(1);
            }
          }
          issue(n);
          iss = n + 1;
        }
        int k = iss - n - 1;
        if (pend && since < k) k = since;
        const int lim = (n + NSTAGE < 64) ? n + NSTAGE : 64;
        if (k >= 2) asm volatile("s_waitcnt vmcnt(16)\n\ts_barrier" ::: "memory");
        else if (k == 1) asm volatile("s_waitcnt vmcnt(8)\n\ts_barrier" ::: "memory");
        else asm volatile("s_waitcnt vmcnt(0)\n\ts_barrier" ::: "memory");
        if (pend) {
          const int c = prefix(pl[0]);
          cov = c > cov ? c : cov;
          if (cov == 64) cov = 1000;
          pend = false;
        }
        if (cov < lim + 4) {
          asm volatile("s_waitcnt lgkmcnt(0)\n\t"
                       "s_mov_b32 m0, %2\n\t"
                       "s_nop 0\n\t"
                       "buffer_load_dwordx4 %0, %1, 0 offen sc0 sc1 lds"
                       :: "v"(foff), "s"(rs_me), "s"(poll_lds) : "memory");
          pend = true;
          since = 0;
        }
        const int top = lim < cov ? lim : cov;
        while (iss < top) { issue(iss); ++iss; ++since; }
        if (!pend && iss == n + NSTAGE) {
          const int last = (cov - 8 < 64 - NSTAGE) ? cov - 8 : 64 - NSTAGE;
          u32 la = (u32)(iss & (NSTAGE - 1)) * 32768u;
          u32 go = (u32)__builtin_amdgcn_readlane((int)goff, iss & 63);
#pragma unroll 1
          while (n < last) {
            asm volatile("s_waitcnt vmcnt(16)\n\ts_barrier" : "+s"(go) :: "memory");
            dma8(rs_me, l16, go, lds_base + la);
            ++n; ++iss;
            la = (la + 32768u) & (u32)(NSTAGE * 32768 - 1);
            go = (u32)__builtin_amdgcn_readlane((int)goff, iss & 63);
          }
        }
      }
    } else {
      const int grp = wave >> 2, wr = (wave >> 1) & 1, wc = wave & 1, fi = lane & 15, kq = lane >> 4;
      u32 ax[2], aw[4];
#pragma unroll
      for (int ks = 0; ks < 2; ++ks) ax[ks] = (u32)((wr * 64 + fi) * 128 + (((kq + 4 * ks) ^ (fi >> 1)) * 16));
#pragma unroll
      for (int f = 0; f < 4; ++f)
        aw[f] = (u32)(16384 + wc * 8192 + (fi >> 3) * 4096 + (fi & 3) * 1024 + ((kq * 8 + 4 * ((fi >> 2) & 1) + (f ^ (fi & 3))) * 16));
      f32x4 acc[4][4];
      bf16x8 x0[4], w0[4], x1[4], w1[4];
      auto rd = [&](int n, int ks, bf16x8 (&xf)[4], bf16x8 (&wf)[4]) {
        const char* st = smem + (n & (NSTAGE - 1)) * 32768;
#pragma unroll
        for (int m = 0; m < 4; ++m) {
          wf[m] = *(const bf16x8*)(st + aw[m] + ks * 512);
          xf[m] = *(const bf16x8*)(st + ax[ks] + m * 2048);
        }
      };
      auto mm = [&](bf16x8 (&xf)[4], bf16x8 (&wf)[4]) {
#pragma unroll
        for (int f = 0; f < 4; ++f)
#pragma unroll
          for (int m = 0; m < 4; ++m)
            acc[m][f] = __builtin_amdgcn_mfma_f32_16x16x32_bf16(wf[f], xf[m], acc[m][f], 0, 0, 0);
      };
#define SGB(m, n) __builtin_amdgcn_sched_group_barrier(m, n, 0)
#define ILV() do { SGB(0x002, 5); SGB(0x100, 2); _Pragma("unroll") for (int i_ = 0; i_ < 6; ++i_) { SGB(0x008, 1); SGB(0x100, 1); } SGB(0x008, 10); } while (0)
      const u64 segs = grp == 0 ? (L < 2 ? 0x270f08000000ull : (early ? 0x00001c000000ull : 0x2b1708000000ull))
                                : (L < 2 ? 0x40270f080000ull : (early ? 0x00004024241cull : 0x402b17101008ull));
      const u32 seq = next_seq(seqv);
      int nb = 0;
#pragma unroll 1
      for (int job = 0; job < 2; ++job) {
        if (job == 0 && (u32)(segs & 0xffffull) == 0u) continue;
#pragma unroll
        for (int m = 0; m < 4; ++m)
#pragma unroll
          for (int f = 0; f < 4; ++f) acc[m][f] = (f32x4){0.f, 0.f, 0.f, 0.f};
#pragma unroll 1
        for (int seg = job; seg < 1 + 2 * job; ++seg) {
          const int n0 = (int)((segs >> (16 * seg)) & 0xffu), n1 = (int)((segs >> (16 * seg + 8)) & 0xffu);
          if (n0 >= n1) continue;
#pragma unroll 1
          for (; nb <= n0; ++nb) asm volatile("s_barrier" ::: "memory");
          STAMP(seg >= 2 ? 2 : 1); STAMPC(12);
          rd(n0, 0, x0, w0);
#pragma unroll 1
          for (int n = n0; n < n1; ++n) {
            rd(n, 1, x1, w1);
            mm(x0, w0);
            ILV();
            __builtin_amdgcn_sched_barrier(0);
            asm volatile("s_waitcnt lgkmcnt(0)\n\ts_barrier" ::: "memory");
            rd(n + 1, 0, x0, w0);
            mm(x1, w1);
            ILV();
            __builtin_amdgcn_sched_barrier(0);
          }
          nb = n1 + 1;
          STAMPC(13);
        }
        if (job == 1) STAMP(3);
        const u32 dtile = (myblk ^ (dquad != 0u ? 64u : 0u) ^ (dquad != 1u ? 32u : 0u)) + (u32)((g & 1) * 16 + (L & 1) * 8) + bb;
        const u32 obase = sgpr(job == 0 ? OFF_DS + dtile * 32768u
                        : (L == 0) ? OFF_PS + (seq & 1u) * 8u * MB + myh * 4u * MB + (myblk + (u32)((g & 1) * 16 + 8 * grp) + bb) * 32768u
                                   : OFF_TL + (u32)L * MB + (u32)(grp * 16 + g) * 32768u);
#pragma unroll
        for (int m = 0; m < 4; ++m)
#pragma unroll
          for (int hf = 0; hf < 2; ++hf) {
            u32x4 o;
            o[0] = nz(pk_bf16(acc[m][2 * hf][0], acc[m][2 * hf][1]));
            o[1] = nz(pk_bf16(acc[m][2 * hf][2], acc[m][2 * hf][3]));
            o[2] = nz(pk_bf16(acc[m][2 * hf + 1][0], acc[m][2 * hf + 1][1]));
            o[3] = nz(pk_bf16(acc[m][2 * hf + 1][2], acc[m][2 * hf + 1][3]));
            st16(o, rs_me, l16, obase + (u32)(((wr * 2 + (m >> 1)) * 2 + wc) * 4096 + ((m & 1) * 2 + hf) * 1024));
          }
        if (grp == 0) asm volatile("s_barrier\n\ts_barrier" ::: "memory");
        if (L != 0) {
          asm volatile("s_waitcnt vmcnt(0)" ::: "memory");
          __builtin_amdgcn_raw_buffer_store_b32(
              seq, rs_me, lane < 2 ? (u32)lane * 8u : OOB,
              (job == 0 ? OFF_DF + dtile * 32u : OFF_TF + (u32)L * 1024u + (u32)(grp * 16 + g) * 32u) + (u32)(wr * 4 + wc) * 4u, WT);
        }
      }
      STAMP(4);
    }
  }
}

}

#define HIP_CHECK(expr) do { hipError_t _e = (expr); if (_e != hipSuccess) { \
    fprintf(stderr, "HIP error %s at %s:%d\n", hipGetErrorString(_e), __FILE__, __LINE__); \
    return; } } while (0)

extern "C" void kernel_launch(int rank, int nranks, ncclComm_t comm, hipStream_t stream, void* const* d_in,
                              const int* in_sizes, int n_in, void* d_out, int out_size, const int* in_dtypes,
                              int out_dtype, void* const* d_out_all, void* const* d_ws_all, size_t ws_bytes) {
  (void)comm; (void)in_dtypes; (void)out_dtype; (void)d_out_all;
  if (nranks != 8 || (unsigned)rank >= 8u || n_in != 2 || in_sizes[0] != 4096 * 512 || in_sizes[1] != 512 * 4096 ||
      out_size != 512 * 4096 || ws_bytes < (size_t)WS_USED) {
    fprintf(stderr, "kernel_launch: unexpected problem shape\n");
    return;
  }
  Params p;
  for (int i = 0; i < 8; ++i) p.wsx[i] = (char*)d_ws_all[rank ^ i];
  p.x = (const float*)d_in[0];
  p.w = (const float*)d_in[1];
  p.out = (signed char*)d_out;
  p.rank = rank;
  HIP_CHECK(hipFuncSetAttribute((const void*)fused_gemm_rs_q8, hipFuncAttributeMaxDynamicSharedMemorySize, LDS_BYTES));
  fused_gemm_rs_q8<<<dim3(256), dim3(NTHREADS), LDS_BYTES, stream>>>(p);
  HIP_CHECK(hipGetLastError());
}
```

```cpp
#include <hip/hip_runtime.h>
#include <rccl/rccl.h>
#include <cstdio>

typedef unsigned u32;
typedef unsigned long long u64;
typedef u32 u32x4 __attribute__((ext_vector_type(4)));
typedef u32 u32x2 __attribute__((ext_vector_type(2)));
typedef float f32x4 __attribute__((ext_vector_type(4)));
typedef float f32x2 __attribute__((ext_vector_type(2)));
typedef __bf16 bf16x2 __attribute__((ext_vector_type(2)));
typedef __bf16 bf16x8 __attribute__((ext_vector_type(8)));

namespace {

#define NP_ 16
#define S0_SLEEP 2
#define AM_SLEEP 1
#define AM_SLOW 1
#define AM_THR 56
#define PRE_COLS 2u
#define SPEC_THR 384
#define GE_ 3
#define NP1_ 20
constexpr int NPX = NP_;
constexpr int NP1 = NP1_;
constexpr u32 GE = GE_;

constexpr u32 OFF_SEQ = 0;
constexpr u32 OFF_TMO = 256;
constexpr u32 OFF_TK = 1024;
constexpr u32 OFF_AM = 4096;
constexpr u32 OFF_XF = 16384;
constexpr u32 OFF_WF = 32768;
constexpr u32 OFF_TF = 49152;
constexpr u32 OFF_PF = 57344;
constexpr u32 OFF_DF = 61440;
constexpr u32 OFF_DBG = 65536;
constexpr u32 MB = 1u << 20;
constexpr u32 OFF_XS = 1 * MB;
constexpr u32 OFF_WS = 11 * MB;
constexpr u32 OFF_PS = 21 * MB;
constexpr u32 OFF_TL = 37 * MB;
constexpr u32 OFF_PR = 45 * MB;
constexpr u32 OFF_DS = 47 * MB;
constexpr u32 WS_USED = 51 * MB;
constexpr u32 SLOT_BYTES = 262144;
constexpr u32 OOB = 0x80000000u;
constexpr u64 DEADLINE = 100000000ull;
constexpr int WT = 17;
constexpr int NSTAGE = 4;
constexpr int LDS_POLL = NSTAGE * 32768;
constexpr int LDS_BYTES = LDS_POLL + 4096;
constexpr int NTHREADS = 768;
static_assert(GE <= 8 && NPX <= 32 && NP1 <= 32, "layout");

struct Params {
  char* wsx[8];
  const float* x;
  const float* w;
  signed char* out;
  int rank;
};

template <bool B> struct BoolTag { static constexpr bool value = B; };
typedef __amdgpu_buffer_rsrc_t rsrc_t;
typedef __attribute__((address_space(3))) u32 lds_u32;

__device__ __forceinline__ rsrc_t mk_rsrc(const void* p, u32 bytes) {
  return __builtin_amdgcn_make_buffer_rsrc((void*)p, (short)0, (int)bytes, 0x00020000);
}
__device__ __forceinline__ u32 pk_bf16(float lo, float hi) {
  f32x2 v = {lo, hi};
  bf16x2 b = __builtin_convertvector(v, bf16x2);
  return __builtin_bit_cast(u32, b);
}
__device__ __forceinline__ float as_f(u32 v) { return __builtin_bit_cast(float, v); }
__device__ __forceinline__ u32 as_u(float v) { return __builtin_bit_cast(u32, v); }
__device__ __forceinline__ float bf_lo(u32 v) { return as_f(v << 16); }
__device__ __forceinline__ float bf_hi(u32 v) { return as_f(v & 0xffff0000u); }
__device__ __forceinline__ u32 rfl(u32 v) { return __builtin_amdgcn_readfirstlane(v); }
__device__ __forceinline__ u32 sgpr(u32 s) { return rfl(s); }
__device__ __forceinline__ u32 next_seq(u32 v) { return rfl(v) + 1u; }
__device__ __forceinline__ void wait_vm0() {
  asm volatile("" ::: "memory");
  __builtin_amdgcn_s_waitcnt(0x0F70);
  asm volatile("" ::: "memory");
}
__device__ __forceinline__ u32 min16(const u32x4 (&d)[4]) {
  u32 m = min(min(d[0][0], d[0][1]), min(d[0][2], d[0][3]));
#pragma unroll
  for (int e = 1; e < 4; ++e) m = min(min(min(m, d[e][0]), d[e][1]), min(d[e][2], d[e][3]));
  return m;
}
__device__ __forceinline__ u32 nz(u32 d) { return d == 0u ? 0x8000u : d; }
__device__ __forceinline__ u32x4 ld16(rsrc_t r, u32 v, u32 s) { return __builtin_amdgcn_raw_buffer_load_b128(r, v, s, WT); }
__device__ __forceinline__ u32 ld4(rsrc_t r, u32 v, u32 s) { return __builtin_amdgcn_raw_buffer_load_b32(r, v, s, WT); }
__device__ __forceinline__ void st16(u32x4 d, rsrc_t r, u32 v, u32 s) { __builtin_amdgcn_raw_buffer_store_b128(d, r, v, s, WT); }

struct Spin {
  u64 t0; u32 n;
  __device__ __forceinline__ void start() { t0 = __builtin_amdgcn_s_memrealtime(); n = 0; }
  __device__ __forceinline__ bool expired(u32* tmo, u32 seq) {
    if ((++n & 31u) != 0u) return false;
    u32 t = rfl(__hip_atomic_load(tmo, __ATOMIC_RELAXED, __HIP_MEMORY_SCOPE_AGENT));
    if (t == seq) return true;
    if (__builtin_amdgcn_s_memrealtime() - t0 > DEADLINE) {
      __hip_atomic_store(tmo, seq, __ATOMIC_RELAXED, __HIP_MEMORY_SCOPE_AGENT);
      return true;
    }
    return false;
  }
};

#define STAMP_AT(off, val) do {} while (0)
#define STAMPC(k) do {} while (0)
#define STAMP(k) do {} while (0)

__device__ __forceinline__ u32 wave_max_u(u32 v) {
  v = max(v, (u32)__builtin_amdgcn_update_dpp(0, (int)v, 0x111, 0xf, 0xf, true));
  v = max(v, (u32)__builtin_amdgcn_update_dpp(0, (int)v, 0x112, 0xf, 0xf, true));
  v = max(v, (u32)__builtin_amdgcn_update_dpp(0, (int)v, 0x114, 0xf, 0xf, true));
  v = max(v, (u32)__builtin_amdgcn_update_dpp(0, (int)v, 0x118, 0xf, 0xf, true));
  v = max(v, (u32)__builtin_amdgcn_update_dpp(0, (int)v, 0x142, 0xa, 0xf, true));
  v = max(v, (u32)__builtin_amdgcn_update_dpp(0, (int)v, 0x143, 0xc, 0xf, true));
  return (u32)__builtin_amdgcn_readlane((int)v, 63);
}

__device__ __forceinline__ void dma8(rsrc_t r, u32 voff, u32 soff, u32 ldsaddr) {
  u32 soff2;
  asm volatile(
      "s_mov_b32 m0, %4\n\t"
      "s_add_u32 %0, %3, 0x1000\n\t"
      "buffer_load_dwordx4 %1, %2, %3 offen sc0 sc1 lds\n\t"
      "buffer_load_dwordx4 %1, %2, %3 offen offset:1024 sc0 sc1 lds\n\t"
      "buffer_load_dwordx4 %1, %2, %3 offen offset:2048 sc0 sc1 lds\n\t"
      "buffer_load_dwordx4 %1, %2, %3 offen offset:3072 sc0 sc1 lds\n\t"
      "s_add_u32 m0, %4, 0x1000\n\t"
      "s_nop 0\n\t"
      "buffer_load_dwordx4 %1, %2, %0 offen sc0 sc1 lds\n\t"
      "buffer_load_dwordx4 %1, %2, %0 offen offset:1024 sc0 sc1 lds\n\t"
      "buffer_load_dwordx4 %1, %2, %0 offen offset:2048 sc0 sc1 lds\n\t"
      "buffer_load_dwordx4 %1, %2, %0 offen offset:3072 sc0 sc1 lds"
      : "=&s"(soff2)
      : "v"(voff), "s"(r), "s"(soff), "s"(ldsaddr)
      : "memory", "scc");
}

__global__ void __launch_bounds__(NTHREADS, 3) fused_gemm_rs_q8(Params p) {
  extern __shared__ __attribute__((aligned(16))) char smem[];
  const int b = blockIdx.x, L = b & 7, q = b >> 3;
  const int tid = threadIdx.x, lane = tid & 63;
  const int wave = (int)rfl((u32)tid >> 6);
  const int rank = p.rank & 7;
  char* myws = p.wsx[0];
  asm volatile("" :: "s"(p.x), "s"(p.w), "s"(p.out));
  const rsrc_t rs_me = mk_rsrc(myws, WS_USED);
  u32* tmo = (u32*)(myws + OFF_TMO);
  const int peer = rank ^ L;
  const rsrc_t rs_peer = mk_rsrc(p.wsx[L], WS_USED);
  const u32 seqv = __hip_atomic_load((u32*)(myws + OFF_SEQ), __ATOMIC_RELAXED, __HIP_MEMORY_SCOPE_AGENT);
  const u32 myh = (u32)rank & 1u;
  const u32 myblk = (u32)((((rank >> 1) & 1) * 2 + ((rank >> 2) & 1)) * 32);
  const u32 l16 = (u32)lane * 16u;
  STAMP(0);

  if (q < 16) {
    const bool pusher = wave >= 8;
    const int NP = L == 1 ? NP1 : NPX;
    if (pusher && (L == 0 || (wave - 8) * 16 + q >= NP)) return;
    {
      const u32 oid = (u32)((q * 2 + L) * 8 + wave);
      const int vp = pusher ? peer : rank ^ (int)(oid & 1u);
      const u32 dr = (u32)(pusher ? peer : rank);
      const rsrc_t rs_x = mk_rsrc(p.x, 4096u * 512u * 4u);
      const rsrc_t rs_w = mk_rsrc(p.w, 512u * 4096u * 4u);
      const rsrc_t rs_dst = pusher ? rs_peer : rs_me;
      const u32 pi = (vp >> 1) & 1, pj = (vp >> 2) & 1, ph = vp & 1;
      const u32 slotbase = 4u * (u32)(pusher ? rank : vp);
      const u32 vx = (u32)(lane >> 3) * 2048u + (u32)(lane & 7) * 16u;
      const u32 vw = (u32)(lane >> 3) * 65536u + (u32)(lane & 7) * 16u;
      const u32 dx0 = (u32)(lane >> 3) * 128u + (u32)(((lane & 7) ^ (lane >> 4)) * 16);
      const u32 fl_lane = lane == 0 ? 0u : OOB;
      u32x4 ld[8];
      auto desc_t = [&](auto push_tag, u32 v, u32& isw, u32& src, u32& dst, u32& flg) {
        constexpr bool P = decltype(push_tag)::value;
        constexpr u32 NX = P ? 10u : 12u, N0 = NX + 2u * GE, NA = NX + 24u - 2u * GE, E0 = 4u * (N0 + NA);
        auto owner = [&](u32 k) { u32 lo = 2u + k; if (P && lo >= (u32)L) ++lo; return dr ^ lo; };
        u32 t, r, slot = slotbase, ii = pi, jj = pj;
        if (v < 4u * N0) {
          t = v / N0; const u32 e = v % N0;
          isw = e >= NX;
          r = isw ? e - NX : ((owner(e >> 1) * 2u + (dr & 1u)) * 2u + (e & 1u));
        } else if (v < E0) {
          const u32 v2 = v - 4u * N0;
          t = v2 / NA; const u32 e = v2 % NA;
          isw = e >= NX + 8u;
          r = isw ? 2u * GE + e - (NX + 8u)
                  : (e < NX ? ((owner(e >> 1) * 2u + ((dr & 1u) ^ 1u)) * 2u + (e & 1u)) : (dr & 6u) * 4u + e - NX);
        } else if (P || v < E0 + 64u) {
          const u32 v2 = v - E0;
          t = v2 >> 4; r = 16u + (v2 & 15u); isw = 1u;
        } else {
          const u32 v2 = v - (E0 + 64u);
          t = v2 / 36u; const u32 e = v2 % 36u;
          isw = e < 32u;
          r = isw ? e : dr * 4u + e - 32u;
          slot = 32u + (oid & 1u) * 4u; ii ^= 1u; jj ^= 1u;
        }
        slot += t;
        const u32 k0 = ph * 256u + t * 64u;
        if (isw) {
          src = k0 * 16384u + (jj * 2048u + r * 64u) * 4u;
          dst = OFF_WS + slot * SLOT_BYTES + r * 8192u;
          flg = OFF_WF + (r >> 1) * 1024u + slot * 16u + (r & 1u) * 8u;
        } else {
          src = ((r >> 2) * 512u + ii * 256u + (r & 3u) * 64u) * 2048u + k0 * 4u;
          dst = OFF_XS + slot * SLOT_BYTES + r * 8192u;
          flg = OFF_XF + (r >> 1) * 1024u + slot * 16u + (r & 1u) * 8u;
        }
      };
      auto loads = [&](u32x4 (&ld)[8], u32 isw, u32 src) {
        const rsrc_t rs = isw ? rs_w : rs_x;
        const u32 vo = isw ? vw : vx, s1 = isw ? 16384u : 128u, s2 = isw ? 32768u : 16384u, s4 = isw ? 524288u : 32768u;
#pragma unroll
        for (int j = 0; j < 8; ++j)
          ld[j] = __builtin_amdgcn_raw_buffer_load_b128(rs, vo, src + ((j & 1) ? s1 : 0u) + ((j & 2) ? s2 : 0u) + ((j & 4) ? s4 : 0u), 0);
      };
      u32 seq;
      auto fdesc = [&](u32 f, u32& s1, u32& f1, u32& s2, u32& f2, u32& dst, u32& dfl) {
        const u32 reg = f & 7u;
        u32 sl = (u32)L, tile, g;
        s2 = OOB; dfl = OOB;
        bool direct = true;
        if (L == 1) {
          if (f < 48u * GE) {
            const u32 k = (f >> 3) % GE;
            sl = 2u + (f >> 3) / GE; tile = 0u; g = 2u * k + myh;
            dst = OFF_PR + (((sl ^ 1u) * 64u) + k * 8u + reg) * 4096u;
            dfl = OFF_PF + (((sl ^ 1u) * 64u) + k * 8u + reg) * 4u;
            direct = false;
          } else { const u32 k = (f - 48u * GE) >> 3; tile = k >> 4; g = k & 15u; }
        } else {
          const u32 k = f >> 3;
          if (k < 16u - 2u * GE) { tile = 0u; g = 2u * GE + k; }
          else if (k < 16u - GE) {
            const u32 kk = k - (16u - 2u * GE);
            tile = 0u; g = 2u * kk + (myh ^ 1u);
            s2 = OFF_PR + ((u32)L * 64u + kk * 8u + reg) * 4096u;
          } else { tile = 1u; g = k - (16u - GE); }
        }
        const u32 fu = tile * 128u + g * 8u + reg;
        s1 = OFF_TL + sl * MB + fu * 4096u;
        f1 = OFF_TF + sl * 1024u + fu * 4u;
        f2 = (s2 == OOB) ? f1 : OFF_PF + ((s2 - OFF_PR) >> 10);
        if (direct)
          dst = OFF_PS + (seq & 1u) * 8u * MB + myh * 4u * MB + ((myblk + (g & 1u) * 16u + (g >> 1) + tile * 8u) * 8u + reg) * 4096u;
      };
      u32 s1, f1, s2, f2, fdst, fdfl;
      const u32 two_lanes = lane < 2 ? 0u : OOB;
      auto fflags = [&]() { return ld4(rs_me, (lane == 1 ? f2 : f1) | two_lanes, 0u); };
      auto fready = [&](u32 v) { return __all(lane >= 2 || v == seq); };
      auto floads = [&]() {
#pragma unroll
        for (int e = 0; e < 4; ++e) ld[e] = ld16(rs_me, l16, s1 + (u32)e * 1024u);
        if (s2 != OOB) {
#pragma unroll
          for (int e = 0; e < 4; ++e) ld[4 + e] = ld16(rs_me, l16, s2 + (u32)e * 1024u);
        }
      };
      const u32 nin = L == 1 ? 0u : 240u;
      const u32 ntot = (L == 1) ? 48u * GE + 256u : 496u - 8u * GE;
      auto draw = [&]() {
        return (u32)__builtin_amdgcn_raw_ptr_buffer_atomic_add_i32(1, rs_me, (int)fl_lane, (int)(OFF_TK + (u32)L * 128u), 0);
      };
      u32 v = pusher ? (u32)((wave - 8) * 16 + q) : oid >> 1;
      u32 vn = v + (u32)NP;
      u32 tkv = 0u, tkbase, pflg = OOB;
      bool have = false;
      auto emit = [&](u32x4 (&ld)[8], u32 isw, u32 dst) {
        u32x4 o[4];
        if (isw) {
#pragma unroll
          for (int c = 0; c < 4; ++c)
#pragma unroll
            for (int d = 0; d < 4; ++d) o[c][d] = pk_bf16(as_f(ld[2 * d][c]), as_f(ld[2 * d + 1][c]));
        } else {
#pragma unroll
          for (int g = 0; g < 4; ++g) {
            o[g][0] = pk_bf16(as_f(ld[2 * g][0]), as_f(ld[2 * g][1]));
            o[g][1] = pk_bf16(as_f(ld[2 * g][2]), as_f(ld[2 * g][3]));
            o[g][2] = pk_bf16(as_f(ld[2 * g + 1][0]), as_f(ld[2 * g + 1][1]));
            o[g][3] = pk_bf16(as_f(ld[2 * g + 1][2]), as_f(ld[2 * g + 1][3]));
          }
        }
#pragma unroll
        for (int c = 0; c < 4; ++c)
          st16(o[c], rs_dst, isw ? (u32)((lane ^ c) * 16) : dx0 ^ (u32)((c & 1) * 64), dst + (u32)c * 1024u);
        asm volatile("" ::: "memory");
      };
      if (!pusher) {
        seq = next_seq(seqv);
        if (L < 2) {
          const u32x2 seq2 = {seq, seq};
          u32x4 lb[8];
#pragma unroll 1
          for (; v < 400u; v += 128u) {
            u32 isw, src, dst, flg;
            desc_t(BoolTag<false>(), v, isw, src, dst, flg);
            loads(ld, isw, src);
            asm volatile("" ::: "memory");
            loads(lb, isw, src + (isw ? 128u : 65536u));
            asm volatile("s_waitcnt vmcnt(8)" ::: "memory");
            __builtin_amdgcn_raw_buffer_store_b64(seq2, rs_me, fl_lane | pflg, 0, WT);
            emit(ld, isw, dst);
            emit(lb, isw, dst + 4096u);
            pflg = flg;
          }
          wait_vm0();
          __builtin_amdgcn_raw_buffer_store_b64(seq2, rs_me, fl_lane | pflg, 0, WT);
        }
      } else if (L != 1) {
        u32 isw, src, dst, flg;
        desc_t(BoolTag<true>(), v, isw, src, dst, flg);
        loads(ld, isw, src);
        seq = next_seq(seqv);
        tkbase = (seq - 1u) * (ntot - (u32)NP) - (u32)(2 * NP);
        tkv = vn + tkbase;
        const u32x2 seq2 = {seq, seq};
#pragma unroll 1
        for (;;) {
          wait_vm0();
          emit(ld, isw, dst);
          __builtin_amdgcn_raw_buffer_store_b64(seq2, rs_dst, fl_lane | pflg, 0, WT);
          loads(ld, isw, src + (isw ? 128u : 65536u));
          vn = rfl(tkv) - tkbase;
          if (vn >= nin) break;
          u32 nisw, nsrc, ndst, nflg;
          desc_t(BoolTag<true>(), vn, nisw, nsrc, ndst, nflg);
          wait_vm0();
          emit(ld, isw, dst + 4096u);
          loads(ld, nisw, nsrc);
          tkv = draw();
          pflg = flg;
          isw = nisw; src = nsrc; dst = ndst; flg = nflg;
        }
        v = vn;
        fdesc(v - 240u, s1, f1, s2, f2, fdst, fdfl);
        wait_vm0();
        const u32 nfl = fflags();
        emit(ld, isw, dst + 4096u);
        pflg = flg;
        have = fready(nfl);
        asm volatile("" ::: "memory");
        if (have) floads();
        tkv = draw();
      } else {
        seq = next_seq(seqv);
        tkbase = (seq - 1u) * (ntot - (u32)NP) - (u32)(2 * NP);
        tkv = vn + tkbase;
        fdesc(v, s1, f1, s2, f2, fdst, fdfl);
      }
      STAMP(1);
      if (pusher) {
        u32 pfl2 = pflg == OOB ? OOB : pflg + 4u;
#pragma unroll 1
        for (;;) {
          wait_vm0();
          vn = rfl(tkv) - tkbase;
          if (!have) {
            __builtin_amdgcn_raw_buffer_store_b32(seq, rs_peer, (lane == 1 ? pfl2 : pflg) | two_lanes, 0, WT);
            pflg = OOB; pfl2 = OOB;
            Spin sp; sp.start();
            for (;;) {
              if (fready(fflags())) break;
              if (sp.expired(tmo, seq)) break;
              __builtin_amdgcn_s_sleep(4);
            }
            asm volatile("" ::: "memory");
            floads();
            wait_vm0();
          }
          const u32 cdst = sgpr(fdst), cs2 = s2, cfl = fdfl;
          STAMP_AT(524288u + ((u32)L * 512u + v - nin) * 8u, __builtin_amdgcn_s_memrealtime());
          u32 nfl = 0u;
          if (vn < ntot) {
            fdesc(vn - nin, s1, f1, s2, f2, fdst, fdfl);
            nfl = fflags();
          }
          if (cs2 != OOB) {
#pragma unroll
            for (int e = 0; e < 4; ++e)
#pragma unroll
              for (int d = 0; d < 4; ++d)
                ld[e][d] = nz(pk_bf16(bf_lo(ld[e][d]) + bf_lo(ld[4 + e][d]), bf_hi(ld[e][d]) + bf_hi(ld[4 + e][d])));
          }
#pragma unroll
          for (int e = 0; e < 4; ++e) st16(ld[e], rs_peer, l16, cdst + (u32)e * 1024u);
          __builtin_amdgcn_raw_buffer_store_b32(seq, rs_peer, (lane == 1 ? pfl2 : pflg) | two_lanes, 0, WT);
          pflg = cfl; pfl2 = OOB;
          if (vn >= ntot) break;
          v = vn;
          have = fready(nfl);
          asm volatile("" ::: "memory");
          if (have) floads();
          tkv = draw();
        }
        STAMP(4);
        wait_vm0();
        __builtin_amdgcn_raw_buffer_store_b32(seq, rs_peer, fl_lane | pflg, 0, WT);
        STAMP(5);
        return;
      }
    }
    volatile lds_u32* lds = (volatile lds_u32*)smem;
    const u32 seq = next_seq(seqv);
    const u32 par = seq & 1u;
    const u32 cwi = (u32)((L - 1) * 16 + q);
    const u32 cw = L == 0 ? (u32)((q >> 1) * 16 + 14 + (q & 1)) : (cwi / 14u) * 16u + cwi % 14u;
    const u32 bcol = cw & 15u, ra = (cw >> 4) & 1u, tj = (cw >> 5) & 1u, ti = (cw >> 6) & 1u;
    const u32 ooff = (ti * 256u + ra * 128u + (u32)(wave >> 1) * 32u + (u32)(lane & 15)) * 4096u + tj * 2048u + bcol * 128u +
                     (u32)(wave & 1) * 64u + (u32)(lane >> 4) * 16u;
    const rsrc_t rs_out = mk_rsrc(p.out, 512u * 4096u);
    u64* am_dst = (u64*)(p.wsx[lane & 7] + OFF_AM + ((u32)rank * 128u + cw) * 8u);
    {
      u32 t0 = __builtin_amdgcn_raw_buffer_load_b32(rs_out, ooff, 0, 0);
      u32 t1 = 0u;
      if (tid < 8) t1 = (u32)__hip_atomic_load(am_dst, __ATOMIC_RELAXED, __HIP_MEMORY_SCOPE_SYSTEM);
      u32 t2 = ld4(rs_peer, tid == 64 ? sgpr(OFF_PS + par * 8u * MB + myh * 4u * MB + myblk * 32768u) : (tid == 65 ? OFF_PR : OOB), 0u);
      asm volatile("" :: "v"(t0), "v"(t1), "v"(t2));
    }
    if (L == 0) {
      const u32x4 zero4 = {0u, 0u, 0u, 0u};
      const u32 zo = sgpr(OFF_PS + (par ^ 1u) * 8u * MB + (u32)(q * 8 + wave) * 65536u);
#pragma unroll 1
      for (u32 k = 0; k < 16u; ++k) {
        __builtin_amdgcn_s_sleep(40);
#pragma unroll
        for (int e = 0; e < 4; ++e) st16(zero4, rs_me, l16, zo + k * 4096u + (u32)e * 1024u);
      }
      STAMP(12);
    }
    float y[2][16];
    {
      const u32 so = sgpr(OFF_PS + par * 8u * MB + (cw * 8u + (u32)wave) * 4096u);
      const bool pre = bcol < GE && (tj * 2u + ti) != ((u32)rank >> 1);
      bool ok0 = pre && ra == 0u, ok1 = pre && ra == 1u;
      u32x4 d0[4], d1[4], d2[4];
#pragma unroll
      for (int e = 0; e < 4; ++e) { d0[e] = (u32x4){0u, 0u, 0u, 0u}; d1[e] = d0[e]; d2[e] = d0[e]; }
      Spin sp; sp.start();
      auto diag = [&]() {
        if ((cw & 96u) == myblk) return;
        for (;;) {
          if (rfl(ld4(rs_me, 0u, OFF_DF + (cw * 8u + (u32)wave) * 4u)) == seq) break;
          if (sp.expired(tmo, seq)) break;
          __builtin_amdgcn_s_sleep(16);
        }
        asm volatile("" ::: "memory");
#pragma unroll
        for (int e = 0; e < 4; ++e) d2[e] = ld16(rs_me, l16, OFF_DS + (cw * 8u + (u32)wave) * 4096u + (u32)e * 1024u);
      };
      diag();
      {
        const u32 pv = (lane == 1 ? (ok1 ? OOB : 4u * MB + 4092u) : (ok0 ? OOB : 4092u)) | (lane < 2 ? 0u : OOB);
        const u32 po = L == 0 ? so - PRE_COLS * 32768u : so;
        for (;;) {
          const u32 v = ld4(rs_me, pv, po);
          if (__all(pv >= OOB || v != 0u)) break;
          if (sp.expired(tmo, seq)) break;
          __builtin_amdgcn_s_sleep(S0_SLEEP);
        }
      }
      STAMP(8); STAMP_AT(((blockIdx.x * 12 + wave) * 16 + 10) * 8, (u64)sp.n);
      for (;;) {
        if (!ok0) {
#pragma unroll
          for (int e = 0; e < 4; ++e) d0[e] = ld16(rs_me, l16, so + (u32)e * 1024u);
        }
        if (!ok1) {
#pragma unroll
          for (int e = 0; e < 4; ++e) d1[e] = ld16(rs_me, l16, so + 4u * MB + (u32)e * 1024u);
        }
        ok0 = ok0 || __all(min16(d0) != 0u);
        ok1 = ok1 || __all(min16(d1) != 0u);
        if (ok0 && ok1) break;
        if (sp.expired(tmo, seq)) break;
        __builtin_amdgcn_s_sleep(1);
      }
      STAMP(4); STAMP_AT(((blockIdx.x * 12 + wave) * 16 + 9) * 8, (u64)sp.n);
      float mx = 0.f;
#pragma unroll
      for (int e = 0; e < 4; ++e)
#pragma unroll
        for (int d = 0; d < 4; ++d) {
          const float lo = (bf_lo(d0[e][d]) + bf_lo(d1[e][d])) + bf_lo(d2[e][d]);
          const float hi = (bf_hi(d0[e][d]) + bf_hi(d1[e][d])) + bf_hi(d2[e][d]);
          y[e >> 1][(e & 1) * 8 + d * 2] = lo;
          y[e >> 1][(e & 1) * 8 + d * 2 + 1] = hi;
          mx = fmaxf(mx, fmaxf(fabsf(lo), fabsf(hi)));
        }
      lds[8 + wave] = wave_max_u(as_u(mx));
      asm volatile("s_waitcnt lgkmcnt(0)\n\ts_barrier" ::: "memory");
      if (wave == 0 && lane < 8) {
        u32 m = lds[8];
#pragma unroll
        for (int k = 1; k < 8; ++k) m = max(m, lds[8 + k]);
        __hip_atomic_store(am_dst, ((u64)seq << 32) | (u64)m, __ATOMIC_RELAXED, __HIP_MEMORY_SCOPE_SYSTEM);
      }
    }
    STAMP(5);
    auto quantize = [&](u32 m) {
      const bool anybad = m == 0xffffffffu;
      const float inv = m ? 127.0f / as_f(m) : 0.f;
#pragma unroll
      for (int mm = 0; mm < 2; ++mm) {
        u32x4 o;
#pragma unroll
        for (int d = 0; d < 4; ++d) {
          u32 t[4];
#pragma unroll
          for (int k = 0; k < 4; ++k)
            t[k] = as_u(__builtin_amdgcn_fmed3f(__builtin_fmaf(y[mm][d * 4 + k], inv, 12582912.f), 12582912.f - 127.f, 12582912.f + 127.f));
          const u32 wd = __builtin_amdgcn_perm(__builtin_amdgcn_perm(t[3], t[2], 0x0c0c0400u),
                                               __builtin_amdgcn_perm(t[1], t[0], 0x0c0c0400u), 0x05040100u);
          o[d] = anybad ? 0x80808080u : wd;
        }
        __builtin_amdgcn_raw_buffer_store_b128(o, rs_out, ooff, (u32)mm * 65536u, 0);
      }
    };
    {
      Spin sp; sp.start();
      u32 done_m = 0x7fffffffu;
      bool spec = true;
#pragma unroll 1
      for (u32 round = 0;; ++round) {
        u32x4 g;
        u32 bad, cnt;
        for (;;) {
          g = ld16(rs_me, (u32)tid * 16u, OFF_AM);
          bad = ld4(rs_me, 0u, OFF_TMO);
          cnt = (u32)__popcll(__ballot(g[1] == seq && g[3] == seq));
          if (cnt == 64u || (spec && cnt >= SPEC_THR / 8)) break;
          if (sp.expired(tmo, seq)) { bad = seq; cnt = 64u; break; }
          if (cnt < 48u) __builtin_amdgcn_s_sleep(8);
        }
        const u32 wm = wave_max_u(max(g[1] == seq ? g[0] : 0u, g[3] == seq ? g[2] : 0u));
        volatile lds_u32* sl = lds + 16 + (round & 1u) * 16u;
        sl[wave] = (rfl(bad) == seq) ? 0xffffffffu : wm;
        sl[8 + wave] = cnt;
        asm volatile("s_waitcnt lgkmcnt(0)\n\ts_barrier" ::: "memory");
        u32 m = sl[0], c = sl[8];
#pragma unroll
        for (int k = 1; k < 8; ++k) { m = max(m, sl[k]); c += sl[8 + k]; }
        m = rfl(m); c = rfl(c);
        if (c == 512u) STAMP(6);
        if (m != done_m) { quantize(m); done_m = m; }
        if (c == 512u) break;
        spec = false;
      }
    }
    if (b == 0 && tid == 0) __hip_atomic_store((u32*)(myws + OFF_SEQ), seq, __ATOMIC_RELAXED, __HIP_MEMORY_SCOPE_AGENT);
    STAMP(7);
  } else {
    const int g = q - 16;
    const u32 bb = (u32)(g >> 1);
    const bool early = L >= 2 && bb < GE && (u32)(g & 1) == myh;
    const u32 dquad = (u32)((L - 2) >> 1);
    if (wave >= 8) {
      __builtin_amdgcn_s_setprio(3);
      const int j = wave - 8;
      const u32 a = (u32)(2 * peer + (g & 1));
      u32 goff, foff;
      {
        const int ROT8 = (int)bb + 4 * (g & 1), ROT6 = (int)bb + 3 * (g & 1);
        const int n = lane;
        int k1, k2, kd, r;
        if (L < 2) {
          k1 = n < 8 ? n : -1; kd = -1;
          k2 = (n >= 8 && n < 15) ? n - 8 : (n == 39 ? 7 : -1);
          r = n < 39 ? n - 15 : n - 40;
        } else {
          kd = n >= 56 ? n - 56 : -1;
          if (early) {
            k1 = (n >= 10 && n < 18) ? n - 10 : -1;
            k2 = (n >= 28 && n < 36) ? n - 28 : -1;
            r = n < 10 ? n : (n < 28 ? n - 8 : n - 36);
          } else {
            k1 = n < 8 ? n : -1;
            k2 = (n >= 8 && n < 15) ? n - 8 : (n == 35 ? 7 : -1);
            r = n < 35 ? n - 15 : n - 36;
          }
        }
        r = r < 0 ? 0 : r;
        u32 xslot, wslot, ab = a, bcol = bb + ((k2 >= 0 || n >= (L < 2 ? 40 : 36)) ? 8u : 0u);
        if (k1 >= 0) xslot = wslot = (u32)(4 * (rank ^ ((k1 + g) & 1)) + (k1 >> 1));
        else if (k2 >= 0) { const int k = (k2 + ROT8) & 7; xslot = wslot = (u32)(4 * (rank ^ (k >> 2)) + (k & 3)); }
        else if (kd >= 0) {
          const int k = (kd + ROT8) & 7;
          const u32 own = (u32)(4 * (rank ^ (k >> 2)) + (k & 3));
          xslot = dquad != 0u ? 32u + (u32)k : own;
          wslot = dquad != 1u ? 32u + (u32)k : own;
          ab = (u32)(2 * rank + (g & 1)); bcol = (u32)(L & 1) * 8u + bb;
        } else if (L < 2) xslot = wslot = (u32)(4 * (rank ^ (2 + (r + ROT6) % 6)) + r / 6);
        else {
          int ls = 2 + (r + ROT6) % 5;
          if (ls >= L) ++ls;
          xslot = wslot = (u32)(4 * (rank ^ ls) + r / 5);
        }
        if (j < 2) {
          goff = OFF_XS + xslot * SLOT_BYTES + (ab * 4u + (u32)(2 * j)) * 4096u;
          foff = OFF_XF + ab * 1024u + xslot * 16u;
        } else {
          goff = OFF_WS + wslot * SLOT_BYTES + (bcol * 4u + (u32)(2 * (j - 2))) * 4096u;
          foff = OFF_WF + bcol * 1024u + wslot * 16u;
        }
      }
      const u32 seq = next_seq(seqv);
      const u32 lds_base0 = rfl((u32)(size_t)(__attribute__((address_space(3))) char*)smem);
      const u32 lds_base = lds_base0 + (u32)j * 8192u;
      int cov = 0;
      int iss = 0, since = 64;
      bool pend = false;
      const u32 poll_lds = lds_base0 + (u32)LDS_POLL + (u32)j * 1024u;
      const volatile lds_u32* pl = (const volatile lds_u32*)(smem + LDS_POLL + j * 1024 + lane * 16 + (j & 1) * 8);
      auto prefix = [&](u32 f) {
        const u64 miss = __ballot((int)(f - seq) < 0);
        return miss ? (int)__builtin_ctzll(miss) : 64;
      };
      auto issue = [&](int k) {
        u32 go = (u32)__builtin_amdgcn_readlane((int)goff, k);
        asm volatile("s_nop 4" : "+s"(go));
        dma8(rs_me, l16, go, lds_base + (u32)(k & (NSTAGE - 1)) * 32768u);
      };
#pragma unroll 1
      for (int n = 0; n < 64; ++n) {
        if (iss <= n) {
          if (cov <= n) {
            Spin sp; sp.start();
            for (;;) {
              cov = prefix(ld4(rs_me, foff + (u32)(j & 1) * 8u, 0u));
              if (cov > n) break;
              if (sp.expired(tmo, seq)) { cov = 64; break; }
              __builtin_amdgcn_s_sleep(1);
            }
          }
          issue(n);
          iss = n + 1;
        }
        int k = iss - n - 1;
        if (pend && since < k) k = since;
        const int lim = (n + NSTAGE < 64) ? n + NSTAGE : 64;
        if (k >= 2) asm volatile("s_waitcnt vmcnt(16)\n\ts_barrier" ::: "memory");
        else if (k == 1) asm volatile("s_waitcnt vmcnt(8)\n\ts_barrier" ::: "memory");
        else asm volatile("s_waitcnt vmcnt(0)\n\ts_barrier" ::: "memory");
        if (pend) {
          const int c = prefix(pl[0]);
          cov = c > cov ? c : cov;
          if (cov == 64) cov = 1000;
          pend = false;
        }
        if (cov < lim + 4) {
          asm volatile("s_waitcnt lgkmcnt(0)\n\t"
                       "s_mov_b32 m0, %2\n\t"
                       "s_nop 0\n\t"
                       "buffer_load_dwordx4 %0, %1, 0 offen sc0 sc1 lds"
                       :: "v"(foff), "s"(rs_me), "s"(poll_lds) : "memory");
          pend = true;
          since = 0;
        }
        const int top = lim < cov ? lim : cov;
        while (iss < top) { issue(iss); ++iss; ++since; }
        if (!pend && iss == n + NSTAGE) {
          const int last = (cov - 8 < 64 - NSTAGE) ? cov - 8 : 64 - NSTAGE;
          u32 la = (u32)(iss & (NSTAGE - 1)) * 32768u;
          u32 go = (u32)__builtin_amdgcn_readlane((int)goff, iss & 63);
#pragma unroll 1
          while (n < last) {
            asm volatile("s_waitcnt vmcnt(16)\n\ts_barrier" : "+s"(go) :: "memory");
            dma8(rs_me, l16, go, lds_base + la);
            ++n; ++iss;
            la = (la + 32768u) & (u32)(NSTAGE * 32768 - 1);
            go = (u32)__builtin_amdgcn_readlane((int)goff, iss & 63);
          }
        }
      }
    } else {
      const int grp = wave >> 2, wr = (wave >> 1) & 1, wc = wave & 1, fi = lane & 15, kq = lane >> 4;
      u32 ax[2], aw[4];
#pragma unroll
      for (int ks = 0; ks < 2; ++ks) ax[ks] = (u32)((wr * 64 + fi) * 128 + (((kq + 4 * ks) ^ (fi >> 1)) * 16));
#pragma unroll
      for (int f = 0; f < 4; ++f)
        aw[f] = (u32)(16384 + wc * 8192 + (fi >> 3) * 4096 + (fi & 3) * 1024 + ((kq * 8 + 4 * ((fi >> 2) & 1) + (f ^ (fi & 3))) * 16));
      f32x4 acc[4][4];
      bf16x8 x0[4], w0[4], x1[4], w1[4];
      auto rd = [&](int n, int ks, bf16x8 (&xf)[4], bf16x8 (&wf)[4]) {
        const char* st = smem + (n & (NSTAGE - 1)) * 32768;
#pragma unroll
        for (int m = 0; m < 4; ++m) {
          wf[m] = *(const bf16x8*)(st + aw[m] + ks * 512);
          xf[m] = *(const bf16x8*)(st + ax[ks] + m * 2048);
        }
      };
      auto mm = [&](bf16x8 (&xf)[4], bf16x8 (&wf)[4]) {
#pragma unroll
        for (int f = 0; f < 4; ++f)
#pragma unroll
          for (int m = 0; m < 4; ++m)
            acc[m][f] = __builtin_amdgcn_mfma_f32_16x16x32_bf16(wf[f], xf[m], acc[m][f], 0, 0, 0);
      };
#define SGB(m, n) __builtin_amdgcn_sched_group_barrier(m, n, 0)
#define ILV() do { SGB(0x002, 5); SGB(0x100, 2); _Pragma("unroll") for (int i_ = 0; i_ < 6; ++i_) { SGB(0x008, 1); SGB(0x100, 1); } SGB(0x008, 10); } while (0)
      const u64 segs = grp == 0 ? (L < 2 ? 0x0000270f0800ull : (early ? 0x000000001c00ull : 0x0000230f0800ull))
                                : (L < 2 ? 0x000040270f08ull : (early ? 0x40380000381cull : 0x403838230f08ull));
      const u32 seq = next_seq(seqv);
      int nb = 0;
#pragma unroll 1
      for (int job = 0; job < 2; ++job) {
        if (job == 1 && (u32)(segs >> 32) == 0u) break;
#pragma unroll
        for (int m = 0; m < 4; ++m)
#pragma unroll
          for (int f = 0; f < 4; ++f) acc[m][f] = (f32x4){0.f, 0.f, 0.f, 0.f};
#pragma unroll 1
        for (int seg = 2 * job; seg < 2 + job; ++seg) {
          const int n0 = (int)((segs >> (16 * seg)) & 0xffu), n1 = (int)((segs >> (16 * seg + 8)) & 0xffu);
          if (n0 >= n1) continue;
#pragma unroll 1
          for (; nb <= n0; ++nb) asm volatile("s_barrier" ::: "memory");
          if (seg < 2) { STAMP(seg + 1); STAMPC(12); }
          rd(n0, 0, x0, w0);
#pragma unroll 1
          for (int n = n0; n < n1; ++n) {
            rd(n, 1, x1, w1);
            mm(x0, w0);
            ILV();
            __builtin_amdgcn_sched_barrier(0);
            asm volatile("s_waitcnt lgkmcnt(0)\n\ts_barrier" ::: "memory");
            rd(n + 1, 0, x0, w0);
            mm(x1, w1);
            ILV();
            __builtin_amdgcn_sched_barrier(0);
          }
          nb = n1 + 1;
          if (seg < 2) STAMPC(13);
        }
        STAMP(job == 0 ? 3 : 5);
        const u32 dtile = (myblk ^ (dquad != 0u ? 64u : 0u) ^ (dquad != 1u ? 32u : 0u)) + (u32)((g & 1) * 16 + (L & 1) * 8) + bb;
        const u32 obase = sgpr(job == 1 ? OFF_DS + dtile * 32768u
                        : (L == 0) ? OFF_PS + (seq & 1u) * 8u * MB + myh * 4u * MB + (myblk + (u32)((g & 1) * 16 + 8 * grp) + bb) * 32768u
                                   : OFF_TL + (u32)L * MB + (u32)(grp * 16 + g) * 32768u);
#pragma unroll
        for (int m = 0; m < 4; ++m)
#pragma unroll
          for (int hf = 0; hf < 2; ++hf) {
            u32x4 o;
            o[0] = nz(pk_bf16(acc[m][2 * hf][0], acc[m][2 * hf][1]));
            o[1] = nz(pk_bf16(acc[m][2 * hf][2], acc[m][2 * hf][3]));
            o[2] = nz(pk_bf16(acc[m][2 * hf + 1][0], acc[m][2 * hf + 1][1]));
            o[3] = nz(pk_bf16(acc[m][2 * hf + 1][2], acc[m][2 * hf + 1][3]));
            st16(o, rs_me, l16, obase + (u32)(((wr * 2 + (m >> 1)) * 2 + wc) * 4096 + ((m & 1) * 2 + hf) * 1024));
          }
        if (grp == 0) asm volatile("s_barrier\n\ts_barrier" ::: "memory");
        if (L != 0) {
          asm volatile("s_waitcnt vmcnt(0)" ::: "memory");
          __builtin_amdgcn_raw_buffer_store_b32(
              seq, rs_me, lane < 2 ? (u32)lane * 8u : OOB,
              (job == 1 ? OFF_DF + dtile * 32u : OFF_TF + (u32)L * 1024u + (u32)(grp * 16 + g) * 32u) + (u32)(wr * 4 + wc) * 4u, WT);
        }
        if (job == 0) STAMP(4);
      }
    }
  }
}

}

#define HIP_CHECK(expr) do { hipError_t _e = (expr); if (_e != hipSuccess) { \
    fprintf(stderr, "HIP error %s at %s:%d\n", hipGetErrorString(_e), __FILE__, __LINE__); \
    return; } } while (0)

extern "C" void kernel_launch(int rank, int nranks, ncclComm_t comm, hipStream_t stream, void* const* d_in,
                              const int* in_sizes, int n_in, void* d_out, int out_size, const int* in_dtypes,
                              int out_dtype, void* const* d_out_all, void* const* d_ws_all, size_t ws_bytes) {
  (void)comm; (void)in_dtypes; (void)out_dtype; (void)d_out_all;
  if (nranks != 8 || (unsigned)rank >= 8u || n_in != 2 || in_sizes[0] != 4096 * 512 || in_sizes[1] != 512 * 4096 ||
      out_size != 512 * 4096 || ws_bytes < (size_t)WS_USED) {
    fprintf(stderr, "kernel_launch: unexpected problem shape\n");
    return;
  }
  Params p;
  for (int i = 0; i < 8; ++i) p.wsx[i] = (char*)d_ws_all[rank ^ i];
  p.x = (const float*)d_in[0];
  p.w = (const float*)d_in[1];
  p.out = (signed char*)d_out;
  p.rank = rank;
  HIP_CHECK(hipFuncSetAttribute((const void*)fused_gemm_rs_q8, hipFuncAttributeMaxDynamicSharedMemorySize, LDS_BYTES));
  fused_gemm_rs_q8<<<dim3(256), dim3(NTHREADS), LDS_BYTES, stream>>>(p);
  HIP_CHECK(hipGetLastError());
}
```

```cpp
#include <hip/hip_runtime.h>
#include <rccl/rccl.h>
#include <cstdio>

typedef unsigned u32;
typedef unsigned long long u64;
typedef u32 u32x4 __attribute__((ext_vector_type(4)));
typedef u32 u32x2 __attribute__((ext_vector_type(2)));
typedef float f32x4 __attribute__((ext_vector_type(4)));
typedef float f32x2 __attribute__((ext_vector_type(2)));
typedef __bf16 bf16x2 __attribute__((ext_vector_type(2)));
typedef __bf16 bf16x8 __attribute__((ext_vector_type(8)));

namespace {

constexpr int NPX = 17;
constexpr int NP1 = 20;
constexpr u32 GE = 3;

constexpr u32 OFF_SEQ = 0;
constexpr u32 OFF_TMO = 256;
constexpr u32 OFF_TK = 1024;
constexpr u32 OFF_AM = 4096;
constexpr u32 OFF_XF = 16384;
constexpr u32 OFF_WF = 32768;
constexpr u32 OFF_TF = 49152;
constexpr u32 OFF_PF = 57344;
constexpr u32 OFF_DF = 61440;
constexpr u32 OFF_DBG = 65536;
constexpr u32 MB = 1u << 20;
constexpr u32 OFF_XS = 1 * MB;
constexpr u32 OFF_WS = 11 * MB;
constexpr u32 OFF_PS = 21 * MB;
constexpr u32 OFF_TL = 37 * MB;
constexpr u32 OFF_PR = 45 * MB;
constexpr u32 OFF_DS = 47 * MB;
constexpr u32 WS_USED = 51 * MB;
constexpr u32 SLOT_BYTES = 262144;
constexpr u32 OOB = 0x80000000u;
constexpr u64 DEADLINE = 100000000ull;
constexpr int WT = 17;
constexpr int NSTAGE = 4;
constexpr int LDS_POLL = NSTAGE * 32768;
constexpr int LDS_BYTES = LDS_POLL + 4096;
constexpr int NTHREADS = 768;
static_assert(GE <= 8 && NPX <= 32 && NP1 <= 32, "layout");

struct Params {
  char* wsx[8];
  const float* x;
  const float* w;
  signed char* out;
  int rank;
};

template <bool B> struct BoolTag { static constexpr bool value = B; };
typedef __amdgpu_buffer_rsrc_t rsrc_t;
typedef __attribute__((address_space(3))) u32 lds_u32;

__device__ __forceinline__ rsrc_t mk_rsrc(const void* p, u32 bytes) {
  return __builtin_amdgcn_make_buffer_rsrc((void*)p, (short)0, (int)bytes, 0x00020000);
}
__device__ __forceinline__ u32 pk_bf16(float lo, float hi) {
  f32x2 v = {lo, hi};
  bf16x2 b = __builtin_convertvector(v, bf16x2);
  return __builtin_bit_cast(u32, b);
}
__device__ __forceinline__ float as_f(u32 v) { return __builtin_bit_cast(float, v); }
__device__ __forceinline__ u32 as_u(float v) { return __builtin_bit_cast(u32, v); }
__device__ __forceinline__ float bf_lo(u32 v) { return as_f(v << 16); }
__device__ __forceinline__ float bf_hi(u32 v) { return as_f(v & 0xffff0000u); }
__device__ __forceinline__ u32 rfl(u32 v) { return __builtin_amdgcn_readfirstlane(v); }
__device__ __forceinline__ u32 sgpr(u32 s) { return rfl(s); }
__device__ __forceinline__ u32 next_seq(u32 v) { return rfl(v) + 1u; }
__device__ __forceinline__ void wait_vm0() {
  asm volatile("" ::: "memory");
  __builtin_amdgcn_s_waitcnt(0x0F70);
  asm volatile("" ::: "memory");
}
__device__ __forceinline__ u32 min16(const u32x4 (&d)[4]) {
  u32 m = min(min(d[0][0], d[0][1]), min(d[0][2], d[0][3]));
#pragma unroll
  for (int e = 1; e < 4; ++e) m = min(min(min(m, d[e][0]), d[e][1]), min(d[e][2], d[e][3]));
  return m;
}
__device__ __forceinline__ u32 nz(u32 d) { return d == 0u ? 0x8000u : d; }
__device__ __forceinline__ u32x4 ld16(rsrc_t r, u32 v, u32 s) { return __builtin_amdgcn_raw_buffer_load_b128(r, v, s, WT); }
__device__ __forceinline__ u32 ld4(rsrc_t r, u32 v, u32 s) { return __builtin_amdgcn_raw_buffer_load_b32(r, v, s, WT); }
__device__ __forceinline__ void st16(u32x4 d, rsrc_t r, u32 v, u32 s) { __builtin_amdgcn_raw_buffer_store_b128(d, r, v, s, WT); }

struct Spin {
  u64 t0; u32 n;
  __device__ __forceinline__ void start() { t0 = __builtin_amdgcn_s_memrealtime(); n = 0; }
  __device__ __forceinline__ bool expired(u32* tmo, u32 seq) {
    if ((++n & 31u) != 0u) return false;
    u32 t = rfl(__hip_atomic_load(tmo, __ATOMIC_RELAXED, __HIP_MEMORY_SCOPE_AGENT));
    if (t == seq) return true;
    if (__builtin_amdgcn_s_memrealtime() - t0 > DEADLINE) {
      __hip_atomic_store(tmo, seq, __ATOMIC_RELAXED, __HIP_MEMORY_SCOPE_AGENT);
      return true;
    }
    return false;
  }
};

#define STAMP_AT(off, val) do {} while (0)
#define STAMPC(k) do {} while (0)
#define STAMP(k) do {} while (0)

__device__ __forceinline__ u32 wave_max_u(u32 v) {
  v = max(v, (u32)__builtin_amdgcn_update_dpp(0, (int)v, 0x111, 0xf, 0xf, true));
  v = max(v, (u32)__builtin_amdgcn_update_dpp(0, (int)v, 0x112, 0xf, 0xf, true));
  v = max(v, (u32)__builtin_amdgcn_update_dpp(0, (int)v, 0x114, 0xf, 0xf, true));
  v = max(v, (u32)__builtin_amdgcn_update_dpp(0, (int)v, 0x118, 0xf, 0xf, true));
  v = max(v, (u32)__builtin_amdgcn_update_dpp(0, (int)v, 0x142, 0xa, 0xf, true));
  v = max(v, (u32)__builtin_amdgcn_update_dpp(0, (int)v, 0x143, 0xc, 0xf, true));
  return (u32)__builtin_amdgcn_readlane((int)v, 63);
}

__device__ __forceinline__ void dma8(rsrc_t r, u32 voff, u32 soff, u32 ldsaddr) {
  u32 soff2;
  asm volatile(
      "s_mov_b32 m0, %4\n\t"
      "s_add_u32 %0, %3, 0x1000\n\t"
      "buffer_load_dwordx4 %1, %2, %3 offen sc0 sc1 lds\n\t"
      "buffer_load_dwordx4 %1, %2, %3 offen offset:1024 sc0 sc1 lds\n\t"
      "buffer_load_dwordx4 %1, %2, %3 offen offset:2048 sc0 sc1 lds\n\t"
      "buffer_load_dwordx4 %1, %2, %3 offen offset:3072 sc0 sc1 lds\n\t"
      "s_add_u32 m0, %4, 0x1000\n\t"
      "s_nop 0\n\t"
      "buffer_load_dwordx4 %1, %2, %0 offen sc0 sc1 lds\n\t"
      "buffer_load_dwordx4 %1, %2, %0 offen offset:1024 sc0 sc1 lds\n\t"
      "buffer_load_dwordx4 %1, %2, %0 offen offset:2048 sc0 sc1 lds\n\t"
      "buffer_load_dwordx4 %1, %2, %0 offen offset:3072 sc0 sc1 lds"
      : "=&s"(soff2)
      : "v"(voff), "s"(r), "s"(soff), "s"(ldsaddr)
      : "memory", "scc");
}

__global__ void __launch_bounds__(NTHREADS, 3) fused_gemm_rs_q8(Params p) {
  extern __shared__ __attribute__((aligned(16))) char smem[];
  const int b = blockIdx.x, L = b & 7, q = b >> 3;
  const int tid = threadIdx.x, lane = tid & 63;
  const int wave = (int)rfl((u32)tid >> 6);
  const int rank = p.rank & 7;
  char* myws = p.wsx[0];
  asm volatile("" :: "s"(p.x), "s"(p.w), "s"(p.out));
  const rsrc_t rs_me = mk_rsrc(myws, WS_USED);
  u32* tmo = (u32*)(myws + OFF_TMO);
  const int peer = rank ^ L;
  const rsrc_t rs_peer = mk_rsrc(p.wsx[L], WS_USED);
  const u32 seqv = __hip_atomic_load((u32*)(myws + OFF_SEQ), __ATOMIC_RELAXED, __HIP_MEMORY_SCOPE_AGENT);
  const u32 myh = (u32)rank & 1u;
  const u32 myblk = (u32)((((rank >> 1) & 1) * 2 + ((rank >> 2) & 1)) * 32);
  const u32 l16 = (u32)lane * 16u;
  STAMP(0);

  if (q < 16) {
    const bool pusher = wave >= 8;
    const int NP = L == 1 ? NP1 : NPX;
    const u32 cwi = (u32)((L - 1) * 16 + q);
    const u32 cw = L == 0 ? (u32)((q >> 1) * 16 + 14 + (q & 1)) : (cwi / 14u) * 16u + cwi % 14u;
    volatile lds_u32* lds = (volatile lds_u32*)smem;
    auto am_dst = [&]() { return (u64*)(p.wsx[lane & 7] + OFF_AM + ((u32)rank * 128u + cw) * 8u); };
    auto publish = [&](u64* dst, u32 seq) {
      if (lane < 8) {
        u32 m = lds[8];
#pragma unroll
        for (int k = 1; k < 8; ++k) m = max(m, lds[8 + k]);
        __hip_atomic_store(dst, ((u64)seq << 32) | (u64)m, __ATOMIC_RELAXED, __HIP_MEMORY_SCOPE_SYSTEM);
      }
    };
    if (L == 0 && wave == 8) {
      u64* dst = am_dst();
      u32 t1 = 0u;
      if (lane < 8) t1 = (u32)__hip_atomic_load(dst, __ATOMIC_RELAXED, __HIP_MEMORY_SCOPE_SYSTEM);
      const u32 seq = next_seq(seqv);
      asm volatile("s_barrier" :: "v"(t1) : "memory");
      publish(dst, seq);
      return;
    }
    if (pusher && (L == 0 || (wave - 8) * 16 + q >= NP)) return;
    {
      const u32 oid = (u32)((q * 2 + L) * 8 + wave);
      const int vp = pusher ? peer : rank ^ (int)(oid & 1u);
      const u32 dr = (u32)(pusher ? peer : rank);
      const rsrc_t rs_x = mk_rsrc(p.x, 4096u * 512u * 4u);
      const rsrc_t rs_w = mk_rsrc(p.w, 512u * 4096u * 4u);
      const rsrc_t rs_dst = pusher ? rs_peer : rs_me;
      const u32 pi = (vp >> 1) & 1, pj = (vp >> 2) & 1, ph = vp & 1;
      const u32 slotbase = 4u * (u32)(pusher ? rank : vp);
      const u32 vx = (u32)(lane >> 3) * 2048u + (u32)(lane & 7) * 16u;
      const u32 vw = (u32)(lane >> 3) * 65536u + (u32)(lane & 7) * 16u;
      const u32 dx0 = (u32)(lane >> 3) * 128u + (u32)(((lane & 7) ^ (lane >> 4)) * 16);
      const u32 fl_lane = lane == 0 ? 0u : OOB;
      u32x4 ld[8];
      auto desc_t = [&](auto push_tag, u32 v, u32& isw, u32& src, u32& dst, u32& flg) {
        constexpr bool P = decltype(push_tag)::value;
        constexpr u32 NX = P ? 10u : 12u, N0 = NX + 2u * GE, NA = NX + 24u - 2u * GE, E0 = 4u * (N0 + NA);
        auto owner = [&](u32 k) { u32 lo = 2u + k; if (P && lo >= (u32)L) ++lo; return dr ^ lo; };
        u32 t, r, slot = slotbase, ii = pi, jj = pj;
        if (v < 4u * N0) {
          t = v / N0; const u32 e = v % N0;
          isw = e >= NX;
          r = isw ? e - NX : ((owner(e >> 1) * 2u + (dr & 1u)) * 2u + (e & 1u));
        } else if (v < E0) {
          const u32 v2 = v - 4u * N0;
          t = v2 / NA; const u32 e = v2 % NA;
          isw = e >= NX + 8u;
          r = isw ? 2u * GE + e - (NX + 8u)
                  : (e < NX ? ((owner(e >> 1) * 2u + ((dr & 1u) ^ 1u)) * 2u + (e & 1u)) : (dr & 6u) * 4u + e - NX);
        } else if (P || v < E0 + 64u) {
          const u32 v2 = v - E0;
          t = v2 >> 4; r = 16u + (v2 & 15u); isw = 1u;
        } else {
          const u32 v2 = v - (E0 + 64u);
          t = v2 / 36u; const u32 e = v2 % 36u;
          isw = e < 32u;
          r = isw ? e : dr * 4u + e - 32u;
          slot = 32u + (oid & 1u) * 4u; ii ^= 1u; jj ^= 1u;
        }
        slot += t;
        const u32 k0 = ph * 256u + t * 64u;
        if (isw) {
          src = k0 * 16384u + (jj * 2048u + r * 64u) * 4u;
          dst = OFF_WS + slot * SLOT_BYTES + r * 8192u;
          flg = OFF_WF + (r >> 1) * 1024u + slot * 16u + (r & 1u) * 8u;
        } else {
          src = ((r >> 2) * 512u + ii * 256u + (r & 3u) * 64u) * 2048u + k0 * 4u;
          dst = OFF_XS + slot * SLOT_BYTES + r * 8192u;
          flg = OFF_XF + (r >> 1) * 1024u + slot * 16u + (r & 1u) * 8u;
        }
      };
      auto loads = [&](u32x4 (&ld)[8], u32 isw, u32 src) {
        const rsrc_t rs = isw ? rs_w : rs_x;
        const u32 vo = isw ? vw : vx, s1 = isw ? 16384u : 128u, s2 = isw ? 32768u : 16384u, s4 = isw ? 524288u : 32768u;
#pragma unroll
        for (int j = 0; j < 8; ++j)
          ld[j] = __builtin_amdgcn_raw_buffer_load_b128(rs, vo, src + ((j & 1) ? s1 : 0u) + ((j & 2) ? s2 : 0u) + ((j & 4) ? s4 : 0u), 0);
      };
      u32 seq;
      auto fdesc = [&](u32 f, u32& s1, u32& f1, u32& s2, u32& f2, u32& dst, u32& dfl) {
        const u32 reg = f & 7u;
        u32 sl = (u32)L, tile, g;
        s2 = OOB; dfl = OOB;
        bool direct = true;
        if (L == 1) {
          if (f < 48u * GE) {
            const u32 k = (f >> 3) % GE;
            sl = 2u + (f >> 3) / GE; tile = 0u; g = 2u * k + myh;
            dst = OFF_PR + (((sl ^ 1u) * 64u) + k * 8u + reg) * 4096u;
            dfl = OFF_PF + (((sl ^ 1u) * 64u) + k * 8u + reg) * 4u;
            direct = false;
          } else { const u32 k = (f - 48u * GE) >> 3; tile = k >> 4; g = k & 15u; }
        } else {
          const u32 k = f >> 3;
          if (k < 16u - 2u * GE) { tile = 0u; g = 2u * GE + k; }
          else if (k < 16u - GE) {
            const u32 kk = k - (16u - 2u * GE);
            tile = 0u; g = 2u * kk + (myh ^ 1u);
            s2 = OFF_PR + ((u32)L * 64u + kk * 8u + reg) * 4096u;
          } else { tile = 1u; g = k - (16u - GE); }
        }
        const u32 fu = tile * 128u + g * 8u + reg;
        s1 = OFF_TL + sl * MB + fu * 4096u;
        f1 = OFF_TF + sl * 1024u + fu * 4u;
        f2 = (s2 == OOB) ? f1 : OFF_PF + ((s2 - OFF_PR) >> 10);
        if (direct)
          dst = OFF_PS + (seq & 1u) * 8u * MB + myh * 4u * MB + ((myblk + (g & 1u) * 16u + (g >> 1) + tile * 8u) * 8u + reg) * 4096u;
      };
      u32 s1, f1, s2, f2, fdst, fdfl;
      const u32 two_lanes = lane < 2 ? 0u : OOB;
      auto fflags = [&]() { return ld4(rs_me, (lane == 1 ? f2 : f1) | two_lanes, 0u); };
      auto fready = [&](u32 v) { return __all(lane >= 2 || v == seq); };
      auto floads = [&]() {
#pragma unroll
        for (int e = 0; e < 4; ++e) ld[e] = ld16(rs_me, l16, s1 + (u32)e * 1024u);
        if (s2 != OOB) {
#pragma unroll
          for (int e = 0; e < 4; ++e) ld[4 + e] = ld16(rs_me, l16, s2 + (u32)e * 1024u);
        }
      };
      const u32 nin = L == 1 ? 0u : 240u;
      const u32 ntot = (L == 1) ? 48u * GE + 256u : 496u - 8u * GE;
      auto draw = [&]() {
        return (u32)__builtin_amdgcn_raw_ptr_buffer_atomic_add_i32(1, rs_me, (int)fl_lane, (int)(OFF_TK + (u32)L * 128u), 0);
      };
      u32 v = pusher ? (u32)((wave - 8) * 16 + q) : oid >> 1;
      u32 vn = v + (u32)NP;
      u32 tkv = 0u, tkbase, pflg = OOB;
      bool have = false;
      auto emit = [&](u32x4 (&ld)[8], u32 isw, u32 dst) {
        u32x4 o[4];
        if (isw) {
#pragma unroll
          for (int c = 0; c < 4; ++c)
#pragma unroll
            for (int d = 0; d < 4; ++d) o[c][d] = pk_bf16(as_f(ld[2 * d][c]), as_f(ld[2 * d + 1][c]));
        } else {
#pragma unroll
          for (int g = 0; g < 4; ++g) {
            o[g][0] = pk_bf16(as_f(ld[2 * g][0]), as_f(ld[2 * g][1]));
            o[g][1] = pk_bf16(as_f(ld[2 * g][2]), as_f(ld[2 * g][3]));
            o[g][2] = pk_bf16(as_f(ld[2 * g + 1][0]), as_f(ld[2 * g + 1][1]));
            o[g][3] = pk_bf16(as_f(ld[2 * g + 1][2]), as_f(ld[2 * g + 1][3]));
          }
        }
#pragma unroll
        for (int c = 0; c < 4; ++c)
          st16(o[c], rs_dst, isw ? (u32)((lane ^ c) * 16) : dx0 ^ (u32)((c & 1) * 64), dst + (u32)c * 1024u);
        asm volatile("" ::: "memory");
      };
      if (!pusher) {
        seq = next_seq(seqv);
        if (L < 2) {
          const u32x2 seq2 = {seq, seq};
          u32x4 lb[8];
#pragma unroll 1
          for (; v < 400u; v += 128u) {
            u32 isw, src, dst, flg;
            desc_t(BoolTag<false>(), v, isw, src, dst, flg);
            loads(ld, isw, src);
            asm volatile("" ::: "memory");
            loads(lb, isw, src + (isw ? 128u : 65536u));
            asm volatile("s_waitcnt vmcnt(8)" ::: "memory");
            __builtin_amdgcn_raw_buffer_store_b64(seq2, rs_me, fl_lane | pflg, 0, WT);
            emit(ld, isw, dst);
            emit(lb, isw, dst + 4096u);
            pflg = flg;
            if (v < 128u) {
              wait_vm0();
              __builtin_amdgcn_raw_buffer_store_b64(seq2, rs_me, fl_lane | pflg, 0, WT);
              pflg = OOB;
            }
          }
          wait_vm0();
          __builtin_amdgcn_raw_buffer_store_b64(seq2, rs_me, fl_lane | pflg, 0, WT);
        }
      } else if (L != 1) {
        u32 isw, src, dst, flg;
        desc_t(BoolTag<true>(), v, isw, src, dst, flg);
        loads(ld, isw, src);
        seq = next_seq(seqv);
        tkbase = (seq - 1u) * (ntot - (u32)NP) - (u32)(2 * NP);
        tkv = vn + tkbase;
        const u32x2 seq2 = {seq, seq};
#pragma unroll 1
        for (;;) {
          wait_vm0();
          emit(ld, isw, dst);
          __builtin_amdgcn_raw_buffer_store_b64(seq2, rs_dst, fl_lane | pflg, 0, WT);
          loads(ld, isw, src + (isw ? 128u : 65536u));
          vn = rfl(tkv) - tkbase;
          if (vn >= nin) break;
          u32 nisw, nsrc, ndst, nflg;
          desc_t(BoolTag<true>(), vn, nisw, nsrc, ndst, nflg);
          wait_vm0();
          emit(ld, isw, dst + 4096u);
          loads(ld, nisw, nsrc);
          tkv = draw();
          pflg = flg;
          isw = nisw; src = nsrc; dst = ndst; flg = nflg;
        }
        v = vn;
        fdesc(v - 240u, s1, f1, s2, f2, fdst, fdfl);
        wait_vm0();
        const u32 nfl = fflags();
        emit(ld, isw, dst + 4096u);
        pflg = flg;
        have = fready(nfl);
        asm volatile("" ::: "memory");
        if (have) floads();
        tkv = draw();
      } else {
        seq = next_seq(seqv);
        tkbase = (seq - 1u) * (ntot - (u32)NP) - (u32)(2 * NP);
        tkv = vn + tkbase;
        fdesc(v, s1, f1, s2, f2, fdst, fdfl);
      }
      STAMP(1);
      if (pusher) {
        u32 pfl2 = pflg == OOB ? OOB : pflg + 4u;
#pragma unroll 1
        for (;;) {
          wait_vm0();
          vn = rfl(tkv) - tkbase;
          if (!have) {
            __builtin_amdgcn_raw_buffer_store_b32(seq, rs_peer, (lane == 1 ? pfl2 : pflg) | two_lanes, 0, WT);
            pflg = OOB; pfl2 = OOB;
            Spin sp; sp.start();
            for (;;) {
              if (fready(fflags())) break;
              if (sp.expired(tmo, seq)) break;
              __builtin_amdgcn_s_sleep(4);
            }
            asm volatile("" ::: "memory");
            floads();
            wait_vm0();
          }
          const u32 cdst = sgpr(fdst), cs2 = s2, cfl = fdfl;
          STAMP_AT(524288u + ((u32)L * 512u + v - nin) * 8u, __builtin_amdgcn_s_memrealtime());
          u32 nfl = 0u;
          if (vn < ntot) {
            fdesc(vn - nin, s1, f1, s2, f2, fdst, fdfl);
            nfl = fflags();
          }
          if (cs2 != OOB) {
#pragma unroll
            for (int e = 0; e < 4; ++e)
#pragma unroll
              for (int d = 0; d < 4; ++d)
                ld[e][d] = nz(pk_bf16(bf_lo(ld[e][d]) + bf_lo(ld[4 + e][d]), bf_hi(ld[e][d]) + bf_hi(ld[4 + e][d])));
          }
#pragma unroll
          for (int e = 0; e < 4; ++e) st16(ld[e], rs_peer, l16, cdst + (u32)e * 1024u);
          __builtin_amdgcn_raw_buffer_store_b32(seq, rs_peer, (lane == 1 ? pfl2 : pflg) | two_lanes, 0, WT);
          pflg = cfl; pfl2 = OOB;
          if (vn >= ntot) break;
          v = vn;
          have = fready(nfl);
          asm volatile("" ::: "memory");
          if (have) floads();
          tkv = draw();
        }
        STAMP(4);
        wait_vm0();
        __builtin_amdgcn_raw_buffer_store_b32(seq, rs_peer, fl_lane | pflg, 0, WT);
        STAMP(5);
        return;
      }
    }
    const u32 seq = next_seq(seqv);
    const u32 par = seq & 1u;
    const u32 bcol = cw & 15u, ra = (cw >> 4) & 1u, tj = (cw >> 5) & 1u, ti = (cw >> 6) & 1u;
    const u32 ooff = (ti * 256u + ra * 128u + (u32)(wave >> 1) * 32u + (u32)(lane & 15)) * 4096u + tj * 2048u + bcol * 128u +
                     (u32)(wave & 1) * 64u + (u32)(lane >> 4) * 16u;
    const rsrc_t rs_out = mk_rsrc(p.out, 512u * 4096u);
    u64* amd = am_dst();
    {
      u32 t0 = __builtin_amdgcn_raw_buffer_load_b32(rs_out, ooff, 0, 0);
      u32 t1 = 0u;
      if (tid < 8) t1 = (u32)__hip_atomic_load(amd, __ATOMIC_RELAXED, __HIP_MEMORY_SCOPE_SYSTEM);
      u32 t2 = ld4(rs_peer, tid == 64 ? sgpr(OFF_PS + par * 8u * MB + myh * 4u * MB + myblk * 32768u) : (tid == 65 ? OFF_PR : OOB), 0u);
      asm volatile("" :: "v"(t0), "v"(t1), "v"(t2));
    }
    if (L == 0) {
      const u32x4 zero4 = {0u, 0u, 0u, 0u};
      const u32 zo = sgpr(OFF_PS + (par ^ 1u) * 8u * MB + (u32)(q * 8 + wave) * 65536u);
#pragma unroll 1
      for (u32 k = 0; k < 16u; ++k) {
        __builtin_amdgcn_s_sleep(40);
#pragma unroll
        for (int e = 0; e < 4; ++e) st16(zero4, rs_me, l16, zo + k * 4096u + (u32)e * 1024u);
      }
      STAMP(12);
    }
    float y[2][16];
    {
      const u32 so = sgpr(OFF_PS + par * 8u * MB + (cw * 8u + (u32)wave) * 4096u);
      const bool pre = bcol < GE && (tj * 2u + ti) != ((u32)rank >> 1);
      bool ok0 = pre && ra == 0u, ok1 = pre && ra == 1u;
      u32x4 d0[4], d1[4], d2[4];
#pragma unroll
      for (int e = 0; e < 4; ++e) { d0[e] = (u32x4){0u, 0u, 0u, 0u}; d1[e] = d0[e]; d2[e] = d0[e]; }
      Spin sp; sp.start();
      auto diag = [&]() {
        if ((cw & 96u) == myblk) return;
        for (;;) {
          if (rfl(ld4(rs_me, 0u, OFF_DF + (cw * 8u + (u32)wave) * 4u)) == seq) break;
          if (sp.expired(tmo, seq)) break;
          __builtin_amdgcn_s_sleep(16);
        }
        asm volatile("" ::: "memory");
#pragma unroll
        for (int e = 0; e < 4; ++e) d2[e] = ld16(rs_me, l16, OFF_DS + (cw * 8u + (u32)wave) * 4096u + (u32)e * 1024u);
      };
      diag();
      {
        const u32 pv = (lane == 1 ? (ok1 ? OOB : 4u * MB + 4092u) : (ok0 ? OOB : 4092u)) | (lane < 2 ? 0u : OOB);
        const u32 po = L == 0 ? so - 2u * 32768u : so;
        for (;;) {
          const u32 v = ld4(rs_me, pv, po);
          if (__all(pv >= OOB || v != 0u)) break;
          if (sp.expired(tmo, seq)) break;
          __builtin_amdgcn_s_sleep(2);
        }
      }
      STAMP(8); STAMP_AT(((blockIdx.x * 12 + wave) * 16 + 10) * 8, (u64)sp.n);
      for (;;) {
        if (!ok0) {
#pragma unroll
          for (int e = 0; e < 4; ++e) d0[e] = ld16(rs_me, l16, so + (u32)e * 1024u);
        }
        if (!ok1) {
#pragma unroll
          for (int e = 0; e < 4; ++e) d1[e] = ld16(rs_me, l16, so + 4u * MB + (u32)e * 1024u);
        }
        ok0 = ok0 || __all(min16(d0) != 0u);
        ok1 = ok1 || __all(min16(d1) != 0u);
        if (ok0 && ok1) break;
        if (sp.expired(tmo, seq)) break;
        __builtin_amdgcn_s_sleep(1);
      }
      STAMP(4); STAMP_AT(((blockIdx.x * 12 + wave) * 16 + 9) * 8, (u64)sp.n);
      float mx = 0.f;
#pragma unroll
      for (int e = 0; e < 4; ++e)
#pragma unroll
        for (int d = 0; d < 4; ++d) {
          const float lo = (bf_lo(d0[e][d]) + bf_lo(d1[e][d])) + bf_lo(d2[e][d]);
          const float hi = (bf_hi(d0[e][d]) + bf_hi(d1[e][d])) + bf_hi(d2[e][d]);
          y[e >> 1][(e & 1) * 8 + d * 2] = lo;
          y[e >> 1][(e & 1) * 8 + d * 2 + 1] = hi;
          mx = fmaxf(mx, fmaxf(fabsf(lo), fabsf(hi)));
        }
      lds[8 + wave] = wave_max_u(as_u(mx));
      asm volatile("s_waitcnt lgkmcnt(0)\n\ts_barrier" ::: "memory");
      if (wave == 0 && L != 0) publish(amd, seq);
    }
    STAMP(5);
    auto quantize = [&](u32 m) {
      const bool anybad = m == 0xffffffffu;
      const float inv = m ? 127.0f / as_f(m) : 0.f;
#pragma unroll
      for (int mm = 0; mm < 2; ++mm) {
        u32x4 o;
#pragma unroll
        for (int d = 0; d < 4; ++d) {
          u32 t[4];
#pragma unroll
          for (int k = 0; k < 4; ++k)
            t[k] = as_u(__builtin_amdgcn_fmed3f(__builtin_fmaf(y[mm][d * 4 + k], inv, 12582912.f), 12582912.f - 127.f, 12582912.f + 127.f));
          const u32 wd = __builtin_amdgcn_perm(__builtin_amdgcn_perm(t[3], t[2], 0x0c0c0400u),
                                               __builtin_amdgcn_perm(t[1], t[0], 0x0c0c0400u), 0x05040100u);
          o[d] = anybad ? 0x80808080u : wd;
        }
        __builtin_amdgcn_raw_buffer_store_b128(o, rs_out, ooff, (u32)mm * 65536u, 0);
      }
    };
    {
      Spin sp; sp.start();
      u32 done_m = 0x7fffffffu;
      bool spec = true;
      const u32 tmo_lane = tid == 0 ? 0u : OOB;
#pragma unroll 1
      for (u32 round = 0;; ++round) {
        u32x4 g;
        u32 bad, cnt;
        for (;;) {
          g = ld16(rs_me, (u32)tid * 16u, OFF_AM);
          bad = ld4(rs_me, tmo_lane, OFF_TMO);
          cnt = (u32)__popcll(__ballot(g[1] == seq && g[3] == seq));
          if (cnt == 64u || (spec && cnt >= 48u)) break;
          if (sp.expired(tmo, seq)) { bad = seq; cnt = 64u; break; }
          if (cnt < 48u) __builtin_amdgcn_s_sleep(8);
        }
        STAMP(round == 0u ? 13 : 11);
        const u32 wm = wave_max_u(max(g[1] == seq ? g[0] : 0u, g[3] == seq ? g[2] : 0u));
        volatile lds_u32* sl = lds + 16 + (round & 1u) * 16u;
        sl[wave] = (rfl(bad) == seq) ? 0xffffffffu : wm;
        sl[8 + wave] = cnt;
        asm volatile("s_waitcnt lgkmcnt(0)\n\ts_barrier" ::: "memory");
        u32 m = sl[0], c = sl[8];
#pragma unroll
        for (int k = 1; k < 8; ++k) { m = max(m, sl[k]); c += sl[8 + k]; }
        m = rfl(m); c = rfl(c);
        if (c == 512u) STAMP(6);
        if (m != done_m) { quantize(m); done_m = m; }
        if (c == 512u) break;
        spec = false;
      }
    }
    if (b == 0 && tid == 0) __hip_atomic_store((u32*)(myws + OFF_SEQ), seq, __ATOMIC_RELAXED, __HIP_MEMORY_SCOPE_AGENT);
    STAMP(7);
  } else {
    const int g = q - 16;
    const u32 bb = (u32)(g >> 1);
    const bool early = L >= 2 && bb < GE && (u32)(g & 1) == myh;
    const u32 dquad = (u32)((L - 2) >> 1);
    if (wave >= 8) {
      __builtin_amdgcn_s_setprio(3);
      const int j = wave - 8;
      const u32 a = (u32)(2 * peer + (g & 1));
      u32 goff, foff;
      {
        const int ROT8 = (int)bb + 4 * (g & 1), ROT6 = (int)bb + 3 * (g & 1);
        const int n = lane;
        int k1, k2, kd, r;
        if (L < 2) {
          const int t2a = L == 0 ? 33 : 39;
          k1 = n < 8 ? n : -1; kd = -1;
          k2 = (n >= 8 && n < 15) ? n - 8 : (n == t2a ? 7 : -1);
          r = n < t2a ? n - 15 : (n < t2a + 25 ? n - t2a - 1 : n - 40);
        } else {
          kd = n >= 56 ? n - 56 : -1;
          if (early) {
            k1 = (n >= 20 && n < 28) ? n - 20 : -1;
            k2 = (n >= 28 && n < 36) ? n - 28 : -1;
            r = n < 20 ? n : n - 36;
          } else {
            k1 = n < 8 ? n : -1;
            k2 = (n >= 8 && n < 15) ? n - 8 : (n == 35 ? 7 : -1);
            r = n < 35 ? n - 15 : n - 36;
          }
        }
        r = r < 0 ? 0 : r;
        const bool ist2 = k2 >= 0 || (L == 0 ? (n >= 34 && n < 58) : n >= (L == 1 ? 40 : 36));
        u32 xslot, wslot, ab = (ist2 && L < 2) ? a ^ 2u : a, bcol = bb + (ist2 ? 8u : 0u);
        if (k1 >= 0) xslot = wslot = (u32)(4 * (rank ^ ((k1 + g) & 1)) + (k1 >> 1));
        else if (k2 >= 0) { const int k = (k2 + ROT8) & 7; xslot = wslot = (u32)(4 * (rank ^ (k >> 2)) + (k & 3)); }
        else if (kd >= 0) {
          const int k = (kd + ROT8) & 7;
          const u32 own = (u32)(4 * (rank ^ (k >> 2)) + (k & 3));
          xslot = dquad != 0u ? 32u + (u32)k : own;
          wslot = dquad != 1u ? 32u + (u32)k : own;
          ab = (u32)(2 * rank + (g & 1)); bcol = (u32)(L & 1) * 8u + bb;
        } else if (L < 2) xslot = wslot = (u32)(4 * (rank ^ (2 + (r + ROT6) % 6)) + r / 6);
        else {
          int ls = 2 + (r + ROT6) % 5;
          if (ls >= L) ++ls;
          xslot = wslot = (u32)(4 * (rank ^ ls) + r / 5);
        }
        if (j < 2) {
          goff = OFF_XS + xslot * SLOT_BYTES + (ab * 4u + (u32)(2 * j)) * 4096u;
          foff = OFF_XF + ab * 1024u + xslot * 16u;
        } else {
          goff = OFF_WS + wslot * SLOT_BYTES + (bcol * 4u + (u32)(2 * (j - 2))) * 4096u;
          foff = OFF_WF + bcol * 1024u + wslot * 16u;
        }
      }
      const u32 seq = next_seq(seqv);
      const u32 lds_base0 = rfl((u32)(size_t)(__attribute__((address_space(3))) char*)smem);
      const u32 lds_base = lds_base0 + (u32)j * 8192u;
      int cov = 0;
      int iss = 0, since = 64;
      bool pend = false;
      const u32 poll_lds = lds_base0 + (u32)LDS_POLL + (u32)j * 1024u;
      const volatile lds_u32* pl = (const volatile lds_u32*)(smem + LDS_POLL + j * 1024 + lane * 16 + (j & 1) * 8);
      auto prefix = [&](u32 f) {
        const u64 miss = __ballot((int)(f - seq) < 0);
        return miss ? (int)__builtin_ctzll(miss) : 64;
      };
      auto issue = [&](int k) {
        u32 go = (u32)__builtin_amdgcn_readlane((int)goff, k);
        asm volatile("s_nop 4" : "+s"(go));
        dma8(rs_me, l16, go, lds_base + (u32)(k & (NSTAGE - 1)) * 32768u);
      };
#pragma unroll 1
      for (int n = 0; n < 64; ++n) {
        if (iss <= n) {
          if (cov <= n) {
            Spin sp; sp.start();
            for (;;) {
              cov = prefix(ld4(rs_me, foff + (u32)(j & 1) * 8u, 0u));
              if (cov > n) break;
              if (sp.expired(tmo, seq)) { cov = 64; break; }
              __builtin_amdgcn_s_sleep(1);
            }
          }
          issue(n);
          iss = n + 1;
        }
        int k = iss - n - 1;
        if (pend && since < k) k = since;
        const int lim = (n + NSTAGE < 64) ? n + NSTAGE : 64;
        if (k >= 2) asm volatile("s_waitcnt vmcnt(16)\n\ts_barrier" ::: "memory");
        else if (k == 1) asm volatile("s_waitcnt vmcnt(8)\n\ts_barrier" ::: "memory");
        else asm volatile("s_waitcnt vmcnt(0)\n\ts_barrier" ::: "memory");
        if (pend) {
          const int c = prefix(pl[0]);
          cov = c > cov ? c : cov;
          if (cov == 64) cov = 1000;
          pend = false;
        }
        if (cov < lim + 4) {
          asm volatile("s_waitcnt lgkmcnt(0)\n\t"
                       "s_mov_b32 m0, %2\n\t"
                       "s_nop 0\n\t"
                       "buffer_load_dwordx4 %0, %1, 0 offen sc0 sc1 lds"
                       :: "v"(foff), "s"(rs_me), "s"(poll_lds) : "memory");
          pend = true;
          since = 0;
        }
        const int top = lim < cov ? lim : cov;
        while (iss < top) { issue(iss); ++iss; ++since; }
        if (!pend && iss == n + NSTAGE) {
          const int last = (cov - 8 < 64 - NSTAGE) ? cov - 8 : 64 - NSTAGE;
          u32 la = (u32)(iss & (NSTAGE - 1)) * 32768u;
          u32 go = (u32)__builtin_amdgcn_readlane((int)goff, iss & 63);
#pragma unroll 1
          while (n < last) {
            asm volatile("s_waitcnt vmcnt(16)\n\ts_barrier" : "+s"(go) :: "memory");
            dma8(rs_me, l16, go, lds_base + la);
            ++n; ++iss;
            la = (la + 32768u) & (u32)(NSTAGE * 32768 - 1);
            go = (u32)__builtin_amdgcn_readlane((int)goff, iss & 63);
          }
        }
      }
    } else {
      const int grp = wave >> 2, wr = (wave >> 1) & 1, wc = wave & 1, fi = lane & 15, kq = lane >> 4;
      u32 ax[2], aw[4];
#pragma unroll
      for (int ks = 0; ks < 2; ++ks) ax[ks] = (u32)((wr * 64 + fi) * 128 + (((kq + 4 * ks) ^ (fi >> 1)) * 16));
#pragma unroll
      for (int f = 0; f < 4; ++f)
        aw[f] = (u32)(16384 + wc * 8192 + (fi >> 3) * 4096 + (fi & 3) * 1024 + ((kq * 8 + 4 * ((fi >> 2) & 1) + (f ^ (fi & 3))) * 16));
      f32x4 acc[4][4];
      bf16x8 x0[4], w0[4], x1[4], w1[4];
      auto rd = [&](int n, int ks, bf16x8 (&xf)[4], bf16x8 (&wf)[4]) {
        const char* st = smem + (n & (NSTAGE - 1)) * 32768;
#pragma unroll
        for (int m = 0; m < 4; ++m) {
          wf[m] = *(const bf16x8*)(st + aw[m] + ks * 512);
          xf[m] = *(const bf16x8*)(st + ax[ks] + m * 2048);
        }
      };
      auto mm = [&](bf16x8 (&xf)[4], bf16x8 (&wf)[4]) {
#pragma unroll
        for (int f = 0; f < 4; ++f)
#pragma unroll
          for (int m = 0; m < 4; ++m)
            acc[m][f] = __builtin_amdgcn_mfma_f32_16x16x32_bf16(wf[f], xf[m], acc[m][f], 0, 0, 0);
      };
#define SGB(m, n) __builtin_amdgcn_sched_group_barrier(m, n, 0)
#define ILV() do { SGB(0x002, 5); SGB(0x100, 2); _Pragma("unroll") for (int i_ = 0; i_ < 6; ++i_) { SGB(0x008, 1); SGB(0x100, 1); } SGB(0x008, 10); } while (0)
      const u64 segs = grp == 0 ? (L == 0 ? 0x403a210f0800ull : L == 1 ? 0x0000270f0800ull : (early ? 0x000000001c00ull : 0x0000230f0800ull))
                                : (L == 0 ? 0x00003a210f08ull : L == 1 ? 0x000040270f08ull : (early ? 0x40380000381cull : 0x403838230f08ull));
      const int nsplit = (grp == 1 && L >= 2) ? 2 : 3;
      const int Lg = (grp == 1 && L < 2) ? L ^ 1 : L;
      const u32 seq = next_seq(seqv);
      int nb = 0;
#pragma unroll 1
      for (int job = 0; job < 2; ++job) {
        if (job == 1 && nsplit == 3) break;
#pragma unroll
        for (int m = 0; m < 4; ++m)
#pragma unroll
          for (int f = 0; f < 4; ++f) acc[m][f] = (f32x4){0.f, 0.f, 0.f, 0.f};
#pragma unroll 1
        for (int seg = job ? nsplit : 0; seg < (job ? 3 : nsplit); ++seg) {
          const int n0 = (int)((segs >> (16 * seg)) & 0xffu), n1 = (int)((segs >> (16 * seg + 8)) & 0xffu);
          if (n0 >= n1) continue;
#pragma unroll 1
          for (; nb <= n0; ++nb) asm volatile("s_barrier" ::: "memory");
          if (seg < 2) { STAMP(seg + 1); STAMPC(12); }
          rd(n0, 0, x0, w0);
#pragma unroll 1
          for (int n = n0; n < n1; ++n) {
            rd(n, 1, x1, w1);
            mm(x0, w0);
            ILV();
            __builtin_amdgcn_sched_barrier(0);
            asm volatile("s_waitcnt lgkmcnt(0)\n\ts_barrier" ::: "memory");
            rd(n + 1, 0, x0, w0);
            mm(x1, w1);
            ILV();
            __builtin_amdgcn_sched_barrier(0);
          }
          nb = n1 + 1;
          if (seg < 2) STAMPC(13);
        }
        STAMP(job == 0 ? 3 : 5);
        const u32 dtile = (myblk ^ (dquad != 0u ? 64u : 0u) ^ (dquad != 1u ? 32u : 0u)) + (u32)((g & 1) * 16 + (L & 1) * 8) + bb;
        const u32 obase = sgpr(job == 1 ? OFF_DS + dtile * 32768u
                        : (Lg == 0) ? OFF_PS + (seq & 1u) * 8u * MB + myh * 4u * MB + (myblk + (u32)((g & 1) * 16 + 8 * grp) + bb) * 32768u
                                   : OFF_TL + (u32)Lg * MB + (u32)(grp * 16 + g) * 32768u);
#pragma unroll
        for (int m = 0; m < 4; ++m)
#pragma unroll
          for (int hf = 0; hf < 2; ++hf) {
            u32x4 o;
            o[0] = nz(pk_bf16(acc[m][2 * hf][0], acc[m][2 * hf][1]));
            o[1] = nz(pk_bf16(acc[m][2 * hf][2], acc[m][2 * hf][3]));
            o[2] = nz(pk_bf16(acc[m][2 * hf + 1][0], acc[m][2 * hf + 1][1]));
            o[3] = nz(pk_bf16(acc[m][2 * hf + 1][2], acc[m][2 * hf + 1][3]));
            st16(o, rs_me, l16, obase + (u32)(((wr * 2 + (m >> 1)) * 2 + wc) * 4096 + ((m & 1) * 2 + hf) * 1024));
          }
        if (grp == 0) asm volatile("s_barrier\n\ts_barrier" ::: "memory");
        if (Lg != 0) {
          asm volatile("s_waitcnt vmcnt(0)" ::: "memory");
          __builtin_amdgcn_raw_buffer_store_b32(
              seq, rs_me, lane < 2 ? (u32)lane * 8u : OOB,
              (job == 1 ? OFF_DF + dtile * 32u : OFF_TF + (u32)Lg * 1024u + (u32)(grp * 16 + g) * 32u) + (u32)(wr * 4 + wc) * 4u, WT);
        }
        if (job == 0) STAMP(4);
      }
    }
  }
}

}

#define HIP_CHECK(expr) do { hipError_t _e = (expr); if (_e != hipSuccess) { \
    fprintf(stderr, "HIP error %s at %s:%d\n", hipGetErrorString(_e), __FILE__, __LINE__); \
    return; } } while (0)

extern "C" void kernel_launch(int rank, int nranks, ncclComm_t comm, hipStream_t stream, void* const* d_in,
                              const int* in_sizes, int n_in, void* d_out, int out_size, const int* in_dtypes,
                              int out_dtype, void* const* d_out_all, void* const* d_ws_all, size_t ws_bytes) {
  (void)comm; (void)in_dtypes; (void)out_dtype; (void)d_out_all;
  if (nranks != 8 || (unsigned)rank >= 8u || n_in != 2 || in_sizes[0] != 4096 * 512 || in_sizes[1] != 512 * 4096 ||
      out_size != 512 * 4096 || ws_bytes < (size_t)WS_USED) {
    fprintf(stderr, "kernel_launch: unexpected problem shape\n");
    return;
  }
  Params p;
  for (int i = 0; i < 8; ++i) p.wsx[i] = (char*)d_ws_all[rank ^ i];
  p.x = (const float*)d_in[0];
  p.w = (const float*)d_in[1];
  p.out = (signed char*)d_out;
  p.rank = rank;
  HIP_CHECK(hipFuncSetAttribute((const void*)fused_gemm_rs_q8, hipFuncAttributeMaxDynamicSharedMemorySize, LDS_BYTES));
  fused_gemm_rs_q8<<<dim3(256), dim3(NTHREADS), LDS_BYTES, stream>>>(p);
  HIP_CHECK(hipGetLastError());
}
```
